```python
import numpy as np

kernel_name = "gpu_gemm_a2a_m4096_k4096_n16384_bf16_qe4m3"


ABSMAX_RTOL = 0.09

COMPUTE_DTYPE = "bfloat16"
EPILOGUE = "quant_e4m3"
IN_QUANT = None
OUT_QUANT = "e4m3"
IN_DTYPES = {"x": "bfloat16", "w_mat": "bfloat16"}
WORLD_SIZE = 8

def _to_bf16(x):
    u = np.ascontiguousarray(x, dtype=np.float32).view(np.uint32)
    return ((u + 0x7FFF + ((u >> 16) & 1)) & 0xFFFF0000).view(np.float32)

def setup_inputs():

    x = _to_bf16(np.random.randn(4096, 4096).astype(np.float32))
    w_mat = _to_bf16((np.random.randn(4096, 16384) / np.sqrt(4096)).astype(np.float32))
    return {"x": x, "w_mat": w_mat}

def _snap_fp8(x, mant_bits, max_abs, min_normal):
    a = np.abs(x.astype(np.float64))
    _m, _e = np.frexp(a)
    q = np.where(a >= min_normal, np.ldexp(1.0, _e - 1 - mant_bits),
                 min_normal * 2.0 ** -mant_bits)
    snapped = np.minimum(np.round(a / q) * q, max_abs)
    return (np.sign(x) * snapped).astype(np.float32)
def _quant_dequant(y, amax):
    scale = amax / 448.0
    q = _snap_fp8(y / scale, 3, 448.0, 2.0 ** -6)
    return (q * scale).astype(np.float32)
def _epilogue(y):
    return _quant_dequant(y, np.abs(y).max())

def reference(x, w_mat):
    y = (x @ w_mat).astype(np.float32)
    return _epilogue(y)
```

```cpp
#include <hip/hip_runtime.h>
#include <rccl/rccl.h>
#include <cstdio>

#define HIP_CHECK(expr) do { hipError_t _e = (expr); if (_e != hipSuccess) { \
    fprintf(stderr, "HIP error %s at %s:%d\n", hipGetErrorString(_e), __FILE__, __LINE__); \
    return; } } while (0)

#define LAS __attribute__((address_space(3)))
#define GAS __attribute__((address_space(1)))
typedef __bf16 bf16x8 __attribute__((ext_vector_type(8)));
typedef short s16x4 __attribute__((ext_vector_type(4)));
typedef float f32x4 __attribute__((ext_vector_type(4)));
typedef unsigned u32x4 __attribute__((ext_vector_type(4)));
typedef unsigned short u16;
typedef unsigned long long u64;

namespace {
constexpr int NR = 8;
constexpr int MG = 4096, NG = 16384, KG = 4096;
constexpr int MS = MG / NR;
constexpr int NS = NG / NR;
constexpr int BM = 256, BN = 128, BK = 64, NT = KG / BK;
constexpr int XB = BM * BK * 2, WB = BK * BN * 2;
constexpr int NXS = 3, NWS = 4;
constexpr int W_OFF = NXS * XB;
constexpr int LDS_BYTES = NXS * XB + NWS * WB;
constexpr int NWG = 256;
constexpr u64 DEADLINE = 50000000ull;

constexpr size_t WS_CNT = 0;
constexpr size_t WS_TMO = 4096;
constexpr size_t WS_LOC = 8192;
constexpr size_t WS_GLB = 16384;
constexpr size_t WS_DONE = 32768;
constexpr size_t WS_END = WS_DONE + (size_t)NWG * 128;

struct Args {
  const u16* x; const u16* w;
  unsigned char* out[NR];
  unsigned char* ws[NR];
  int rank; int pad;
};
}

#define BAR() do { asm volatile("" ::: "memory"); __builtin_amdgcn_s_barrier(); asm volatile("" ::: "memory"); } while (0)
#define WAIT_V(n) asm volatile("s_waitcnt vmcnt(%0)" ::"n"(n) : "memory")
#define WAIT_L(n) asm volatile("s_waitcnt lgkmcnt(%0)" ::"n"(n) : "memory")
#define SCHED() __builtin_amdgcn_sched_barrier(0)
#define RLX_SYS __ATOMIC_RELAXED, __HIP_MEMORY_SCOPE_SYSTEM
#define RLX_AGT __ATOMIC_RELAXED, __HIP_MEMORY_SCOPE_AGENT

#define STAMP(i) do {} while (0)

struct Frags { bf16x8 x[8]; bf16x8 w[4]; };

__device__ __forceinline__ bf16x8 rd_x(LAS char* p) { return *(LAS bf16x8*)p; }
__device__ __forceinline__ bf16x8 rd_w(LAS char* p) {
  s16x4 a = __builtin_amdgcn_ds_read_tr16_b64_v4i16((LAS s16x4*)p);
  s16x4 b = __builtin_amdgcn_ds_read_tr16_b64_v4i16((LAS s16x4*)(p + 1024));
  return __builtin_bit_cast(bf16x8, __builtin_shufflevector(a, b, 0, 1, 2, 3, 4, 5, 6, 7));
}
__device__ __forceinline__ unsigned wave_umax(unsigned v) {
#pragma unroll
  for (int o = 32; o >= 1; o >>= 1) { unsigned t = __shfl_xor(v, o); v = t > v ? t : v; }
  return v;
}

__global__ __launch_bounds__(512, 2) void fused_gemm_a2a(Args A) {
  __shared__ __attribute__((aligned(16))) char smem[LDS_BYTES];
  LAS char* lds = (LAS char*)smem;
  const int tid = threadIdx.x, wid = __builtin_amdgcn_readfirstlane(tid >> 6), lane = tid & 63;
  const int bid = blockIdx.x, xcd = bid & 7, idx = bid >> 3;
  const int rank = A.rank;
  const int dest = (xcd + rank) & 7;
  const int tm = idx & 1, tc = idx >> 1;
  const int m0 = tm * BM, n0 = dest * NS + tc * BN;
  unsigned char* myws = A.ws[rank];
  LAS unsigned* red = (LAS unsigned*)lds;

  if (wid >= 4) {
    if (wid < 6) {
      const int j = wid - 4;
      auto rs = __builtin_amdgcn_make_buffer_rsrc((void*)(A.x + (size_t)(m0 + j * 128) * KG), 0, 128 * KG * 2, 0x00020000);
      const int v0 = (lane >> 3) * (KG * 2) + (((lane & 7) ^ (lane >> 4)) << 4);
      const int v1 = (lane >> 3) * (KG * 2) + (((lane & 7) ^ (4 | (lane >> 4))) << 4);
      LAS char* base = lds + j * 16384;
      auto issue = [&](int t, int slot) {
#pragma unroll
        for (int i = 0; i < 16; ++i)
          __builtin_amdgcn_raw_ptr_buffer_load_lds(rs, (LAS void*)(base + slot * XB + i * 1024), 16, (i & 1) ? v1 : v0, i * 8 * (KG * 2) + t * 128, 0, 0);
      };
      issue(0, 0); issue(1, 1);
      int slot = 2;
      WAIT_V(16); BAR();
      for (int t = 0; t < NT - 1; ++t) {
        if (t + 2 < NT) { issue(t + 2, slot); WAIT_V(16); } else WAIT_V(0);
        slot = slot == NXS - 1 ? 0 : slot + 1;
        BAR();
      }
      return;
    }
    if (wid == 6) {
      auto rs = __builtin_amdgcn_make_buffer_rsrc((void*)(A.w + n0), 0, (unsigned)((size_t)KG * NG * 2 - (size_t)n0 * 2), 0x00020000);
      const int v0 = (lane >> 4) * (NG * 2) + (((lane & 15) ^ ((lane >> 4) << 1)) << 4);
      const int v1 = (lane >> 4) * (NG * 2) + (((lane & 15) ^ ((4 | (lane >> 4)) << 1)) << 4);
      LAS char* base = lds + W_OFF;
      auto issue = [&](int t, int slot) {
#pragma unroll
        for (int i = 0; i < 16; ++i)
          __builtin_amdgcn_raw_ptr_buffer_load_lds(rs, (LAS void*)(base + slot * WB + i * 1024), 16, (i & 2) ? v1 : v0, (t * 64 + i * 4) * (NG * 2), 0, 0);
      };
      issue(0, 0); issue(1, 1); issue(2, 2);
      int slot = 3;
      WAIT_V(32); BAR();
      for (int t = 0; t < NT - 1; ++t) {
        if (t + 3 < NT) { issue(t + 3, slot); WAIT_V(32); }
        else if (t + 2 < NT) WAIT_V(16);
        else WAIT_V(0);
        slot = (slot + 1) & 3;
        BAR();
      }
      return;
    }
    STAMP(0);
    GAS unsigned* cnt = (GAS unsigned*)(myws + WS_CNT) + bid;
    GAS unsigned* tmo = (GAS unsigned*)(myws + WS_TMO);
    const unsigned call = __builtin_amdgcn_readfirstlane(__hip_atomic_load(cnt, RLX_AGT)) + 1u;
    if (lane == 0) __hip_atomic_store(cnt, call, RLX_AGT);
    const unsigned par = call & 1u;
    for (int t = 0; t < NT; ++t) BAR();
    BAR();
    BAR();
    STAMP(1);
    {
      unsigned m = red[lane & 3];
      m = max(m, (unsigned)__shfl_xor(m, 1)); m = max(m, (unsigned)__shfl_xor(m, 2));
      if (lane == 0) __hip_atomic_store((GAS u64*)(myws + WS_LOC) + par * NWG + bid, ((u64)call << 32) | m, RLX_AGT);
    }
    bool bad = false;
    const u64 t0 = __builtin_amdgcn_s_memrealtime();
    if (bid == 0) {
      GAS u64* g = (GAS u64*)(myws + WS_LOC) + par * NWG + lane;
      unsigned m;
      for (;;) {
        u64 a = __hip_atomic_load(g, RLX_AGT), b = __hip_atomic_load(g + 64, RLX_AGT), c = __hip_atomic_load(g + 128, RLX_AGT), d = __hip_atomic_load(g + 192, RLX_AGT);
        const bool ok = (unsigned)(a >> 32) == call && (unsigned)(b >> 32) == call && (unsigned)(c >> 32) == call && (unsigned)(d >> 32) == call;
        m = max(max((unsigned)a, (unsigned)b), max((unsigned)c, (unsigned)d));
        if (__all(ok)) break;
        if (__builtin_amdgcn_s_memrealtime() - t0 > DEADLINE) { bad = true; break; }
        __builtin_amdgcn_s_sleep(1);
      }
      m = wave_umax(m);
      if (!bad) {
        const unsigned p = lane >> 3, r = lane & 7;
        unsigned char* pw = A.ws[0];
#pragma unroll
        for (int i = 1; i < NR; ++i) pw = p == (unsigned)i ? A.ws[i] : pw;
        __hip_atomic_store((GAS u64*)(pw + WS_GLB + (par * 8 + r) * 128) + rank, ((u64)call << 32) | m, RLX_SYS);
      }
      STAMP(2);
    }
    unsigned gm = 0;
    {
      GAS u64* g = (GAS u64*)(myws + WS_GLB + (par * 8 + xcd) * 128) + (lane & 7);
      for (;;) {
        const u64 a = __hip_atomic_load(g, RLX_SYS);
        const unsigned tv = __hip_atomic_load(tmo, RLX_AGT);
        gm = (unsigned)a;
        if (tv == call) { bad = true; break; }
        if (__all((unsigned)(a >> 32) == call)) break;
        if (bad || __builtin_amdgcn_s_memrealtime() - t0 > DEADLINE) { bad = true; break; }
        __builtin_amdgcn_s_sleep(1);
      }
      gm = max(gm, (unsigned)__shfl_xor(gm, 1)); gm = max(gm, (unsigned)__shfl_xor(gm, 2)); gm = max(gm, (unsigned)__shfl_xor(gm, 4));
    }
    if (bad && lane == 0) __hip_atomic_store(tmo, call, RLX_AGT);
    if (lane == 0) {
      const float s = __uint_as_float(gm) / 448.0f;
      red[4] = bad ? 0x7fc00000u : __float_as_uint(1.0f / s);
    }
    WAIT_L(0);
    BAR();
    STAMP(3);
    BAR();
    STAMP(4);
    if (dest != rank && lane == 0)
      __hip_atomic_store((GAS unsigned*)(A.ws[dest] + WS_DONE + (size_t)(rank * 32 + idx) * 128), call, RLX_SYS);
    if ((bid >> 5) != rank) {
      GAS unsigned* f = (GAS unsigned*)(myws + WS_DONE + (size_t)bid * 128);
      const u64 t1 = __builtin_amdgcn_s_memrealtime();
      for (;;) {
        const unsigned v = __hip_atomic_load(f, RLX_SYS);
        if (v >= call) break;
        if (__hip_atomic_load(tmo, RLX_AGT) == call || __builtin_amdgcn_s_memrealtime() - t1 > DEADLINE) {
          if (lane == 0) { __hip_atomic_store(tmo, call, RLX_AGT); A.out[rank][(size_t)rank * MS * NS + bid] = 0x7f; }
          break;
        }
        __builtin_amdgcn_s_sleep(1);
      }
    }
    STAMP(5);
    return;
  }

  const int wm = wid >> 1, wn = wid & 1;
  const int fr = lane & 15, g = lane >> 4;
  const int xa0 = (wm * 128 + fr) * 128 + ((g ^ ((fr >> 1) & 7)) << 4);
  const int xa1 = xa0 ^ 64;
  const int q = fr >> 2, p = fr & 3, h = q | ((g & 1) << 2);
  int wa[4];
#pragma unroll
  for (int nt = 0; nt < 4; ++nt) wa[nt] = W_OFF + (8 * g + q) * 256 + (((4 * wn + nt) ^ h) << 5) + p * 8;
  f32x4 acc[8][4];
#pragma unroll
  for (int a = 0; a < 8; ++a)
#pragma unroll
    for (int b = 0; b < 4; ++b) acc[a][b] = (f32x4){0.f, 0.f, 0.f, 0.f};
  Frags P, Q;
  auto load = [&](Frags& F, int xoff, int woff, int ks) {
    LAS char* xp = lds + (ks ? xa1 : xa0) + xoff;
#pragma unroll
    for (int i = 0; i < 4; ++i) { F.w[i] = rd_w(lds + wa[i] + woff + ks * 8192); F.x[i] = rd_x(xp + i * 2048); }
#pragma unroll
    for (int i = 4; i < 8; ++i) F.x[i] = rd_x(xp + i * 2048);
  };
  auto mma = [&](Frags& F) {
#pragma unroll
    for (int a = 0; a < 8; ++a)
#pragma unroll
      for (int b = 0; b < 4; ++b) acc[a][b] = __builtin_amdgcn_mfma_f32_16x16x32_bf16(F.w[b], F.x[a], acc[a][b], 0, 0, 0);
  };
  auto interleave = [&]() {
#pragma unroll
    for (int i = 0; i < 8; ++i) {
      __builtin_amdgcn_sched_group_barrier(0x8, 1, 0);
      __builtin_amdgcn_sched_group_barrier(0x100, 1, 0);
      __builtin_amdgcn_sched_group_barrier(0x8, 1, 0);
      __builtin_amdgcn_sched_group_barrier(0x100, 1, 0);
      __builtin_amdgcn_sched_group_barrier(0x8, 1, 0);
    }
    __builtin_amdgcn_sched_group_barrier(0x8, 8, 0);
  };
  int xoff = 0, woff = 0;
  BAR();
  load(P, 0, 0, 0);
  SCHED();
  load(Q, 0, 0, 1); mma(P); interleave();
  SCHED();
  for (int t = 1; t < NT; ++t) {
    xoff = xoff == (NXS - 1) * XB ? 0 : xoff + XB;
    woff = (woff + WB) & (NWS * WB - 1);
    WAIT_L(0);
    BAR();
    SCHED();
    load(P, xoff, woff, 0); mma(Q); interleave();
    SCHED();
    load(Q, xoff, woff, 1); mma(P); interleave();
    SCHED();
  }
  WAIT_L(0);
  BAR();
  mma(Q);
  float mx = 0.f;
#pragma unroll
  for (int a = 0; a < 8; ++a)
#pragma unroll
    for (int b = 0; b < 4; ++b) {
      mx = fmaxf(mx, fmaxf(fabsf(acc[a][b][0]), fabsf(acc[a][b][1])));
      mx = fmaxf(mx, fmaxf(fabsf(acc[a][b][2]), fabsf(acc[a][b][3])));
    }
  {
    const unsigned m = wave_umax(__float_as_uint(mx));
    if (lane == 0) red[wid] = m;
  }
  WAIT_L(0);
  BAR();
  BAR();
  const float inv = __uint_as_float(red[4]);
  unsigned char* op = A.out[dest] + (size_t)(rank * MS + m0 + wm * 128 + fr) * NS + tc * BN + wn * 64 + g * 16;
#pragma unroll
  for (int a = 0; a < 8; ++a) {
    unsigned d[4];
#pragma unroll
    for (int b = 0; b < 4; ++b) {
      const f32x4 v = acc[a][b] * inv;
      int pk = __builtin_amdgcn_cvt_pk_fp8_f32(v[0], v[1], 0, false);
      pk = __builtin_amdgcn_cvt_pk_fp8_f32(v[2], v[3], pk, true);
      d[b] = pk;
    }
    { auto r = __builtin_amdgcn_permlane32_swap(d[0], d[2], false, false); d[0] = r[0]; d[2] = r[1]; }
    { auto r = __builtin_amdgcn_permlane32_swap(d[1], d[3], false, false); d[1] = r[0]; d[3] = r[1]; }
    { auto r = __builtin_amdgcn_permlane16_swap(d[0], d[1], false, false); d[0] = r[0]; d[1] = r[1]; }
    { auto r = __builtin_amdgcn_permlane16_swap(d[2], d[3], false, false); d[2] = r[0]; d[3] = r[1]; }
    const u32x4 o = {d[0], d[1], d[2], d[3]};
    asm volatile("global_store_dwordx4 %0, %1, off sc0 sc1\n\ts_nop 1" ::"v"(op + (size_t)a * 16 * NS), "v"(o) : "memory");
  }
  WAIT_V(0);
  BAR();
}

extern "C" void kernel_launch(int rank, int nranks, ncclComm_t comm, hipStream_t stream,
                              void* const* d_in, const int* in_sizes, int n_in,
                              void* d_out, int out_size, const int* in_dtypes, int out_dtype,
                              void* const* d_out_all, void* const* d_ws_all, size_t ws_bytes) {
  (void)comm; (void)in_sizes; (void)n_in; (void)d_out; (void)out_size; (void)in_dtypes; (void)out_dtype;
  if (nranks != NR || (unsigned)rank >= (unsigned)NR || ws_bytes < (2u << 20)) { fprintf(stderr, "unexpected configuration\n"); return; }
  Args a;
  a.x = (const u16*)d_in[0]; a.w = (const u16*)d_in[1];
  for (int i = 0; i < NR; ++i) { a.out[i] = (unsigned char*)d_out_all[i]; a.ws[i] = (unsigned char*)d_ws_all[i]; }
  a.rank = rank; a.pad = 0;
  hipLaunchKernelGGL(fused_gemm_a2a, dim3(NWG), dim3(512), 0, stream, a);
  HIP_CHECK(hipGetLastError());
}
```

```cpp
#include <hip/hip_runtime.h>
#include <rccl/rccl.h>
#include <cstdio>

#define HIP_CHECK(expr) do { hipError_t _e = (expr); if (_e != hipSuccess) { \
    fprintf(stderr, "HIP error %s at %s:%d\n", hipGetErrorString(_e), __FILE__, __LINE__); \
    return; } } while (0)

#define LAS __attribute__((address_space(3)))
#define GAS __attribute__((address_space(1)))
typedef __bf16 bf16x8 __attribute__((ext_vector_type(8)));
typedef short s16x4 __attribute__((ext_vector_type(4)));
typedef float f32x4 __attribute__((ext_vector_type(4)));
typedef unsigned u32x4 __attribute__((ext_vector_type(4)));
typedef unsigned short u16;
typedef unsigned long long u64;

namespace {
constexpr int NR = 8;
constexpr int MG = 4096, NG = 16384, KG = 4096;
constexpr int MS = MG / NR;
constexpr int NS = NG / NR;
constexpr int BM = 256, BN = 128, BK = 64, NT = KG / BK;
constexpr int XB = BM * BK * 2, WB = BK * BN * 2;
constexpr int NXS = 3, NWS = 4;
constexpr int W_OFF = NXS * XB;
constexpr int LDS_BYTES = NXS * XB + NWS * WB;
constexpr int NWG = 256;
constexpr u64 DEADLINE = 50000000ull;

constexpr size_t WS_CNT = 0;
constexpr size_t WS_TMO = 4096;
constexpr size_t WS_LOC = 8192;
constexpr size_t WS_GLB = 16384;
constexpr size_t WS_DONE = 32768;
constexpr size_t WS_END = WS_DONE + (size_t)NWG * 128;

struct Args {
  const u16* x; const u16* w;
  unsigned char* out[NR];
  unsigned char* ws[NR];
  int rank; int pad;
};
}

#define WAUX 2
#define WSL 5
#define BAR() do { asm volatile("" ::: "memory"); __builtin_amdgcn_s_barrier(); asm volatile("" ::: "memory"); } while (0)
#define WAIT_V(n) asm volatile("s_waitcnt vmcnt(%0)" ::"n"(n) : "memory")
#define WAIT_L(n) asm volatile("s_waitcnt lgkmcnt(%0)" ::"n"(n) : "memory")
#define SCHED() __builtin_amdgcn_sched_barrier(0)
#define RLX_SYS __ATOMIC_RELAXED, __HIP_MEMORY_SCOPE_SYSTEM
#define RLX_AGT __ATOMIC_RELAXED, __HIP_MEMORY_SCOPE_AGENT

#define STAMP(i) do {} while (0)

struct Frags { bf16x8 x[8]; bf16x8 w[4]; };

__device__ __forceinline__ bf16x8 rd_x(LAS char* p) { return *(LAS bf16x8*)p; }
__device__ __forceinline__ bf16x8 rd_w(LAS char* p) {
  s16x4 a = __builtin_amdgcn_ds_read_tr16_b64_v4i16((LAS s16x4*)p);
  s16x4 b = __builtin_amdgcn_ds_read_tr16_b64_v4i16((LAS s16x4*)(p + 1024));
  return __builtin_bit_cast(bf16x8, __builtin_shufflevector(a, b, 0, 1, 2, 3, 4, 5, 6, 7));
}
__device__ __forceinline__ unsigned wave_umax(unsigned v) {
#pragma unroll
  for (int o = 32; o >= 1; o >>= 1) { unsigned t = __shfl_xor(v, o); v = t > v ? t : v; }
  return v;
}

__global__ __launch_bounds__(512, 2) void fused_gemm_a2a(Args A) {
  __shared__ __attribute__((aligned(16))) char smem[LDS_BYTES];
  LAS char* lds = (LAS char*)smem;
  const int tid = threadIdx.x, wid = __builtin_amdgcn_readfirstlane(tid >> 6), lane = tid & 63;
  const int bid = blockIdx.x, xcd = bid & 7, idx = bid >> 3;
  const int rank = A.rank;
  const int dest = (xcd + rank) & 7;
  const int tm = idx & 1, tc = idx >> 1;
  const int m0 = tm * BM, n0 = dest * NS + tc * BN;
  const int koff = xcd * 8;
  unsigned char* myws = A.ws[rank];
  LAS unsigned* red = (LAS unsigned*)lds;

  if (wid >= 4) {
    if (wid < 6) {
      const int j = wid - 4;
      auto rs = __builtin_amdgcn_make_buffer_rsrc((void*)(A.x + (size_t)(m0 + j * 128) * KG), 0, 128 * KG * 2, 0x00020000);
      const int v0 = (lane >> 3) * (KG * 2) + (((lane & 7) ^ (lane >> 4)) << 4);
      const int v1 = (lane >> 3) * (KG * 2) + (((lane & 7) ^ (4 | (lane >> 4))) << 4);
      LAS char* base = lds + j * 16384;
      auto issue = [&](int t, int slot) {
#pragma unroll
        for (int i = 0; i < 16; ++i)
          __builtin_amdgcn_raw_ptr_buffer_load_lds(rs, (LAS void*)(base + slot * XB + i * 1024), 16, (i & 1) ? v1 : v0, i * 8 * (KG * 2) + ((t + koff) & 63) * 128, 0, 0);
      };
      issue(0, 0); issue(1, 1);
      int slot = 2;
      WAIT_V(16); BAR();
      for (int t = 0; t < NT - 1; ++t) {
        if (t + 2 < NT) { issue(t + 2, slot); WAIT_V(16); } else WAIT_V(0);
        slot = slot == NXS - 1 ? 0 : slot + 1;
        BAR();
      }
      return;
    }
    if (wid == 6) {
      auto rs = __builtin_amdgcn_make_buffer_rsrc((void*)(A.w + n0), 0, (unsigned)((size_t)KG * NG * 2 - (size_t)n0 * 2), 0x00020000);
      const int v0 = (lane >> 4) * (NG * 2) + (((lane & 15) ^ ((lane >> 4) << 1)) << 4);
      const int v1 = (lane >> 4) * (NG * 2) + (((lane & 15) ^ ((4 | (lane >> 4)) << 1)) << 4);
      LAS char* base = lds + W_OFF;
      auto issue = [&](int t, int slot) {
#pragma unroll
        for (int i = 0; i < 16; ++i)
          __builtin_amdgcn_raw_ptr_buffer_load_lds(rs, (LAS void*)(base + slot * WB + i * 1024), 16, (i & 2) ? v1 : v0, (((t + koff) & 63) * 64 + i * 4) * (NG * 2), 0, WAUX);
      };
      issue(0, 0); issue(1, 1);
      WAIT_V(16); BAR();
      for (int t = 0; t < NT; t += 2) {
        if (t + 2 < NT) {
          __builtin_amdgcn_s_sleep(WSL);
          issue(t + 2, (t + 2) & 3); issue(t + 3, (t + 3) & 3);
          WAIT_V(32); BAR();
          WAIT_V(16); BAR();
        } else { WAIT_V(0); BAR(); }
      }
      return;
    }
    STAMP(0);
    GAS unsigned* cnt = (GAS unsigned*)(myws + WS_CNT) + bid;
    GAS unsigned* tmo = (GAS unsigned*)(myws + WS_TMO);
    const unsigned call = __builtin_amdgcn_readfirstlane(__hip_atomic_load(cnt, RLX_AGT)) + 1u;
    if (lane == 0) __hip_atomic_store(cnt, call, RLX_AGT);
    const unsigned par = call & 1u;
    for (int t = 0; t < NT; ++t) BAR();
    BAR();
    BAR();
    STAMP(1);
    {
      unsigned m = red[lane & 3];
      m = max(m, (unsigned)__shfl_xor(m, 1)); m = max(m, (unsigned)__shfl_xor(m, 2));
      if (lane == 0) __hip_atomic_store((GAS u64*)(myws + WS_LOC) + par * NWG + bid, ((u64)call << 32) | m, RLX_AGT);
    }
    bool bad = false;
    const u64 t0 = __builtin_amdgcn_s_memrealtime();
    if (bid == 0) {
      GAS u64* g = (GAS u64*)(myws + WS_LOC) + par * NWG + lane;
      unsigned m;
      for (;;) {
        u64 a = __hip_atomic_load(g, RLX_AGT), b = __hip_atomic_load(g + 64, RLX_AGT), c = __hip_atomic_load(g + 128, RLX_AGT), d = __hip_atomic_load(g + 192, RLX_AGT);
        const bool ok = (unsigned)(a >> 32) == call && (unsigned)(b >> 32) == call && (unsigned)(c >> 32) == call && (unsigned)(d >> 32) == call;
        m = max(max((unsigned)a, (unsigned)b), max((unsigned)c, (unsigned)d));
        if (__all(ok)) break;
        if (__builtin_amdgcn_s_memrealtime() - t0 > DEADLINE) { bad = true; break; }
        __builtin_amdgcn_s_sleep(1);
      }
      m = wave_umax(m);
      if (!bad) {
        const unsigned p = lane >> 3, r = lane & 7;
        unsigned char* pw = A.ws[0];
#pragma unroll
        for (int i = 1; i < NR; ++i) pw = p == (unsigned)i ? A.ws[i] : pw;
        __hip_atomic_store((GAS u64*)(pw + WS_GLB + (par * 8 + r) * 128) + rank, ((u64)call << 32) | m, RLX_SYS);
      }
      STAMP(2);
    }
    unsigned gm = 0;
    {
      GAS u64* g = (GAS u64*)(myws + WS_GLB + (par * 8 + xcd) * 128) + (lane & 7);
      for (;;) {
        const u64 a = __hip_atomic_load(g, RLX_SYS);
        const unsigned tv = __hip_atomic_load(tmo, RLX_AGT);
        gm = (unsigned)a;
        if (tv == call) { bad = true; break; }
        if (__all((unsigned)(a >> 32) == call)) break;
        if (bad || __builtin_amdgcn_s_memrealtime() - t0 > DEADLINE) { bad = true; break; }
        __builtin_amdgcn_s_sleep(1);
      }
      gm = max(gm, (unsigned)__shfl_xor(gm, 1)); gm = max(gm, (unsigned)__shfl_xor(gm, 2)); gm = max(gm, (unsigned)__shfl_xor(gm, 4));
    }
    if (bad && lane == 0) __hip_atomic_store(tmo, call, RLX_AGT);
    if (lane == 0) {
      const float s = __uint_as_float(gm) / 448.0f;
      red[4] = bad ? 0x7fc00000u : __float_as_uint(1.0f / s);
    }
    WAIT_L(0);
    BAR();
    STAMP(3);
    BAR();
    STAMP(4);
    if (dest != rank && lane == 0)
      __hip_atomic_store((GAS unsigned*)(A.ws[dest] + WS_DONE + (size_t)(rank * 32 + idx) * 128), call, RLX_SYS);
    if ((bid >> 5) != rank) {
      GAS unsigned* f = (GAS unsigned*)(myws + WS_DONE + (size_t)bid * 128);
      const u64 t1 = __builtin_amdgcn_s_memrealtime();
      for (;;) {
        const unsigned v = __hip_atomic_load(f, RLX_SYS);
        if (v >= call) break;
        if (__hip_atomic_load(tmo, RLX_AGT) == call || __builtin_amdgcn_s_memrealtime() - t1 > DEADLINE) {
          if (lane == 0) { __hip_atomic_store(tmo, call, RLX_AGT); A.out[rank][(size_t)rank * MS * NS + bid] = 0x7f; }
          break;
        }
        __builtin_amdgcn_s_sleep(1);
      }
    }
    STAMP(5);
    return;
  }

  const int wm = wid >> 1, wn = wid & 1;
  const int fr = lane & 15, g = lane >> 4;
  const int xa0 = (wm * 128 + fr) * 128 + ((g ^ ((fr >> 1) & 7)) << 4);
  const int xa1 = xa0 ^ 64;
  const int q = fr >> 2, p = fr & 3, h = q | ((g & 1) << 2);
  int wa[4];
#pragma unroll
  for (int nt = 0; nt < 4; ++nt) wa[nt] = W_OFF + (8 * g + q) * 256 + (((4 * wn + nt) ^ h) << 5) + p * 8;
  f32x4 acc[8][4];
#pragma unroll
  for (int a = 0; a < 8; ++a)
#pragma unroll
    for (int b = 0; b < 4; ++b) acc[a][b] = (f32x4){0.f, 0.f, 0.f, 0.f};
  Frags P, Q;
  auto load = [&](Frags& F, int xoff, int woff, int ks) {
    LAS char* xp = lds + (ks ? xa1 : xa0) + xoff;
#pragma unroll
    for (int i = 0; i < 4; ++i) { F.w[i] = rd_w(lds + wa[i] + woff + ks * 8192); F.x[i] = rd_x(xp + i * 2048); }
#pragma unroll
    for (int i = 4; i < 8; ++i) F.x[i] = rd_x(xp + i * 2048);
  };
  auto mma = [&](Frags& F) {
#pragma unroll
    for (int a = 0; a < 8; ++a)
#pragma unroll
      for (int b = 0; b < 4; ++b) acc[a][b] = __builtin_amdgcn_mfma_f32_16x16x32_bf16(F.w[b], F.x[a], acc[a][b], 0, 0, 0);
  };
  auto interleave = [&]() {
#pragma unroll
    for (int i = 0; i < 8; ++i) {
      __builtin_amdgcn_sched_group_barrier(0x8, 1, 0);
      __builtin_amdgcn_sched_group_barrier(0x100, 1, 0);
      __builtin_amdgcn_sched_group_barrier(0x8, 1, 0);
      __builtin_amdgcn_sched_group_barrier(0x100, 1, 0);
      __builtin_amdgcn_sched_group_barrier(0x8, 1, 0);
    }
    __builtin_amdgcn_sched_group_barrier(0x8, 8, 0);
  };
  int xoff = 0, woff = 0;
  BAR();
  load(P, 0, 0, 0);
  SCHED();
  load(Q, 0, 0, 1); mma(P); interleave();
  SCHED();
  for (int t = 1; t < NT; ++t) {
    xoff = xoff == (NXS - 1) * XB ? 0 : xoff + XB;
    woff = (woff + WB) & (NWS * WB - 1);
    WAIT_L(0);
    BAR();
    SCHED();
    load(P, xoff, woff, 0); mma(Q); interleave();
    SCHED();
    load(Q, xoff, woff, 1); mma(P); interleave();
    SCHED();
  }
  WAIT_L(0);
  BAR();
  mma(Q);
  float mx = 0.f;
#pragma unroll
  for (int a = 0; a < 8; ++a)
#pragma unroll
    for (int b = 0; b < 4; ++b) {
      mx = fmaxf(mx, fmaxf(fabsf(acc[a][b][0]), fabsf(acc[a][b][1])));
      mx = fmaxf(mx, fmaxf(fabsf(acc[a][b][2]), fabsf(acc[a][b][3])));
    }
  {
    const unsigned m = wave_umax(__float_as_uint(mx));
    if (lane == 0) red[wid] = m;
  }
  WAIT_L(0);
  BAR();
  BAR();
  const float inv = __uint_as_float(red[4]);
  unsigned char* op = A.out[dest] + (size_t)(rank * MS + m0 + wm * 128 + fr) * NS + tc * BN + wn * 64 + g * 16;
#pragma unroll
  for (int a = 0; a < 8; ++a) {
    unsigned d[4];
#pragma unroll
    for (int b = 0; b < 4; ++b) {
      const f32x4 v = acc[a][b] * inv;
      int pk = __builtin_amdgcn_cvt_pk_fp8_f32(v[0], v[1], 0, false);
      pk = __builtin_amdgcn_cvt_pk_fp8_f32(v[2], v[3], pk, true);
      d[b] = pk;
    }
    { auto r = __builtin_amdgcn_permlane32_swap(d[0], d[2], false, false); d[0] = r[0]; d[2] = r[1]; }
    { auto r = __builtin_amdgcn_permlane32_swap(d[1], d[3], false, false); d[1] = r[0]; d[3] = r[1]; }
    { auto r = __builtin_amdgcn_permlane16_swap(d[0], d[1], false, false); d[0] = r[0]; d[1] = r[1]; }
    { auto r = __builtin_amdgcn_permlane16_swap(d[2], d[3], false, false); d[2] = r[0]; d[3] = r[1]; }
    const u32x4 o = {d[0], d[1], d[2], d[3]};
    asm volatile("global_store_dwordx4 %0, %1, off sc0 sc1\n\ts_nop 1" ::"v"(op + (size_t)a * 16 * NS), "v"(o) : "memory");
  }
  WAIT_V(0);
  BAR();
}

extern "C" void kernel_launch(int rank, int nranks, ncclComm_t comm, hipStream_t stream,
                              void* const* d_in, const int* in_sizes, int n_in,
                              void* d_out, int out_size, const int* in_dtypes, int out_dtype,
                              void* const* d_out_all, void* const* d_ws_all, size_t ws_bytes) {
  (void)comm; (void)in_sizes; (void)n_in; (void)d_out; (void)out_size; (void)in_dtypes; (void)out_dtype;
  if (nranks != NR || (unsigned)rank >= (unsigned)NR || ws_bytes < (2u << 20)) { fprintf(stderr, "unexpected configuration\n"); return; }
  Args a;
  a.x = (const u16*)d_in[0]; a.w = (const u16*)d_in[1];
  for (int i = 0; i < NR; ++i) { a.out[i] = (unsigned char*)d_out_all[i]; a.ws[i] = (unsigned char*)d_ws_all[i]; }
  a.rank = rank; a.pad = 0;
  hipLaunchKernelGGL(fused_gemm_a2a, dim3(NWG), dim3(512), 0, stream, a);
  HIP_CHECK(hipGetLastError());
}
```

```cpp
#include <hip/hip_runtime.h>
#include <rccl/rccl.h>
#include <cstdio>

#define HIP_CHECK(expr) do { hipError_t _e = (expr); if (_e != hipSuccess) { \
    fprintf(stderr, "HIP error %s at %s:%d\n", hipGetErrorString(_e), __FILE__, __LINE__); \
    return; } } while (0)

#define LAS __attribute__((address_space(3)))
#define GAS __attribute__((address_space(1)))
typedef __bf16 bf16x8 __attribute__((ext_vector_type(8)));
typedef short s16x4 __attribute__((ext_vector_type(4)));
typedef float f32x4 __attribute__((ext_vector_type(4)));
typedef unsigned u32x4 __attribute__((ext_vector_type(4)));
typedef unsigned short u16;
typedef unsigned long long u64;

#define NF 3
#define PACE 790
#define POFF 100
#define WSL 5
#define WSLS 7
#define WSLF 4
#define FTH 6
#define QTH 0
#define PACEF 690
#define PACED 39
#define XT 1
#define KOFF 8
#define ADAPT 1
#define TS 64
#define PACE2 PACE

namespace {
constexpr int NR = 8;
constexpr int MG = 4096, NG = 16384, KG = 4096;
constexpr int MS = MG / NR;
constexpr int NS = NG / NR;
constexpr int BM = 256, BN = 128, BK = 64, NT = KG / BK;
constexpr int XB = BM * BK * 2, WB = BK * BN * 2;
constexpr int NXS = 3, NWS = 4;
constexpr int X_OFF = NWS * WB;
constexpr int LDS_BYTES = NXS * XB + NWS * WB;
constexpr int NWG = 256;
constexpr int TPP = 32;
constexpr u64 DEADLINE = 50000000ull;

constexpr size_t WS_CNT = 0;
constexpr size_t WS_TMO = 4096;
constexpr size_t WS_LAST = 4224;
constexpr size_t WS_LOC = 8192;
constexpr size_t WS_GLB = 16384;
constexpr size_t WS_DONE = 32768;
constexpr size_t WS_YF = 36864;
constexpr size_t WS_YS = 2u << 20;
constexpr size_t YS_TILE = (size_t)BM * BN * 2;
constexpr size_t WS_END = WS_YS + (size_t)2 * NR * TPP * YS_TILE;
static_assert(NF <= TPP && WS_YF + NR * TPP * 128 <= (1u << 20), "layout");

struct Args {
  unsigned char* ws[NR];
  unsigned char* out[NR];
  const u16* x; const u16* w;
  int rank; int pad;
};
}

#define BAR() do { asm volatile("" ::: "memory"); __builtin_amdgcn_s_barrier(); asm volatile("" ::: "memory"); } while (0)
#define WAIT_V(n) asm volatile("s_waitcnt vmcnt(%0)" ::"n"(n) : "memory")
#define WAIT_L(n) asm volatile("s_waitcnt lgkmcnt(%0)" ::"n"(n) : "memory")
#define SCHED() __builtin_amdgcn_sched_barrier(0)
#define RLX_SYS __ATOMIC_RELAXED, __HIP_MEMORY_SCOPE_SYSTEM
#define RLX_AGT __ATOMIC_RELAXED, __HIP_MEMORY_SCOPE_AGENT
#define NOW() __builtin_amdgcn_s_memrealtime()

#define STAMP(i) do {} while (0)

template <int CTRL> __device__ __forceinline__ unsigned dpp_umax(unsigned v) {
  const unsigned t = __builtin_amdgcn_update_dpp(v, v, CTRL, 0xf, 0xf, false);
  return t > v ? t : v;
}
__device__ __forceinline__ unsigned row_umax(unsigned v) {
  v = dpp_umax<0xB1>(v); v = dpp_umax<0x4E>(v); v = dpp_umax<0x141>(v); return dpp_umax<0x140>(v);
}
__device__ __forceinline__ unsigned wave_umax(unsigned v) {
  v = row_umax(v);
  const unsigned a = __builtin_amdgcn_readlane(v, 0), b = __builtin_amdgcn_readlane(v, 16), c = __builtin_amdgcn_readlane(v, 32), d = __builtin_amdgcn_readlane(v, 48);
  return max(max(a, b), max(c, d));
}

__global__ __launch_bounds__(512, 2) void fused_gemm_a2a(Args A) {
  __shared__ __attribute__((aligned(16))) char smem[LDS_BYTES];
  LAS char* lds = (LAS char*)smem;
  const int tid = threadIdx.x, wid = __builtin_amdgcn_readfirstlane(tid >> 6), lane = tid & 63;
  const int bid = blockIdx.x, xcd = bid & 7, idx = (bid >> 3) & 31;
  const int rank = A.rank;
  const int half = idx & 1;
  const int drel = xcd;
  const int tc = idx >> 1;
  const int n0 = ((rank + drel) & 7) * NS + tc * BN;
  const int koff = xcd * KOFF;
  auto early_index = [](int c) { return (c & 3) == 1 ? 99 : c - ((c + 2) >> 2); };
  const int fidx = early_index(tc);
  const bool early = drel != 0 && fidx < NF;
  const int tile = half * 16 + tc;
  unsigned char* myws = A.ws[0];
  LAS unsigned* red = (LAS unsigned*)lds;

  if (wid >= 4) {
    if (wid < 6) {
      const int j = wid - 4;
      LAS char* base = lds + X_OFF + j * 16384;
      auto rs = __builtin_amdgcn_make_buffer_rsrc((void*)(A.x + (size_t)(half * BM + j * 128) * KG), 0, 128 * KG * 2, 0x00020000);
      const int v0 = (lane >> 3) * (KG * 2) + (((lane & 7) ^ (lane >> 4)) << 4);
      const int v1 = (lane >> 3) * (KG * 2) + (((lane & 7) ^ (4 | (lane >> 4))) << 4);
      auto issue = [&](int t, int slot) {
#pragma unroll
        for (int i = 0; i < 16; ++i)
          __builtin_amdgcn_raw_ptr_buffer_load_lds(rs, (LAS void*)(base + slot * XB + i * 1024), 16, (i & 1) ? v1 : v0, i * 8 * (KG * 2) + ((t + koff) & 63) * 128, 0, 0);
      };
      issue(0, 0); issue(1, 1);
      int slot = 2;
      WAIT_V(16); BAR();
      for (int t = 0; t < NT - 1; ++t) {
        if (t + 2 < NT) { issue(t + 2, slot); WAIT_V(16); } else WAIT_V(0);
        slot = slot == NXS - 1 ? 0 : slot + 1;
        BAR();
      }
      return;
    }
    if (wid == 6) {
      auto rs = __builtin_amdgcn_make_buffer_rsrc((void*)(A.w + n0), 0, (unsigned)((size_t)KG * NG * 2 - (size_t)n0 * 2), 0x00020000);
      const int v0 = (lane >> 4) * (NG * 2) + (((lane & 15) ^ ((lane >> 4) << 1)) << 4);
      const int v1 = (lane >> 4) * (NG * 2) + (((lane & 15) ^ ((4 | (lane >> 4)) << 1)) << 4);
      auto issue = [&](int t, int slot, bool stream) {
        if (stream) {
#pragma unroll
          for (int i = 0; i < 16; ++i)
            __builtin_amdgcn_raw_ptr_buffer_load_lds(rs, (LAS void*)(lds + slot * WB + i * 1024), 16, (i & 2) ? v1 : v0, (((t + koff) & 63) * 64 + i * 4) * (NG * 2), 0, 2);
        } else {
#pragma unroll
          for (int i = 0; i < 16; ++i)
            __builtin_amdgcn_raw_ptr_buffer_load_lds(rs, (LAS void*)(lds + slot * WB + i * 1024), 16, (i & 2) ? v1 : v0, (((t + koff) & 63) * 64 + i * 4) * (NG * 2), 0, 0);
        }
      };
      issue(0, 0, false); issue(1, 1, false);
      unsigned last;
      asm volatile("global_load_dword %0, %1, off sc1" : "=v"(last) : "v"(myws + WS_LAST) : "memory");
      WAIT_V(17); BAR();
      __builtin_amdgcn_s_sleep(WSL);
      issue(2, 2, false); issue(3, 3, false);
      asm volatile("s_waitcnt vmcnt(32)" : "+v"(last) :: "memory");
      BAR();
      const bool stream = ADAPT ? (unsigned)NOW() - (unsigned)__builtin_amdgcn_readfirstlane(last) > 30000u : true;
      WAIT_V(16); BAR();
      auto run = [&](bool st) {
        for (int t = 2; t < NT - 2; t += 2) {
          if (fidx >= 11) __builtin_amdgcn_s_sleep(WSLS); else if (early) __builtin_amdgcn_s_sleep(WSLF); else __builtin_amdgcn_s_sleep(WSL);
          issue(t + 2, (t + 2) & 3, st); issue(t + 3, (t + 3) & 3, st);
          WAIT_V(32); BAR();
          WAIT_V(16); BAR();
        }
      };
      if (stream) run(true); else run(false);
      WAIT_V(0); BAR();
      return;
    }
    STAMP(0);
    const u64 t0 = NOW();
    GAS unsigned* cnt = (GAS unsigned*)(myws + WS_CNT) + bid;
    GAS unsigned* tmo = (GAS unsigned*)(myws + WS_TMO);
    if (XT) {
      const char* tp = (const char*)A.x + (size_t)(idx * 16 + (lane & 7)) * (KG * 2) + (xcd * 8 + (lane >> 3)) * 128;
      asm volatile("global_load_dword v253, %0, off\n\tglobal_load_dword v253, %1, off" ::"v"(tp), "v"(tp + 8 * KG * 2) : "memory", "v253");
    }
    const int pace = early ? PACEF + fidx * PACED : PACE;
    for (int t = 0; t < NT; ++t) {
      const u64 due = t0 + POFF + (unsigned)(t * pace - (t > TS && !early ? (t - TS) * (PACE - PACE2) : 0)) / 10u;
      while (NOW() < due) __builtin_amdgcn_s_sleep(1);
      BAR();
      if (t == 0) {
        __builtin_amdgcn_s_sleep(8);
        const GAS void* tab = (const GAS char*)(const GAS void*)__builtin_amdgcn_kernarg_segment_ptr() + (lane >> 3) * 8;
        asm volatile("global_load_dwordx2 v[250:251], %0, off\n\tglobal_load_dword v252, %1, off sc1" ::"v"(tab), "v"(cnt) : "memory", "v250", "v251", "v252");
      }
    }
    unsigned plo, phi, c0;
    asm volatile("s_waitcnt vmcnt(0)\n\tv_mov_b32 %0, v250\n\tv_mov_b32 %1, v251\n\tv_mov_b32 %2, v252\n\ts_nop 0" : "=v"(plo), "=v"(phi), "=v"(c0) :: "memory");
    const unsigned call = __builtin_amdgcn_readfirstlane(c0) + 1u;
    const unsigned par = call & 1u;
    GAS u64* gp = (GAS u64*)((((u64)phi << 32) | plo) + WS_GLB + (par * 8 + (lane & 7)) * 128) + rank;
    if (lane == 0) red[6] = par;
    WAIT_L(0);
    BAR();
    STAMP(1);
    {
      const unsigned m = dpp_umax<0x4E>(dpp_umax<0xB1>(red[lane & 3]));
      if (lane == 0) __hip_atomic_store((GAS u64*)(myws + WS_LOC) + par * NWG + bid, ((u64)call << 32) | m, RLX_AGT);
    }
    bool bad = false;
    if (early) {
      BAR();
      if (lane == 0) __hip_atomic_store((GAS unsigned*)(A.ws[drel] + WS_YF + (rank * TPP + tile) * 128), call, RLX_SYS);
      STAMP(6);
      GAS unsigned* f = (GAS unsigned*)(myws + WS_YF + (((rank - drel) & 7) * TPP + tile) * 128);
      while (__hip_atomic_load(f, RLX_SYS) < call) {
        if (__hip_atomic_load(tmo, RLX_AGT) == call || NOW() - t0 > DEADLINE) { bad = true; break; }
        __builtin_amdgcn_s_sleep(4);
      }
      if (lane == 0) red[5] = bad;
      WAIT_L(0);
      BAR();
      STAMP(7);
    }
    if (bid == 0) {
      GAS u64* g = (GAS u64*)(myws + WS_LOC) + par * NWG + lane;
      unsigned m;
      for (;;) {
        const u64 a = __hip_atomic_load(g, RLX_AGT), b = __hip_atomic_load(g + 64, RLX_AGT), c = __hip_atomic_load(g + 128, RLX_AGT), d = __hip_atomic_load(g + 192, RLX_AGT);
        const bool ok = (unsigned)(a >> 32) == call && (unsigned)(b >> 32) == call && (unsigned)(c >> 32) == call && (unsigned)(d >> 32) == call;
        m = max(max((unsigned)a, (unsigned)b), max((unsigned)c, (unsigned)d));
        if (__all(ok)) break;
        if (NOW() - t0 > DEADLINE) { bad = true; break; }
      }
      m = wave_umax(m);
      if (!bad) __hip_atomic_store(gp, ((u64)call << 32) | m, RLX_SYS);
      STAMP(2);
    }
    {
      GAS u64* g = lane == 8 ? (GAS u64*)tmo : (GAS u64*)(myws + WS_GLB + (par * 8 + xcd) * 128) + (lane & 7);
      unsigned gm;
      for (;;) {
        const u64 a = __hip_atomic_load(g, RLX_SYS);
        gm = lane == 8 ? 0u : (unsigned)a;
        if (__builtin_amdgcn_readlane((unsigned)a, 8) == call) { bad = true; break; }
        if (__all(lane == 8 || (unsigned)(a >> 32) == call)) break;
        if (bad || NOW() - t0 > DEADLINE) { bad = true; break; }
        __builtin_amdgcn_s_sleep(2);
      }
      gm = row_umax(gm);
      if (lane == 0) red[4] = bad ? 0x7fc00000u : __float_as_uint(448.0f / __uint_as_float(gm));
      if (bad && lane == 0) __hip_atomic_store(tmo, call, RLX_AGT);
    }
    WAIT_L(0);
    BAR();
    if (lane == 0) __hip_atomic_store(cnt, call, RLX_AGT);
    if (bid == 8 && lane == 0) __hip_atomic_store((GAS unsigned*)(myws + WS_LAST), (unsigned)NOW(), RLX_AGT);
    STAMP(3);
    BAR();
    STAMP(4);
    if (drel != 0 && !early && lane == 0)
      __hip_atomic_store((GAS unsigned*)(A.ws[drel] + WS_DONE) + rank * TPP + tile, call, RLX_SYS);
    if (bid == 8) {
      const bool mine = (lane >> 3) == rank;
      GAS u64* f = mine ? (GAS u64*)tmo : (GAS u64*)(myws + WS_DONE) + 2 * lane;
      const int c0 = 4 * (lane & 3);
      const bool e0 = early_index(c0) < NF, e1 = early_index(c0 + 1) < NF, e2 = early_index(c0 + 2) < NF, e3 = early_index(c0 + 3) < NF;
      for (;;) {
        const u64 a = __hip_atomic_load(f, RLX_SYS), b = __hip_atomic_load(f + 1, RLX_SYS);
        const bool ok = mine || ((e0 || (unsigned)a >= call) && (e1 || (unsigned)(a >> 32) >= call) && (e2 || (unsigned)b >= call) && (e3 || (unsigned)(b >> 32) >= call));
        if (__all(ok)) break;
        if (__any(mine && (unsigned)a == call) || NOW() - t0 > DEADLINE) {
          if (lane == 0) { __hip_atomic_store(tmo, call, RLX_AGT); A.out[0][(size_t)(rank * MS + half * BM) * NS + tc * BN] = 0x7f; }
          break;
        }
      }
    }
    STAMP(5);
    return;
  }

  const int wm = wid >> 1, wn = wid & 1;
  const int fr = lane & 15, g = lane >> 4;
  const int xa0 = X_OFF + (wm * 128 + fr) * 128 + ((g ^ ((fr >> 1) & 7)) << 4);
  const int xa1 = xa0 ^ 64;
  const int q = fr >> 2, p = fr & 3, h = q | ((g & 1) << 2);
  int wa[4];
#pragma unroll
  for (int nt = 0; nt < 4; ++nt) wa[nt] = (8 * g + q) * 256 + (((4 * wn + nt) ^ h) << 5) + p * 8;
  f32x4 acc[8][4];
#pragma unroll
  for (int a = 0; a < 8; ++a)
#pragma unroll
    for (int b = 0; b < 4; ++b) { acc[a][b] = (f32x4){0.f, 0.f, 0.f, 0.f}; asm volatile("" : "+v"(acc[a][b])); }
  s16x4 WA[4][4], WC[4][4];
  bf16x8 X[8][2];
#define MF(a, b, Wf, ks) do { if (!skip) { const bf16x8 wv = __builtin_bit_cast(bf16x8, __builtin_shufflevector(Wf[b][2 * ks], Wf[b][2 * ks + 1], 0, 1, 2, 3, 4, 5, 6, 7)); \
    asm volatile("v_mfma_f32_16x16x32_bf16 %0, %1, %2, %0" : "+v"(acc[a][b]) : "v"(wv), "v"(X[a][ks])); } SCHED(); } while (0)
#define RW(Wf, b, i) do { Wf[b][i] = __builtin_amdgcn_ds_read_tr16_b64_v4i16((LAS s16x4*)(lds + wp[b] + wimm + (i >> 1) * 8192 + (i & 1) * 1024)); SCHED(); } while (0)
#define RX(a, ks) do { X[a][ks] = *(LAS bf16x8*)(lds + (ks ? xp1 : xp0) + a * 2048); SCHED(); } while (0)
#define GRP(a, r0, r1, r2, r3) MF(a, 0, Wc, 0); r0; MF(a, 0, Wc, 1); MF(a, 1, Wc, 0); r1; MF(a, 1, Wc, 1); MF(a, 2, Wc, 0); r2; MF(a, 2, Wc, 1); MF(a, 3, Wc, 0); r3; MF(a, 3, Wc, 1);
  auto step = [&](s16x4 (&Wp)[4][4], s16x4 (&Wc)[4][4], int xoff, const int (&wp)[4], int wimm, bool first) {
    bool skip = first;
    const int xp0 = xa0 + xoff, xp1 = xa1 + xoff;
    MF(5, 0, Wp, 0); RW(Wc, 0, 0); MF(5, 0, Wp, 1); RW(Wc, 0, 1); MF(5, 1, Wp, 0); RW(Wc, 0, 2); MF(5, 1, Wp, 1); RW(Wc, 0, 3);
    MF(5, 2, Wp, 0); RW(Wc, 1, 0); MF(5, 2, Wp, 1); RW(Wc, 1, 1); MF(5, 3, Wp, 0); RW(Wc, 1, 2); MF(5, 3, Wp, 1); RW(Wc, 1, 3);
    MF(6, 0, Wp, 0); RW(Wc, 2, 0); MF(6, 0, Wp, 1); RW(Wc, 2, 1); MF(6, 1, Wp, 0); RW(Wc, 2, 2); MF(6, 1, Wp, 1); RW(Wc, 2, 3);
    MF(6, 2, Wp, 0); RW(Wc, 3, 0); MF(6, 2, Wp, 1); RW(Wc, 3, 1); MF(6, 3, Wp, 0); RW(Wc, 3, 2); MF(6, 3, Wp, 1); RW(Wc, 3, 3);
    MF(7, 0, Wp, 0); RX(0, 0); MF(7, 0, Wp, 1); RX(0, 1); MF(7, 1, Wp, 0); MF(7, 1, Wp, 1); RX(1, 0);
    MF(7, 2, Wp, 0); MF(7, 2, Wp, 1); RX(1, 1); MF(7, 3, Wp, 0); MF(7, 3, Wp, 1);
    skip = false;
    GRP(0, RX(2, 0), RX(2, 1), , )
    GRP(1, RX(3, 0), RX(3, 1), , )
    GRP(2, RX(4, 0), RX(4, 1), RX(5, 0), RX(5, 1))
    GRP(3, RX(6, 0), RX(6, 1), RX(7, 0), RX(7, 1))
    GRP(4, , , , )
  };
  BAR();
  SCHED();
  step(WC, WA, 0, wa, 0, true);
  int xoff = 0;
  for (int t = 1; t < NT - 1; t += 2) {
    xoff = xoff == (NXS - 1) * XB ? 0 : xoff + XB;
    WAIT_L(0); BAR(); SCHED();
    step(WA, WC, xoff, wa, WB, false);
    xoff = xoff == (NXS - 1) * XB ? 0 : xoff + XB;
#pragma unroll
    for (int b = 0; b < 4; ++b) wa[b] ^= 2 * WB;
    WAIT_L(0); BAR(); SCHED();
    step(WC, WA, xoff, wa, 0, false);
  }
  xoff = xoff == (NXS - 1) * XB ? 0 : xoff + XB;
  WAIT_L(0); BAR(); SCHED();
  step(WA, WC, xoff, wa, WB, false);
  float mx = 0.f;
#define MX(a, b) do { asm volatile("v_max3_f32 %0, |%1|, |%2|, %0\n\tv_max3_f32 %0, |%3|, |%4|, %0" : "+v"(mx) : "v"(acc[a][b][0]), "v"(acc[a][b][1]), "v"(acc[a][b][2]), "v"(acc[a][b][3])); SCHED(); } while (0)
#define MX2(a, b) MX(a, b); MX(a + 1, b)
  {
    const bool skip = false;
    auto& Wc = WC;
    GRP(5, MX2(0, 0), MX2(0, 1), MX2(0, 2), MX2(0, 3))
    GRP(6, MX2(2, 0), MX2(2, 1), MX2(2, 2), MX2(2, 3))
    GRP(7, MX2(4, 0), MX2(4, 1), MX2(4, 2), MX2(4, 3))
    MX(6, 0); MX(6, 1); MX(6, 2); MX(6, 3);
    asm volatile("s_nop 7" ::: "memory");
    MX(7, 0); MX(7, 1); MX(7, 2); MX(7, 3);
    asm volatile("s_nop 1" : "+v"(mx));
  }
  {
    const unsigned m = wave_umax(__float_as_uint(mx));
    if (lane == 0) red[wid] = m;
  }
  WAIT_L(0);
  BAR();
  const unsigned lo = (unsigned)((wm * 128 + fr) * NS + wn * 64 + g * 16);
  unsigned char* op = A.out[drel] + (size_t)(rank * MS + half * BM) * NS + tc * BN + lo;
  if (early) {
    const size_t slot = WS_YS + (size_t)((red[6] & 1u) * (NR * TPP) + tile) * YS_TILE + wid * 16384 + lane * 16;
    unsigned char* sp = A.ws[drel] + slot + (size_t)rank * TPP * YS_TILE;
#pragma unroll
    for (int a = 0; a < 8; ++a)
#pragma unroll
      for (int hh = 0; hh < 2; ++hh) {
        const f32x4 u = acc[a][2 * hh], v = acc[a][2 * hh + 1];
        const bf16x8 o = {(__bf16)u[0], (__bf16)u[1], (__bf16)u[2], (__bf16)u[3], (__bf16)v[0], (__bf16)v[1], (__bf16)v[2], (__bf16)v[3]};
        asm volatile("global_store_dwordx4 %0, %1, off sc0 sc1\n\ts_nop 1" ::"v"(sp + (a * 2 + hh) * 1024), "v"(o) : "memory");
        WAIT_V(FTH);
      }
    WAIT_V(0);
    BAR();
    BAR();
    const int from = (rank - drel) & 7;
    op = A.out[0] + (size_t)(from * MS + half * BM) * NS + tc * BN + lo;
    if (__builtin_amdgcn_readfirstlane(red[5]) == 0) {
      const unsigned char* lp = myws + slot + (size_t)from * TPP * YS_TILE;
      u32x4 r[16];
#pragma unroll
      for (int hh = 0; hh < 2; ++hh) {
        u32x4* q8 = r + hh * 8;
        const unsigned char* l0 = lp + hh * 8192;
        asm volatile("global_load_dwordx4 %0, %8, off sc0 sc1\n\tglobal_load_dwordx4 %1, %8, off offset:1024 sc0 sc1\n\t"
                     "global_load_dwordx4 %2, %8, off offset:2048 sc0 sc1\n\tglobal_load_dwordx4 %3, %8, off offset:3072 sc0 sc1\n\t"
                     "global_load_dwordx4 %4, %9, off sc0 sc1\n\tglobal_load_dwordx4 %5, %9, off offset:1024 sc0 sc1\n\t"
                     "global_load_dwordx4 %6, %9, off offset:2048 sc0 sc1\n\tglobal_load_dwordx4 %7, %9, off offset:3072 sc0 sc1\n\ts_waitcnt vmcnt(0)"
                     : "=&v"(q8[0]), "=&v"(q8[1]), "=&v"(q8[2]), "=&v"(q8[3]), "=&v"(q8[4]), "=&v"(q8[5]), "=&v"(q8[6]), "=&v"(q8[7])
                     : "v"(l0), "v"(l0 + 4096) : "memory");
      }
#pragma unroll
      for (int a = 0; a < 8; ++a)
#pragma unroll
        for (int hh = 0; hh < 2; ++hh) {
          const u32x4 w4 = r[a * 2 + hh];
          acc[a][2 * hh] = (f32x4){__uint_as_float(w4[0] << 16), __uint_as_float(w4[0] & 0xffff0000u), __uint_as_float(w4[1] << 16), __uint_as_float(w4[1] & 0xffff0000u)};
          acc[a][2 * hh + 1] = (f32x4){__uint_as_float(w4[2] << 16), __uint_as_float(w4[2] & 0xffff0000u), __uint_as_float(w4[3] << 16), __uint_as_float(w4[3] & 0xffff0000u)};
        }
    }
  }
  BAR();
  const float inv = __uint_as_float(red[4]);
  const bool throttle = drel != 0 && !early;
#pragma unroll
  for (int a = 0; a < 8; ++a) {
    unsigned d[4];
#pragma unroll
    for (int b = 0; b < 4; ++b) {
      const f32x4 v = acc[a][b] * inv;
      int pk = __builtin_amdgcn_cvt_pk_fp8_f32(v[0], v[1], 0, false);
      pk = __builtin_amdgcn_cvt_pk_fp8_f32(v[2], v[3], pk, true);
      d[b] = pk;
    }
    { auto r = __builtin_amdgcn_permlane32_swap(d[0], d[2], false, false); d[0] = r[0]; d[2] = r[1]; }
    { auto r = __builtin_amdgcn_permlane32_swap(d[1], d[3], false, false); d[1] = r[0]; d[3] = r[1]; }
    { auto r = __builtin_amdgcn_permlane16_swap(d[0], d[1], false, false); d[0] = r[0]; d[1] = r[1]; }
    { auto r = __builtin_amdgcn_permlane16_swap(d[2], d[3], false, false); d[2] = r[0]; d[3] = r[1]; }
    const u32x4 o = {d[0], d[1], d[2], d[3]};
    asm volatile("global_store_dwordx4 %0, %1, off sc0 sc1\n\ts_nop 1" ::"v"(op + (size_t)a * 16 * NS), "v"(o) : "memory");
    if (throttle) WAIT_V(QTH);
  }
  WAIT_V(0);
  BAR();
}

extern "C" void kernel_launch(int rank, int nranks, ncclComm_t comm, hipStream_t stream,
                              void* const* d_in, const int* in_sizes, int n_in,
                              void* d_out, int out_size, const int* in_dtypes, int out_dtype,
                              void* const* d_out_all, void* const* d_ws_all, size_t ws_bytes) {
  (void)comm; (void)in_sizes; (void)n_in; (void)d_out; (void)out_size; (void)in_dtypes; (void)out_dtype;
  if (nranks != NR || (unsigned)rank >= (unsigned)NR || ws_bytes < WS_END) { fprintf(stderr, "unexpected configuration\n"); return; }
  Args a;
  a.x = (const u16*)d_in[0]; a.w = (const u16*)d_in[1];
  for (int i = 0; i < NR; ++i) { a.out[i] = (unsigned char*)d_out_all[(rank + i) & 7]; a.ws[i] = (unsigned char*)d_ws_all[(rank + i) & 7]; }
  a.rank = rank; a.pad = 0;
  hipLaunchKernelGGL(fused_gemm_a2a, dim3(NWG), dim3(512), 0, stream, a);
  HIP_CHECK(hipGetLastError());
}
```

```cpp
#include <hip/hip_runtime.h>
#include <rccl/rccl.h>
#include <cstdio>

#define HIP_CHECK(expr) do { hipError_t _e = (expr); if (_e != hipSuccess) { \
    fprintf(stderr, "HIP error %s at %s:%d\n", hipGetErrorString(_e), __FILE__, __LINE__); \
    return; } } while (0)

#define LAS __attribute__((address_space(3)))
#define GAS __attribute__((address_space(1)))
typedef __bf16 bf16x8 __attribute__((ext_vector_type(8)));
typedef short s16x4 __attribute__((ext_vector_type(4)));
typedef float f32x4 __attribute__((ext_vector_type(4)));
typedef unsigned u32x4 __attribute__((ext_vector_type(4)));
typedef unsigned short u16;
typedef unsigned long long u64;

namespace {
constexpr int NR = 8;
constexpr int MG = 4096, NG = 16384, KG = 4096;
constexpr int MS = MG / NR;
constexpr int NS = NG / NR;
constexpr int BM = 256, BN = 128, BK = 64, NT = KG / BK;
constexpr int XB = BM * BK * 2, WB = BK * BN * 2;
constexpr int NXS = 3, NWS = 4;
constexpr int X_OFF = NWS * WB;
constexpr int LDS_BYTES = NXS * XB + NWS * WB;
constexpr int NWG = 256;
constexpr int TPP = 32;
constexpr u64 DEADLINE = 50000000ull;
constexpr int NF = 3;
constexpr int POFF = 100;
constexpr int PACE = 875, TS = 20, PACE2 = 731;
constexpr int PACES = 775;
constexpr int PACEF = 676, PACED = 39;
constexpr int WSL = 5, WSLS = 7, WSLF = 4;
constexpr int FTH = 6, QTH = 0;

constexpr size_t WS_CNT = 0;
constexpr size_t WS_TMO = 4096;
constexpr size_t WS_LAST = 4224;
constexpr size_t WS_LOC = 8192;
constexpr size_t WS_GLB = 16384;
constexpr size_t WS_DONE = 32768;
constexpr size_t WS_YF = 36864;
constexpr size_t WS_YS = 2u << 20;
constexpr size_t YS_TILE = (size_t)BM * BN * 2;
constexpr size_t WS_END = WS_YS + (size_t)2 * NR * TPP * YS_TILE;
static_assert(NF <= TPP && WS_YF + NR * TPP * 128 <= (1u << 20), "layout");

struct Args {
  unsigned char* ws[NR];
  unsigned char* out[NR];
  const u16* x; const u16* w;
  int rank; int pad;
};
}

#define BAR() do { asm volatile("" ::: "memory"); __builtin_amdgcn_s_barrier(); asm volatile("" ::: "memory"); } while (0)
#define WAIT_V(n) asm volatile("s_waitcnt vmcnt(%0)" ::"n"(n) : "memory")
#define WAIT_L(n) asm volatile("s_waitcnt lgkmcnt(%0)" ::"n"(n) : "memory")
#define SCHED() __builtin_amdgcn_sched_barrier(0)
#define RLX_SYS __ATOMIC_RELAXED, __HIP_MEMORY_SCOPE_SYSTEM
#define RLX_AGT __ATOMIC_RELAXED, __HIP_MEMORY_SCOPE_AGENT
#define NOW() __builtin_amdgcn_s_memrealtime()


template <int CTRL> __device__ __forceinline__ unsigned dpp_umax(unsigned v) {
  const unsigned t = __builtin_amdgcn_update_dpp(v, v, CTRL, 0xf, 0xf, false);
  return t > v ? t : v;
}
__device__ __forceinline__ unsigned row_umax(unsigned v) {
  v = dpp_umax<0xB1>(v); v = dpp_umax<0x4E>(v); v = dpp_umax<0x141>(v); return dpp_umax<0x140>(v);
}
__device__ __forceinline__ unsigned wave_umax(unsigned v) {
  v = row_umax(v);
  const unsigned a = __builtin_amdgcn_readlane(v, 0), b = __builtin_amdgcn_readlane(v, 16), c = __builtin_amdgcn_readlane(v, 32), d = __builtin_amdgcn_readlane(v, 48);
  return max(max(a, b), max(c, d));
}

__global__ __launch_bounds__(512, 2) void fused_gemm_a2a(Args A) {
  __shared__ __attribute__((aligned(16))) char smem[LDS_BYTES];
  LAS char* lds = (LAS char*)smem;
  const int tid = threadIdx.x, wid = __builtin_amdgcn_readfirstlane(tid >> 6), lane = tid & 63;
  const int bid = blockIdx.x, xcd = bid & 7, idx = (bid >> 3) & 31;
  const int rank = A.rank;
  const int half = idx & 1;
  const int drel = xcd;
  const int tc = idx >> 1;
  const int n0 = ((rank + drel) & 7) * NS + tc * BN;
  auto early_index = [](int c) { return (c & 3) == 1 ? 99 : c - ((c + 2) >> 2); };
  const int fidx = early_index(tc);
  const bool early = drel != 0 && fidx < NF;
  const int tile = half * 16 + tc;
  const int koff = xcd * 8;
  unsigned char* myws = A.ws[0];
  LAS unsigned* red = (LAS unsigned*)lds;

  if (wid >= 4) {
    if (wid < 6) {
      const int j = wid - 4;
      LAS char* base = lds + X_OFF + j * 16384;
      auto rs = __builtin_amdgcn_make_buffer_rsrc((void*)(A.x + (size_t)(half * BM + j * 128) * KG), 0, 128 * KG * 2, 0x00020000);
      const int v0 = (lane >> 3) * (KG * 2) + (((lane & 7) ^ (lane >> 4)) << 4);
      const int v1 = (lane >> 3) * (KG * 2) + (((lane & 7) ^ (4 | (lane >> 4))) << 4);
      auto issue = [&](int t, int slot) {
#pragma unroll
        for (int i = 0; i < 16; ++i)
          __builtin_amdgcn_raw_ptr_buffer_load_lds(rs, (LAS void*)(base + slot * XB + i * 1024), 16, (i & 1) ? v1 : v0, i * 8 * (KG * 2) + ((t + koff) & 63) * 128, 0, 0);
      };
      issue(0, 0); issue(1, 1);
      int slot = 2;
      WAIT_V(16); BAR();
      for (int t = 0; t < NT - 1; ++t) {
        if (t + 2 < NT) { issue(t + 2, slot); WAIT_V(16); } else WAIT_V(0);
        slot = slot == NXS - 1 ? 0 : slot + 1;
        BAR();
      }
      return;
    }
    if (wid == 6) {
      auto rs = __builtin_amdgcn_make_buffer_rsrc((void*)(A.w + n0), 0, (unsigned)((size_t)KG * NG * 2 - (size_t)n0 * 2), 0x00020000);
      const int v0 = (lane >> 4) * (NG * 2) + (((lane & 15) ^ ((lane >> 4) << 1)) << 4);
      const int v1 = (lane >> 4) * (NG * 2) + (((lane & 15) ^ ((4 | (lane >> 4)) << 1)) << 4);
      auto issue = [&](int t, int slot, bool stream) {
        if (stream) {
#pragma unroll
          for (int i = 0; i < 16; ++i)
            __builtin_amdgcn_raw_ptr_buffer_load_lds(rs, (LAS void*)(lds + slot * WB + i * 1024), 16, (i & 2) ? v1 : v0, (((t + koff) & 63) * 64 + i * 4) * (NG * 2), 0, 2);
        } else {
#pragma unroll
          for (int i = 0; i < 16; ++i)
            __builtin_amdgcn_raw_ptr_buffer_load_lds(rs, (LAS void*)(lds + slot * WB + i * 1024), 16, (i & 2) ? v1 : v0, (((t + koff) & 63) * 64 + i * 4) * (NG * 2), 0, 0);
        }
      };
      issue(0, 0, false); issue(1, 1, false);
      unsigned last;
      asm volatile("global_load_dword %0, %1, off sc1" : "=v"(last) : "v"(myws + WS_LAST) : "memory");
      WAIT_V(16); BAR();
      __builtin_amdgcn_s_sleep(WSL);
      issue(2, 2, false); issue(3, 3, false);
      asm volatile("s_waitcnt vmcnt(32)" : "+v"(last) :: "memory");
      BAR();
      const bool stream = (unsigned)NOW() - (unsigned)__builtin_amdgcn_readfirstlane(last) > 30000u;
      WAIT_V(16); BAR();
      auto run = [&](bool st) {
        for (int t = 2; t < NT - 2; t += 2) {
          if (fidx >= 11) __builtin_amdgcn_s_sleep(WSLS); else if (early) __builtin_amdgcn_s_sleep(WSLF); else __builtin_amdgcn_s_sleep(WSL);
          issue(t + 2, (t + 2) & 3, st); issue(t + 3, (t + 3) & 3, st);
          WAIT_V(32); BAR();
          WAIT_V(16); BAR();
        }
      };
      if (stream) run(true); else run(false);
      WAIT_V(0); BAR();
      return;
    }
    const u64 t0 = NOW();
    GAS unsigned* cnt = (GAS unsigned*)(myws + WS_CNT) + bid;
    GAS unsigned* tmo = (GAS unsigned*)(myws + WS_TMO);
    {
      const char* tp = (const char*)A.x + (size_t)(idx * 16 + (lane & 7)) * (KG * 2) + (xcd * 8 + (lane >> 3)) * 128;
      asm volatile("global_load_dword v253, %0, off\n\tglobal_load_dword v253, %1, off" ::"v"(tp), "v"(tp + 8 * KG * 2) : "memory", "v253");
    }
    const bool plain = !early && fidx < 11;
    const int pace = early ? PACEF + fidx * PACED : plain ? PACE : PACES;
    for (int t = 0; t < NT; ++t) {
      const u64 due = t0 + POFF + (unsigned)(t * pace - (t > TS && plain ? (t - TS) * (PACE - PACE2) : 0)) / 10u;
      while (NOW() < due) __builtin_amdgcn_s_sleep(1);
      BAR();
      if (t == 0) {
        __builtin_amdgcn_s_sleep(8);
        const GAS void* tab = (const GAS char*)(const GAS void*)__builtin_amdgcn_kernarg_segment_ptr() + (lane >> 3) * 8;
        asm volatile("global_load_dwordx2 v[250:251], %0, off\n\tglobal_load_dword v252, %1, off sc1" ::"v"(tab), "v"(cnt) : "memory", "v250", "v251", "v252");
      }
    }
    unsigned plo, phi, c0;
    asm volatile("s_waitcnt vmcnt(0)\n\tv_mov_b32 %0, v250\n\tv_mov_b32 %1, v251\n\tv_mov_b32 %2, v252\n\ts_nop 0" : "=v"(plo), "=v"(phi), "=v"(c0) :: "memory");
    const unsigned call = __builtin_amdgcn_readfirstlane(c0) + 1u;
    const unsigned par = call & 1u;
    GAS u64* gp = (GAS u64*)((((u64)phi << 32) | plo) + WS_GLB + (par * 8 + (lane & 7)) * 128) + rank;
    if (lane == 0) red[6] = par;
    WAIT_L(0);
    BAR();
    {
      const unsigned m = dpp_umax<0x4E>(dpp_umax<0xB1>(red[lane & 3]));
      if (lane == 0) __hip_atomic_store((GAS u64*)(myws + WS_LOC) + par * NWG + bid, ((u64)call << 32) | m, RLX_AGT);
    }
    bool bad = false;
    if (early) {
      BAR();
      if (lane == 0) __hip_atomic_store((GAS unsigned*)(A.ws[drel] + WS_YF + (rank * TPP + tile) * 128), call, RLX_SYS);
      GAS unsigned* f = (GAS unsigned*)(myws + WS_YF + (((rank - drel) & 7) * TPP + tile) * 128);
      while (__hip_atomic_load(f, RLX_SYS) < call) {
        if (__hip_atomic_load(tmo, RLX_AGT) == call || NOW() - t0 > DEADLINE) { bad = true; break; }
        __builtin_amdgcn_s_sleep(4);
      }
      if (lane == 0) red[5] = bad;
      WAIT_L(0);
      BAR();
    }
    if (bid == 0) {
      GAS u64* g = (GAS u64*)(myws + WS_LOC) + par * NWG + lane;
      unsigned m;
      for (;;) {
        const u64 a = __hip_atomic_load(g, RLX_AGT), b = __hip_atomic_load(g + 64, RLX_AGT), c = __hip_atomic_load(g + 128, RLX_AGT), d = __hip_atomic_load(g + 192, RLX_AGT);
        const bool ok = (unsigned)(a >> 32) == call && (unsigned)(b >> 32) == call && (unsigned)(c >> 32) == call && (unsigned)(d >> 32) == call;
        m = max(max((unsigned)a, (unsigned)b), max((unsigned)c, (unsigned)d));
        if (__all(ok)) break;
        if (NOW() - t0 > DEADLINE) { bad = true; break; }
      }
      m = wave_umax(m);
      if (!bad) __hip_atomic_store(gp, ((u64)call << 32) | m, RLX_SYS);
    }
    {
      GAS u64* g = lane == 8 ? (GAS u64*)tmo : (GAS u64*)(myws + WS_GLB + (par * 8 + xcd) * 128) + (lane & 7);
      unsigned gm;
      for (;;) {
        const u64 a = __hip_atomic_load(g, RLX_SYS);
        gm = lane == 8 ? 0u : (unsigned)a;
        if (__builtin_amdgcn_readlane((unsigned)a, 8) == call) { bad = true; break; }
        if (__all(lane == 8 || (unsigned)(a >> 32) == call)) break;
        if (bad || NOW() - t0 > DEADLINE) { bad = true; break; }
        __builtin_amdgcn_s_sleep(2);
      }
      gm = row_umax(gm);
      if (lane == 0) red[4] = bad ? 0x7fc00000u : __float_as_uint(448.0f / __uint_as_float(gm));
      if (bad && lane == 0) __hip_atomic_store(tmo, call, RLX_AGT);
    }
    WAIT_L(0);
    BAR();
    if (lane == 0) __hip_atomic_store(cnt, call, RLX_AGT);
    if (bid == 8 && lane == 0) __hip_atomic_store((GAS unsigned*)(myws + WS_LAST), (unsigned)NOW(), RLX_AGT);
    BAR();
    if (drel != 0 && !early && lane == 0)
      __hip_atomic_store((GAS unsigned*)(A.ws[drel] + WS_DONE) + rank * TPP + tile, call, RLX_SYS);
    if (bid == 8) {
      const bool mine = (lane >> 3) == rank;
      GAS u64* f = mine ? (GAS u64*)tmo : (GAS u64*)(myws + WS_DONE) + 2 * lane;
      const int c0 = 4 * (lane & 3);
      const bool e0 = early_index(c0) < NF, e1 = early_index(c0 + 1) < NF, e2 = early_index(c0 + 2) < NF, e3 = early_index(c0 + 3) < NF;
      for (;;) {
        const u64 a = __hip_atomic_load(f, RLX_SYS), b = __hip_atomic_load(f + 1, RLX_SYS);
        const bool ok = mine || ((e0 || (unsigned)a >= call) && (e1 || (unsigned)(a >> 32) >= call) && (e2 || (unsigned)b >= call) && (e3 || (unsigned)(b >> 32) >= call));
        if (__all(ok)) break;
        if (__any(mine && (unsigned)a == call) || NOW() - t0 > DEADLINE) {
          if (lane == 0) { __hip_atomic_store(tmo, call, RLX_AGT); A.out[0][(size_t)(rank * MS + half * BM) * NS + tc * BN] = 0x7f; }
          break;
        }
      }
    }
    return;
  }

  const int wm = wid >> 1, wn = wid & 1;
  const int fr = lane & 15, g = lane >> 4;
  const int xa0 = X_OFF + (wm * 128 + fr) * 128 + ((g ^ ((fr >> 1) & 7)) << 4);
  const int xa1 = xa0 ^ 64;
  const int q = fr >> 2, p = fr & 3, h = q | ((g & 1) << 2);
  int wa[4];
#pragma unroll
  for (int nt = 0; nt < 4; ++nt) wa[nt] = (8 * g + q) * 256 + (((4 * wn + nt) ^ h) << 5) + p * 8;
  f32x4 acc[8][4];
#pragma unroll
  for (int a = 0; a < 8; ++a)
#pragma unroll
    for (int b = 0; b < 4; ++b) { acc[a][b] = (f32x4){0.f, 0.f, 0.f, 0.f}; asm volatile("" : "+v"(acc[a][b])); }
  s16x4 WA[4][4], WC[4][4];
  bf16x8 X[8][2];
#define MF(a, b, Wf, ks) do { if (!skip) { const bf16x8 wv = __builtin_bit_cast(bf16x8, __builtin_shufflevector(Wf[b][2 * ks], Wf[b][2 * ks + 1], 0, 1, 2, 3, 4, 5, 6, 7)); \
    asm volatile("v_mfma_f32_16x16x32_bf16 %0, %1, %2, %0" : "+v"(acc[a][b]) : "v"(wv), "v"(X[a][ks])); } SCHED(); } while (0)
#define RW(Wf, b, i) do { Wf[b][i] = __builtin_amdgcn_ds_read_tr16_b64_v4i16((LAS s16x4*)(lds + wp[b] + wimm + (i >> 1) * 8192 + (i & 1) * 1024)); SCHED(); } while (0)
#define RX(a, ks) do { X[a][ks] = *(LAS bf16x8*)(lds + (ks ? xp1 : xp0) + a * 2048); SCHED(); } while (0)
#define GRP(a, r0, r1, r2, r3) MF(a, 0, Wc, 0); r0; MF(a, 0, Wc, 1); MF(a, 1, Wc, 0); r1; MF(a, 1, Wc, 1); MF(a, 2, Wc, 0); r2; MF(a, 2, Wc, 1); MF(a, 3, Wc, 0); r3; MF(a, 3, Wc, 1);
  auto step = [&](s16x4 (&Wp)[4][4], s16x4 (&Wc)[4][4], int xoff, const int (&wp)[4], int wimm, bool first) {
    bool skip = first;
    const int xp0 = xa0 + xoff, xp1 = xa1 + xoff;
    MF(5, 0, Wp, 0); RW(Wc, 0, 0); MF(5, 0, Wp, 1); RW(Wc, 0, 1); MF(5, 1, Wp, 0); RW(Wc, 0, 2); MF(5, 1, Wp, 1); RW(Wc, 0, 3);
    MF(5, 2, Wp, 0); RW(Wc, 1, 0); MF(5, 2, Wp, 1); RW(Wc, 1, 1); MF(5, 3, Wp, 0); RW(Wc, 1, 2); MF(5, 3, Wp, 1); RW(Wc, 1, 3);
    MF(6, 0, Wp, 0); RW(Wc, 2, 0); MF(6, 0, Wp, 1); RW(Wc, 2, 1); MF(6, 1, Wp, 0); RW(Wc, 2, 2); MF(6, 1, Wp, 1); RW(Wc, 2, 3);
    MF(6, 2, Wp, 0); RW(Wc, 3, 0); MF(6, 2, Wp, 1); RW(Wc, 3, 1); MF(6, 3, Wp, 0); RW(Wc, 3, 2); MF(6, 3, Wp, 1); RW(Wc, 3, 3);
    MF(7, 0, Wp, 0); RX(0, 0); MF(7, 0, Wp, 1); RX(0, 1); MF(7, 1, Wp, 0); MF(7, 1, Wp, 1); RX(1, 0);
    MF(7, 2, Wp, 0); MF(7, 2, Wp, 1); RX(1, 1); MF(7, 3, Wp, 0); MF(7, 3, Wp, 1);
    skip = false;
    GRP(0, RX(2, 0), RX(2, 1), , )
    GRP(1, RX(3, 0), RX(3, 1), , )
    GRP(2, RX(4, 0), RX(4, 1), RX(5, 0), RX(5, 1))
    GRP(3, RX(6, 0), RX(6, 1), RX(7, 0), RX(7, 1))
    GRP(4, , , , )
  };
  BAR();
  SCHED();
  step(WC, WA, 0, wa, 0, true);
  int xoff = 0;
  for (int t = 1; t < NT - 1; t += 2) {
    xoff = xoff == (NXS - 1) * XB ? 0 : xoff + XB;
    WAIT_L(0); BAR(); SCHED();
    step(WA, WC, xoff, wa, WB, false);
    xoff = xoff == (NXS - 1) * XB ? 0 : xoff + XB;
#pragma unroll
    for (int b = 0; b < 4; ++b) wa[b] ^= 2 * WB;
    WAIT_L(0); BAR(); SCHED();
    step(WC, WA, xoff, wa, 0, false);
  }
  xoff = xoff == (NXS - 1) * XB ? 0 : xoff + XB;
  WAIT_L(0); BAR(); SCHED();
  step(WA, WC, xoff, wa, WB, false);
  float mx = 0.f;
#define MX(a, b) do { asm volatile("v_max3_f32 %0, |%1|, |%2|, %0\n\tv_max3_f32 %0, |%3|, |%4|, %0" : "+v"(mx) : "v"(acc[a][b][0]), "v"(acc[a][b][1]), "v"(acc[a][b][2]), "v"(acc[a][b][3])); SCHED(); } while (0)
#define MX2(a, b) MX(a, b); MX(a + 1, b)
  {
    const bool skip = false;
    auto& Wc = WC;
    GRP(5, MX2(0, 0), MX2(0, 1), MX2(0, 2), MX2(0, 3))
    GRP(6, MX2(2, 0), MX2(2, 1), MX2(2, 2), MX2(2, 3))
    GRP(7, MX2(4, 0), MX2(4, 1), MX2(4, 2), MX2(4, 3))
    MX(6, 0); MX(6, 1); MX(6, 2); MX(6, 3);
    asm volatile("s_nop 7" ::: "memory");
    MX(7, 0); MX(7, 1); MX(7, 2); MX(7, 3);
    asm volatile("s_nop 1" : "+v"(mx));
  }
  {
    const unsigned m = wave_umax(__float_as_uint(mx));
    if (lane == 0) red[wid] = m;
  }
  WAIT_L(0);
  BAR();
  const unsigned lo = (unsigned)((wm * 128 + fr) * NS + wn * 64 + g * 16);
  unsigned char* op = A.out[drel] + (size_t)(rank * MS + half * BM) * NS + tc * BN + lo;
  if (early) {
    const size_t slot = WS_YS + (size_t)((red[6] & 1u) * (NR * TPP) + tile) * YS_TILE + wid * 16384 + lane * 16;
    unsigned char* sp = A.ws[drel] + slot + (size_t)rank * TPP * YS_TILE;
#pragma unroll
    for (int a = 0; a < 8; ++a)
#pragma unroll
      for (int hh = 0; hh < 2; ++hh) {
        const f32x4 u = acc[a][2 * hh], v = acc[a][2 * hh + 1];
        const bf16x8 o = {(__bf16)u[0], (__bf16)u[1], (__bf16)u[2], (__bf16)u[3], (__bf16)v[0], (__bf16)v[1], (__bf16)v[2], (__bf16)v[3]};
        asm volatile("global_store_dwordx4 %0, %1, off sc0 sc1\n\ts_nop 1" ::"v"(sp + (a * 2 + hh) * 1024), "v"(o) : "memory");
        WAIT_V(FTH);
      }
    WAIT_V(0);
    BAR();
    BAR();
    const int from = (rank - drel) & 7;
    op = A.out[0] + (size_t)(from * MS + half * BM) * NS + tc * BN + lo;
    if (__builtin_amdgcn_readfirstlane(red[5]) == 0) {
      const unsigned char* lp = myws + slot + (size_t)from * TPP * YS_TILE;
      u32x4 r[16];
#pragma unroll
      for (int hh = 0; hh < 2; ++hh) {
        u32x4* q8 = r + hh * 8;
        const unsigned char* l0 = lp + hh * 8192;
        asm volatile("global_load_dwordx4 %0, %8, off sc0 sc1\n\tglobal_load_dwordx4 %1, %8, off offset:1024 sc0 sc1\n\t"
                     "global_load_dwordx4 %2, %8, off offset:2048 sc0 sc1\n\tglobal_load_dwordx4 %3, %8, off offset:3072 sc0 sc1\n\t"
                     "global_load_dwordx4 %4, %9, off sc0 sc1\n\tglobal_load_dwordx4 %5, %9, off offset:1024 sc0 sc1\n\t"
                     "global_load_dwordx4 %6, %9, off offset:2048 sc0 sc1\n\tglobal_load_dwordx4 %7, %9, off offset:3072 sc0 sc1\n\ts_waitcnt vmcnt(0)"
                     : "=&v"(q8[0]), "=&v"(q8[1]), "=&v"(q8[2]), "=&v"(q8[3]), "=&v"(q8[4]), "=&v"(q8[5]), "=&v"(q8[6]), "=&v"(q8[7])
                     : "v"(l0), "v"(l0 + 4096) : "memory");
      }
#pragma unroll
      for (int a = 0; a < 8; ++a)
#pragma unroll
        for (int hh = 0; hh < 2; ++hh) {
          const u32x4 w4 = r[a * 2 + hh];
          acc[a][2 * hh] = (f32x4){__uint_as_float(w4[0] << 16), __uint_as_float(w4[0] & 0xffff0000u), __uint_as_float(w4[1] << 16), __uint_as_float(w4[1] & 0xffff0000u)};
          acc[a][2 * hh + 1] = (f32x4){__uint_as_float(w4[2] << 16), __uint_as_float(w4[2] & 0xffff0000u), __uint_as_float(w4[3] << 16), __uint_as_float(w4[3] & 0xffff0000u)};
        }
    }
  }
  BAR();
  const float inv = __uint_as_float(red[4]);
  const bool throttle = drel != 0 && !early;
#pragma unroll
  for (int a = 0; a < 8; ++a) {
    unsigned d[4];
#pragma unroll
    for (int b = 0; b < 4; ++b) {
      const f32x4 v = acc[a][b] * inv;
      int pk = __builtin_amdgcn_cvt_pk_fp8_f32(v[0], v[1], 0, false);
      pk = __builtin_amdgcn_cvt_pk_fp8_f32(v[2], v[3], pk, true);
      d[b] = pk;
    }
    { auto r = __builtin_amdgcn_permlane32_swap(d[0], d[2], false, false); d[0] = r[0]; d[2] = r[1]; }
    { auto r = __builtin_amdgcn_permlane32_swap(d[1], d[3], false, false); d[1] = r[0]; d[3] = r[1]; }
    { auto r = __builtin_amdgcn_permlane16_swap(d[0], d[1], false, false); d[0] = r[0]; d[1] = r[1]; }
    { auto r = __builtin_amdgcn_permlane16_swap(d[2], d[3], false, false); d[2] = r[0]; d[3] = r[1]; }
    const u32x4 o = {d[0], d[1], d[2], d[3]};
    asm volatile("global_store_dwordx4 %0, %1, off sc0 sc1\n\ts_nop 1" ::"v"(op + (size_t)a * 16 * NS), "v"(o) : "memory");
    if (throttle) WAIT_V(QTH);
  }
  WAIT_V(0);
  BAR();
}

extern "C" void kernel_launch(int rank, int nranks, ncclComm_t comm, hipStream_t stream,
                              void* const* d_in, const int* in_sizes, int n_in,
                              void* d_out, int out_size, const int* in_dtypes, int out_dtype,
                              void* const* d_out_all, void* const* d_ws_all, size_t ws_bytes) {
  (void)comm; (void)in_sizes; (void)n_in; (void)d_out; (void)out_size; (void)in_dtypes; (void)out_dtype;
  if (nranks != NR || (unsigned)rank >= (unsigned)NR || ws_bytes < WS_END) { fprintf(stderr, "unexpected configuration\n"); return; }
  Args a;
  a.x = (const u16*)d_in[0]; a.w = (const u16*)d_in[1];
  for (int i = 0; i < NR; ++i) { a.out[i] = (unsigned char*)d_out_all[(rank + i) & 7]; a.ws[i] = (unsigned char*)d_ws_all[(rank + i) & 7]; }
  a.rank = rank; a.pad = 0;
  hipLaunchKernelGGL(fused_gemm_a2a, dim3(NWG), dim3(512), 0, stream, a);
  HIP_CHECK(hipGetLastError());
}
```

```cpp
#include <hip/hip_runtime.h>
#include <rccl/rccl.h>
#include <cstdio>

#define HIP_CHECK(expr) do { hipError_t _e = (expr); if (_e != hipSuccess) { \
    fprintf(stderr, "HIP error %s at %s:%d\n", hipGetErrorString(_e), __FILE__, __LINE__); \
    return; } } while (0)

#define LAS __attribute__((address_space(3)))
#define GAS __attribute__((address_space(1)))
typedef __bf16 bf16x8 __attribute__((ext_vector_type(8)));
typedef short s16x4 __attribute__((ext_vector_type(4)));
typedef float f32x4 __attribute__((ext_vector_type(4)));
typedef unsigned u32x4 __attribute__((ext_vector_type(4)));
typedef unsigned short u16;
typedef unsigned long long u64;

namespace {
constexpr int NR = 8;
constexpr int MG = 4096, NG = 16384, KG = 4096;
constexpr int MS = MG / NR;
constexpr int NS = NG / NR;
constexpr int BM = 256, BN = 128, BK = 64, NT = KG / BK;
constexpr int XB = BM * BK * 2, WB = BK * BN * 2;
constexpr int NXS = 3, NWS = 4;
constexpr int X_OFF = NWS * WB;
constexpr int LDS_BYTES = NXS * XB + NWS * WB;
constexpr int NWG = 256;
constexpr int TPP = 32;
constexpr u64 DEADLINE = 50000000ull;
constexpr int NF = 3;
constexpr int POFF = 100;
constexpr int PACE = 885, PACE2 = 752, PACE3 = 690;
constexpr int TS = 20, TW = 6, TS2 = 50;
constexpr int PACES = 780;
constexpr int PACEF = 676, PACED = 40;
constexpr int WSL = 5, WSLS = 7, WSLF = 4;
constexpr int FTH = 6, QTH = 0;

constexpr size_t WS_CNT = 0;
constexpr size_t WS_TMO = 4096;
constexpr size_t WS_LAST = 4224;
constexpr size_t WS_LOC = 8192;
constexpr size_t WS_GLB = 16384;
constexpr size_t WS_DONE = 32768;
constexpr size_t WS_YF = 36864;
constexpr size_t WS_YS = 2u << 20;
constexpr size_t YS_TILE = (size_t)BM * BN * 2;
constexpr size_t WS_END = WS_YS + (size_t)2 * NR * TPP * YS_TILE;
static_assert(NF <= TPP && WS_YF + NR * TPP * 128 <= (1u << 20), "layout");

struct Args {
  unsigned char* ws[NR];
  unsigned char* out[NR];
  const u16* x; const u16* w;
  int rank; int pad;
};
}

#define BAR() do { asm volatile("" ::: "memory"); __builtin_amdgcn_s_barrier(); asm volatile("" ::: "memory"); } while (0)
#define WAIT_V(n) asm volatile("s_waitcnt vmcnt(%0)" ::"n"(n) : "memory")
#define WAIT_L(n) asm volatile("s_waitcnt lgkmcnt(%0)" ::"n"(n) : "memory")
#define SCHED() __builtin_amdgcn_sched_barrier(0)
#define RLX_SYS __ATOMIC_RELAXED, __HIP_MEMORY_SCOPE_SYSTEM
#define RLX_AGT __ATOMIC_RELAXED, __HIP_MEMORY_SCOPE_AGENT
#define NOW() __builtin_amdgcn_s_memrealtime()


template <int CTRL> __device__ __forceinline__ unsigned dpp_umax(unsigned v) {
  const unsigned t = __builtin_amdgcn_update_dpp(v, v, CTRL, 0xf, 0xf, false);
  return t > v ? t : v;
}
__device__ __forceinline__ unsigned row_umax(unsigned v) {
  v = dpp_umax<0xB1>(v); v = dpp_umax<0x4E>(v); v = dpp_umax<0x141>(v); return dpp_umax<0x140>(v);
}
__device__ __forceinline__ unsigned wave_umax(unsigned v) {
  v = row_umax(v);
  const unsigned a = __builtin_amdgcn_readlane(v, 0), b = __builtin_amdgcn_readlane(v, 16), c = __builtin_amdgcn_readlane(v, 32), d = __builtin_amdgcn_readlane(v, 48);
  return max(max(a, b), max(c, d));
}

__global__ __launch_bounds__(512, 2) void fused_gemm_a2a(Args A) {
  __shared__ __attribute__((aligned(16))) char smem[LDS_BYTES];
  LAS char* lds = (LAS char*)smem;
  const int tid = threadIdx.x, wid = __builtin_amdgcn_readfirstlane(tid >> 6), lane = tid & 63;
  const int bid = blockIdx.x, xcd = bid & 7, idx = (bid >> 3) & 31;
  const int rank = A.rank;
  const int half = idx & 1;
  const int drel = xcd;
  const int tc = idx >> 1;
  const int n0 = ((rank + drel) & 7) * NS + tc * BN;
  auto early_index = [](int c) { return (c & 3) == 1 ? 99 : c - ((c + 2) >> 2); };
  const int fidx = early_index(tc);
  const bool early = drel != 0 && fidx < NF;
  const int tile = half * 16 + tc;
  const int koff = xcd * 8;
  unsigned char* myws = A.ws[0];
  LAS unsigned* red = (LAS unsigned*)lds;

  if (wid >= 4) {
    if (wid < 6) {
      const int j = wid - 4;
      LAS char* base = lds + X_OFF + j * 16384;
      auto rs = __builtin_amdgcn_make_buffer_rsrc((void*)(A.x + (size_t)(half * BM + j * 128) * KG), 0, 128 * KG * 2, 0x00020000);
      const int v0 = (lane >> 3) * (KG * 2) + (((lane & 7) ^ (lane >> 4)) << 4);
      const int v1 = (lane >> 3) * (KG * 2) + (((lane & 7) ^ (4 | (lane >> 4))) << 4);
      auto issue = [&](int t, int slot) {
#pragma unroll
        for (int i = 0; i < 16; ++i)
          __builtin_amdgcn_raw_ptr_buffer_load_lds(rs, (LAS void*)(base + slot * XB + i * 1024), 16, (i & 1) ? v1 : v0, i * 8 * (KG * 2) + ((t + koff) & 63) * 128, 0, 0);
      };
      issue(0, 0); issue(1, 1);
      int slot = 2;
      WAIT_V(16); BAR();
      for (int t = 0; t < NT - 1; ++t) {
        if (t + 2 < NT) { issue(t + 2, slot); WAIT_V(16); } else WAIT_V(0);
        slot = slot == NXS - 1 ? 0 : slot + 1;
        BAR();
      }
      return;
    }
    if (wid == 6) {
      auto rs = __builtin_amdgcn_make_buffer_rsrc((void*)(A.w + n0), 0, (unsigned)((size_t)KG * NG * 2 - (size_t)n0 * 2), 0x00020000);
      const int v0 = (lane >> 4) * (NG * 2) + (((lane & 15) ^ ((lane >> 4) << 1)) << 4);
      const int v1 = (lane >> 4) * (NG * 2) + (((lane & 15) ^ ((4 | (lane >> 4)) << 1)) << 4);
      auto issue = [&](int t, int slot, bool stream) {
        if (stream) {
#pragma unroll
          for (int i = 0; i < 16; ++i)
            __builtin_amdgcn_raw_ptr_buffer_load_lds(rs, (LAS void*)(lds + slot * WB + i * 1024), 16, (i & 2) ? v1 : v0, (((t + koff) & 63) * 64 + i * 4) * (NG * 2), 0, 2);
        } else {
#pragma unroll
          for (int i = 0; i < 16; ++i)
            __builtin_amdgcn_raw_ptr_buffer_load_lds(rs, (LAS void*)(lds + slot * WB + i * 1024), 16, (i & 2) ? v1 : v0, (((t + koff) & 63) * 64 + i * 4) * (NG * 2), 0, 0);
        }
      };
      issue(0, 0, false); issue(1, 1, false);
      unsigned last;
      asm volatile("global_load_dword %0, %1, off sc1" : "=v"(last) : "v"(myws + WS_LAST) : "memory");
      WAIT_V(16); BAR();
      __builtin_amdgcn_s_sleep(WSL);
      issue(2, 2, false); issue(3, 3, false);
      asm volatile("s_waitcnt vmcnt(32)" : "+v"(last) :: "memory");
      BAR();
      const bool stream = (unsigned)NOW() - (unsigned)__builtin_amdgcn_readfirstlane(last) > 30000u;
      WAIT_V(16); BAR();
      auto run = [&](bool st) {
        for (int t = 2; t < NT - 2; t += 2) {
          if (fidx >= 11) __builtin_amdgcn_s_sleep(WSLS); else if (early) __builtin_amdgcn_s_sleep(WSLF); else __builtin_amdgcn_s_sleep(WSL);
          issue(t + 2, (t + 2) & 3, st); issue(t + 3, (t + 3) & 3, st);
          WAIT_V(32); BAR();
          WAIT_V(16); BAR();
        }
      };
      if (stream) run(true); else run(false);
      WAIT_V(0); BAR();
      return;
    }
    const u64 t0 = NOW();
    GAS unsigned* cnt = (GAS unsigned*)(myws + WS_CNT) + bid;
    GAS unsigned* tmo = (GAS unsigned*)(myws + WS_TMO);
    {
      const char* tp = (const char*)A.x + (size_t)(idx * 16 + (lane & 7)) * (KG * 2) + (xcd * 8 + (lane >> 3)) * 128;
      asm volatile("global_load_dword v253, %0, off\n\tglobal_load_dword v253, %1, off" ::"v"(tp), "v"(tp + 8 * KG * 2) : "memory", "v253");
    }
    const bool plain = !early && fidx < 11;
    const int pace = early ? PACEF + fidx * PACED : PACES;
    unsigned sched = 0;
    for (int t = 0; t < NT; ++t) {
      const u64 due = t0 + POFF + sched / 10u;
      sched += !plain ? pace : t < TS - TW ? PACE : t < TS + TW ? PACE - (PACE - PACE2) * (2 * (t - TS + TW) + 1) / (4 * TW) : t < TS2 ? PACE2 : PACE3;
      while (NOW() < due) __builtin_amdgcn_s_sleep(1);
      BAR();
      if (t == 0) {
        __builtin_amdgcn_s_sleep(8);
        const GAS void* tab = (const GAS char*)(const GAS void*)__builtin_amdgcn_kernarg_segment_ptr() + (lane >> 3) * 8;
        asm volatile("global_load_dwordx2 v[250:251], %0, off\n\tglobal_load_dword v252, %1, off sc1" ::"v"(tab), "v"(cnt) : "memory", "v250", "v251", "v252");
      }
    }
    unsigned plo, phi, c0;
    asm volatile("s_waitcnt vmcnt(0)\n\tv_mov_b32 %0, v250\n\tv_mov_b32 %1, v251\n\tv_mov_b32 %2, v252\n\ts_nop 0" : "=v"(plo), "=v"(phi), "=v"(c0) :: "memory");
    const unsigned call = __builtin_amdgcn_readfirstlane(c0) + 1u;
    const unsigned par = call & 1u;
    GAS u64* gp = (GAS u64*)((((u64)phi << 32) | plo) + WS_GLB + (par * 8 + (lane & 7)) * 128) + rank;
    if (lane == 0) red[6] = par;
    WAIT_L(0);
    BAR();
    {
      const unsigned m = dpp_umax<0x4E>(dpp_umax<0xB1>(red[lane & 3]));
      if (lane == 0) __hip_atomic_store((GAS u64*)(myws + WS_LOC) + par * NWG + bid, ((u64)call << 32) | m, RLX_AGT);
    }
    bool bad = false;
    if (early) {
      BAR();
      if (lane == 0) __hip_atomic_store((GAS unsigned*)(A.ws[drel] + WS_YF + (rank * TPP + tile) * 128), call, RLX_SYS);
      GAS unsigned* f = (GAS unsigned*)(myws + WS_YF + (((rank - drel) & 7) * TPP + tile) * 128);
      while (__hip_atomic_load(f, RLX_SYS) < call) {
        if (__hip_atomic_load(tmo, RLX_AGT) == call || NOW() - t0 > DEADLINE) { bad = true; break; }
        __builtin_amdgcn_s_sleep(4);
      }
      if (lane == 0) red[5] = bad;
      WAIT_L(0);
      BAR();
    }
    if (bid == 0) {
      GAS u64* g = (GAS u64*)(myws + WS_LOC) + par * NWG + lane;
      unsigned m;
      for (;;) {
        const u64 a = __hip_atomic_load(g, RLX_AGT), b = __hip_atomic_load(g + 64, RLX_AGT), c = __hip_atomic_load(g + 128, RLX_AGT), d = __hip_atomic_load(g + 192, RLX_AGT);
        const bool ok = (unsigned)(a >> 32) == call && (unsigned)(b >> 32) == call && (unsigned)(c >> 32) == call && (unsigned)(d >> 32) == call;
        m = max(max((unsigned)a, (unsigned)b), max((unsigned)c, (unsigned)d));
        if (__all(ok)) break;
        if (NOW() - t0 > DEADLINE) { bad = true; break; }
      }
      m = wave_umax(m);
      if (!bad) __hip_atomic_store(gp, ((u64)call << 32) | m, RLX_SYS);
    }
    {
      GAS u64* g = lane == 8 ? (GAS u64*)tmo : (GAS u64*)(myws + WS_GLB + (par * 8 + xcd) * 128) + (lane & 7);
      unsigned gm;
      for (;;) {
        const u64 a = __hip_atomic_load(g, RLX_SYS);
        gm = lane == 8 ? 0u : (unsigned)a;
        if (__builtin_amdgcn_readlane((unsigned)a, 8) == call) { bad = true; break; }
        if (__all(lane == 8 || (unsigned)(a >> 32) == call)) break;
        if (bad || NOW() - t0 > DEADLINE) { bad = true; break; }
        __builtin_amdgcn_s_sleep(1);
      }
      gm = row_umax(gm);
      if (lane == 0) red[4] = bad ? 0x7fc00000u : __float_as_uint(448.0f / __uint_as_float(gm));
      if (bad && lane == 0) __hip_atomic_store(tmo, call, RLX_AGT);
    }
    WAIT_L(0);
    BAR();
    if (lane == 0) __hip_atomic_store(cnt, call, RLX_AGT);
    if (bid == 8 && lane == 0) __hip_atomic_store((GAS unsigned*)(myws + WS_LAST), (unsigned)NOW(), RLX_AGT);
    BAR();
    if (drel != 0 && !early && lane == 0)
      __hip_atomic_store((GAS unsigned*)(A.ws[drel] + WS_DONE) + rank * TPP + tile, call, RLX_SYS);
    if (bid == 8) {
      const bool mine = (lane >> 3) == rank;
      GAS u64* f = mine ? (GAS u64*)tmo : (GAS u64*)(myws + WS_DONE) + 2 * lane;
      const int c0 = 4 * (lane & 3);
      const bool e0 = early_index(c0) < NF, e1 = early_index(c0 + 1) < NF, e2 = early_index(c0 + 2) < NF, e3 = early_index(c0 + 3) < NF;
      for (;;) {
        const u64 a = __hip_atomic_load(f, RLX_SYS), b = __hip_atomic_load(f + 1, RLX_SYS);
        const bool ok = mine || ((e0 || (unsigned)a >= call) && (e1 || (unsigned)(a >> 32) >= call) && (e2 || (unsigned)b >= call) && (e3 || (unsigned)(b >> 32) >= call));
        if (__all(ok)) break;
        if (__any(mine && (unsigned)a == call) || NOW() - t0 > DEADLINE) {
          if (lane == 0) { __hip_atomic_store(tmo, call, RLX_AGT); A.out[0][(size_t)(rank * MS + half * BM) * NS + tc * BN] = 0x7f; }
          break;
        }
      }
    }
    return;
  }

  const int wm = wid >> 1, wn = wid & 1;
  const int fr = lane & 15, g = lane >> 4;
  const int xa0 = X_OFF + (wm * 128 + fr) * 128 + ((g ^ ((fr >> 1) & 7)) << 4);
  const int xa1 = xa0 ^ 64;
  const int q = fr >> 2, p = fr & 3, h = q | ((g & 1) << 2);
  int wa[4];
#pragma unroll
  for (int nt = 0; nt < 4; ++nt) wa[nt] = (8 * g + q) * 256 + (((4 * wn + nt) ^ h) << 5) + p * 8;
  f32x4 acc[8][4];
#pragma unroll
  for (int a = 0; a < 8; ++a)
#pragma unroll
    for (int b = 0; b < 4; ++b) { acc[a][b] = (f32x4){0.f, 0.f, 0.f, 0.f}; asm volatile("" : "+v"(acc[a][b])); }
  s16x4 WA[4][4], WC[4][4];
  bf16x8 X[8][2];
#define MF(a, b, Wf, ks) do { if (!skip) { const bf16x8 wv = __builtin_bit_cast(bf16x8, __builtin_shufflevector(Wf[b][2 * ks], Wf[b][2 * ks + 1], 0, 1, 2, 3, 4, 5, 6, 7)); \
    asm volatile("v_mfma_f32_16x16x32_bf16 %0, %1, %2, %0" : "+v"(acc[a][b]) : "v"(wv), "v"(X[a][ks])); } SCHED(); } while (0)
#define RW(Wf, b, i) do { Wf[b][i] = __builtin_amdgcn_ds_read_tr16_b64_v4i16((LAS s16x4*)(lds + wp[b] + wimm + (i >> 1) * 8192 + (i & 1) * 1024)); SCHED(); } while (0)
#define RX(a, ks) do { X[a][ks] = *(LAS bf16x8*)(lds + (ks ? xp1 : xp0) + a * 2048); SCHED(); } while (0)
#define GRP(a, r0, r1, r2, r3) MF(a, 0, Wc, 0); r0; MF(a, 0, Wc, 1); MF(a, 1, Wc, 0); r1; MF(a, 1, Wc, 1); MF(a, 2, Wc, 0); r2; MF(a, 2, Wc, 1); MF(a, 3, Wc, 0); r3; MF(a, 3, Wc, 1);
  auto step = [&](s16x4 (&Wp)[4][4], s16x4 (&Wc)[4][4], int xoff, const int (&wp)[4], int wimm, bool first) {
    bool skip = first;
    const int xp0 = xa0 + xoff, xp1 = xa1 + xoff;
    MF(5, 0, Wp, 0); RW(Wc, 0, 0); MF(5, 0, Wp, 1); RW(Wc, 0, 1); MF(5, 1, Wp, 0); RW(Wc, 0, 2); MF(5, 1, Wp, 1); RW(Wc, 0, 3);
    MF(5, 2, Wp, 0); RW(Wc, 1, 0); MF(5, 2, Wp, 1); RW(Wc, 1, 1); MF(5, 3, Wp, 0); RW(Wc, 1, 2); MF(5, 3, Wp, 1); RW(Wc, 1, 3);
    MF(6, 0, Wp, 0); RW(Wc, 2, 0); MF(6, 0, Wp, 1); RW(Wc, 2, 1); MF(6, 1, Wp, 0); RW(Wc, 2, 2); MF(6, 1, Wp, 1); RW(Wc, 2, 3);
    MF(6, 2, Wp, 0); RW(Wc, 3, 0); MF(6, 2, Wp, 1); RW(Wc, 3, 1); MF(6, 3, Wp, 0); RW(Wc, 3, 2); MF(6, 3, Wp, 1); RW(Wc, 3, 3);
    MF(7, 0, Wp, 0); RX(0, 0); MF(7, 0, Wp, 1); RX(0, 1); MF(7, 1, Wp, 0); MF(7, 1, Wp, 1); RX(1, 0);
    MF(7, 2, Wp, 0); MF(7, 2, Wp, 1); RX(1, 1); MF(7, 3, Wp, 0); MF(7, 3, Wp, 1);
    skip = false;
    GRP(0, RX(2, 0), RX(2, 1), , )
    GRP(1, RX(3, 0), RX(3, 1), , )
    GRP(2, RX(4, 0), RX(4, 1), RX(5, 0), RX(5, 1))
    GRP(3, RX(6, 0), RX(6, 1), RX(7, 0), RX(7, 1))
    GRP(4, , , , )
  };
  BAR();
  SCHED();
  step(WC, WA, 0, wa, 0, true);
  int xoff = 0;
  for (int t = 1; t < NT - 1; t += 2) {
    xoff = xoff == (NXS - 1) * XB ? 0 : xoff + XB;
    WAIT_L(0); BAR(); SCHED();
    step(WA, WC, xoff, wa, WB, false);
    xoff = xoff == (NXS - 1) * XB ? 0 : xoff + XB;
#pragma unroll
    for (int b = 0; b < 4; ++b) wa[b] ^= 2 * WB;
    WAIT_L(0); BAR(); SCHED();
    step(WC, WA, xoff, wa, 0, false);
  }
  xoff = xoff == (NXS - 1) * XB ? 0 : xoff + XB;
  WAIT_L(0); BAR(); SCHED();
  step(WA, WC, xoff, wa, WB, false);
  float mx = 0.f;
#define MX(a, b) do { asm volatile("v_max3_f32 %0, |%1|, |%2|, %0\n\tv_max3_f32 %0, |%3|, |%4|, %0" : "+v"(mx) : "v"(acc[a][b][0]), "v"(acc[a][b][1]), "v"(acc[a][b][2]), "v"(acc[a][b][3])); SCHED(); } while (0)
#define MX2(a, b) MX(a, b); MX(a + 1, b)
  {
    const bool skip = false;
    auto& Wc = WC;
    GRP(5, MX2(0, 0), MX2(0, 1), MX2(0, 2), MX2(0, 3))
    GRP(6, MX2(2, 0), MX2(2, 1), MX2(2, 2), MX2(2, 3))
    GRP(7, MX2(4, 0), MX2(4, 1), MX2(4, 2), MX2(4, 3))
    MX(6, 0); MX(6, 1); MX(6, 2); MX(6, 3);
    asm volatile("s_nop 7" ::: "memory");
    MX(7, 0); MX(7, 1); MX(7, 2); MX(7, 3);
    asm volatile("s_nop 1" : "+v"(mx));
  }
  {
    const unsigned m = wave_umax(__float_as_uint(mx));
    if (lane == 0) red[wid] = m;
  }
  WAIT_L(0);
  BAR();
  const unsigned lo = (unsigned)((wm * 128 + fr) * NS + wn * 64 + g * 16);
  unsigned char* op = A.out[drel] + (size_t)(rank * MS + half * BM) * NS + tc * BN + lo;
  if (early) {
    const size_t slot = WS_YS + (size_t)((red[6] & 1u) * (NR * TPP) + tile) * YS_TILE + wid * 16384 + lane * 16;
    unsigned char* sp = A.ws[drel] + slot + (size_t)rank * TPP * YS_TILE;
#pragma unroll
    for (int a = 0; a < 8; ++a)
#pragma unroll
      for (int hh = 0; hh < 2; ++hh) {
        const f32x4 u = acc[a][2 * hh], v = acc[a][2 * hh + 1];
        const bf16x8 o = {(__bf16)u[0], (__bf16)u[1], (__bf16)u[2], (__bf16)u[3], (__bf16)v[0], (__bf16)v[1], (__bf16)v[2], (__bf16)v[3]};
        asm volatile("global_store_dwordx4 %0, %1, off sc0 sc1\n\ts_nop 1" ::"v"(sp + (a * 2 + hh) * 1024), "v"(o) : "memory");
        WAIT_V(FTH);
      }
    WAIT_V(0);
    BAR();
    BAR();
    const int from = (rank - drel) & 7;
    op = A.out[0] + (size_t)(from * MS + half * BM) * NS + tc * BN + lo;
    if (__builtin_amdgcn_readfirstlane(red[5]) == 0) {
      const unsigned char* lp = myws + slot + (size_t)from * TPP * YS_TILE;
      u32x4 r[16];
#pragma unroll
      for (int hh = 0; hh < 2; ++hh) {
        u32x4* q8 = r + hh * 8;
        const unsigned char* l0 = lp + hh * 8192;
        asm volatile("global_load_dwordx4 %0, %8, off sc0 sc1\n\tglobal_load_dwordx4 %1, %8, off offset:1024 sc0 sc1\n\t"
                     "global_load_dwordx4 %2, %8, off offset:2048 sc0 sc1\n\tglobal_load_dwordx4 %3, %8, off offset:3072 sc0 sc1\n\t"
                     "global_load_dwordx4 %4, %9, off sc0 sc1\n\tglobal_load_dwordx4 %5, %9, off offset:1024 sc0 sc1\n\t"
                     "global_load_dwordx4 %6, %9, off offset:2048 sc0 sc1\n\tglobal_load_dwordx4 %7, %9, off offset:3072 sc0 sc1\n\ts_waitcnt vmcnt(0)"
                     : "=&v"(q8[0]), "=&v"(q8[1]), "=&v"(q8[2]), "=&v"(q8[3]), "=&v"(q8[4]), "=&v"(q8[5]), "=&v"(q8[6]), "=&v"(q8[7])
                     : "v"(l0), "v"(l0 + 4096) : "memory");
      }
#pragma unroll
      for (int a = 0; a < 8; ++a)
#pragma unroll
        for (int hh = 0; hh < 2; ++hh) {
          const u32x4 w4 = r[a * 2 + hh];
          acc[a][2 * hh] = (f32x4){__uint_as_float(w4[0] << 16), __uint_as_float(w4[0] & 0xffff0000u), __uint_as_float(w4[1] << 16), __uint_as_float(w4[1] & 0xffff0000u)};
          acc[a][2 * hh + 1] = (f32x4){__uint_as_float(w4[2] << 16), __uint_as_float(w4[2] & 0xffff0000u), __uint_as_float(w4[3] << 16), __uint_as_float(w4[3] & 0xffff0000u)};
        }
    }
  }
  BAR();
  const float inv = __uint_as_float(red[4]);
  const bool throttle = drel != 0 && !early;
#pragma unroll
  for (int a = 0; a < 8; ++a) {
    unsigned d[4];
#pragma unroll
    for (int b = 0; b < 4; ++b) {
      const f32x4 v = acc[a][b] * inv;
      int pk = __builtin_amdgcn_cvt_pk_fp8_f32(v[0], v[1], 0, false);
      pk = __builtin_amdgcn_cvt_pk_fp8_f32(v[2], v[3], pk, true);
      d[b] = pk;
    }
    { auto r = __builtin_amdgcn_permlane32_swap(d[0], d[2], false, false); d[0] = r[0]; d[2] = r[1]; }
    { auto r = __builtin_amdgcn_permlane32_swap(d[1], d[3], false, false); d[1] = r[0]; d[3] = r[1]; }
    { auto r = __builtin_amdgcn_permlane16_swap(d[0], d[1], false, false); d[0] = r[0]; d[1] = r[1]; }
    { auto r = __builtin_amdgcn_permlane16_swap(d[2], d[3], false, false); d[2] = r[0]; d[3] = r[1]; }
    const u32x4 o = {d[0], d[1], d[2], d[3]};
    asm volatile("global_store_dwordx4 %0, %1, off sc0 sc1\n\ts_nop 1" ::"v"(op + (size_t)a * 16 * NS), "v"(o) : "memory");
    if (throttle) WAIT_V(QTH);
  }
  WAIT_V(0);
  BAR();
}

extern "C" void kernel_launch(int rank, int nranks, ncclComm_t comm, hipStream_t stream,
                              void* const* d_in, const int* in_sizes, int n_in,
                              void* d_out, int out_size, const int* in_dtypes, int out_dtype,
                              void* const* d_out_all, void* const* d_ws_all, size_t ws_bytes) {
  (void)comm; (void)in_sizes; (void)n_in; (void)d_out; (void)out_size; (void)in_dtypes; (void)out_dtype;
  if (nranks != NR || (unsigned)rank >= (unsigned)NR || ws_bytes < WS_END) { fprintf(stderr, "unexpected configuration\n"); return; }
  Args a;
  a.x = (const u16*)d_in[0]; a.w = (const u16*)d_in[1];
  for (int i = 0; i < NR; ++i) { a.out[i] = (unsigned char*)d_out_all[(rank + i) & 7]; a.ws[i] = (unsigned char*)d_ws_all[(rank + i) & 7]; }
  a.rank = rank; a.pad = 0;
  hipLaunchKernelGGL(fused_gemm_a2a, dim3(NWG), dim3(512), 0, stream, a);
  HIP_CHECK(hipGetLastError());
}
```

```cpp
#include <hip/hip_runtime.h>
#include <rccl/rccl.h>
#include <cstdio>

#define HIP_CHECK(expr) do { hipError_t _e = (expr); if (_e != hipSuccess) { \
    fprintf(stderr, "HIP error %s at %s:%d\n", hipGetErrorString(_e), __FILE__, __LINE__); \
    return; } } while (0)

#define LAS __attribute__((address_space(3)))
#define GAS __attribute__((address_space(1)))
typedef __bf16 bf16x8 __attribute__((ext_vector_type(8)));
typedef short s16x4 __attribute__((ext_vector_type(4)));
typedef float f32x4 __attribute__((ext_vector_type(4)));
typedef unsigned u32x4 __attribute__((ext_vector_type(4)));
typedef unsigned short u16;
typedef unsigned long long u64;

#define NF 12
#define TEND 59800
#define TF0 48500
#define TFD 960
#define POFF 1000
#define POFFB 7000
#define TAU 730
#define TAUE 640
#define EOFF 600
#define LEAD 6
#define DSL 9
#define WSL 5
#define WSLS 7
#define WSLF 4
#define FTH 12
#define QTH 3
#define DESYNC 1600
#define DRAMP 32
#define PADX 1
#define SP 44
#define SOFT 2400
#define SOFTA 5000
#define SLK 0
#define LT1 40
#define LT2 120
#define FTH1 3
#define FTH2 1
#define XT 1
#define DSA 1

namespace {
constexpr int NR = 8;
constexpr int MG = 4096, NG = 16384, KG = 4096;
constexpr int MS = MG / NR;
constexpr int NS = NG / NR;
constexpr int BM = 256, BN = 128, BK = 64, NT = KG / BK;
constexpr int XB = BM * BK * 2, WB = BK * BN * 2;
constexpr int NXS = 3, NWS = 4;
constexpr int X_OFF = NWS * WB;
constexpr int LDS_BYTES = NXS * XB + NWS * WB;
constexpr int NWG = 256;
constexpr int TPP = 16;
constexpr u64 DEADLINE = 50000000ull;

constexpr size_t WS_CNT = 0;
constexpr size_t WS_TMO = 4096;
constexpr size_t WS_LAST = 4224;
constexpr size_t WS_LOC = 8192;
constexpr size_t WS_GLB = 16384;
constexpr size_t WS_DONE = 32768;
constexpr size_t WS_YF = 36864;
constexpr size_t WS_XF = 65536;
constexpr size_t WS_XS = 2u << 20;
constexpr int PCS = PADX ? 2048 : 1024;
constexpr size_t XS_SLOT = (size_t)NT * 32 * PCS;
constexpr size_t WS_YS = WS_XS + NR * XS_SLOT;
constexpr int YS_WAVE = 17 * 1024;
constexpr size_t YS_TILE = 4 * YS_WAVE;
constexpr size_t WS_END = WS_YS + (size_t)2 * NR * TPP * YS_TILE;
static_assert(NF <= TPP && WS_YF + NR * TPP * 128 <= WS_XF, "layout");

struct Args {
  unsigned char* ws[NR];
  unsigned char* out[NR];
  const u16* x; const u16* w;
  int rank; int pad;
};
}

#define BAR() do { asm volatile("" ::: "memory"); __builtin_amdgcn_s_barrier(); asm volatile("" ::: "memory"); } while (0)
#define WAIT_V(n) asm volatile("s_waitcnt vmcnt(%0)" ::"n"(n) : "memory")
#define WAIT_L(n) asm volatile("s_waitcnt lgkmcnt(%0)" ::"n"(n) : "memory")
#define SCHED() __builtin_amdgcn_sched_barrier(0)
#define RLX_SYS __ATOMIC_RELAXED, __HIP_MEMORY_SCOPE_SYSTEM
#define RLX_AGT __ATOMIC_RELAXED, __HIP_MEMORY_SCOPE_AGENT
#define NOW() __builtin_amdgcn_s_memrealtime()

#define STAMP(i) do {} while (0)

template <int CTRL> __device__ __forceinline__ unsigned dpp_umax(unsigned v) {
  const unsigned t = __builtin_amdgcn_update_dpp(v, v, CTRL, 0xf, 0xf, false);
  return t > v ? t : v;
}
__device__ __forceinline__ unsigned row_umax(unsigned v) {
  v = dpp_umax<0xB1>(v); v = dpp_umax<0x4E>(v); v = dpp_umax<0x141>(v); return dpp_umax<0x140>(v);
}
__device__ __forceinline__ unsigned wave_umax(unsigned v) {
  v = row_umax(v);
  const unsigned a = __builtin_amdgcn_readlane(v, 0), b = __builtin_amdgcn_readlane(v, 16), c = __builtin_amdgcn_readlane(v, 32), d = __builtin_amdgcn_readlane(v, 48);
  return max(max(a, b), max(c, d));
}

__device__ __forceinline__ unsigned vm_busy() { return __builtin_amdgcn_s_getreg(7 | (31 << 11)) & 0x00C0000Fu; }

__global__ __launch_bounds__(512, 2) void fused_gemm_a2a(Args A) {
  __shared__ __attribute__((aligned(16))) char smem[LDS_BYTES];
  LAS char* lds = (LAS char*)smem;
  const int tid = threadIdx.x, wid = __builtin_amdgcn_readfirstlane(tid >> 6), lane = tid & 63;
  const int bid = blockIdx.x, xcd = bid & 7, idx = (bid >> 3) & 31;
  const int rank = A.rank;
  int src = rank, half = 0, drel = 0, tc;
  if (idx < 14) { const int a = xcd * 14 + idx; drel = 1 + (a >> 4); tc = a & 15; }
  else if (idx < 16) tc = (xcd >> 1) * 4 + (xcd & 1) * 2 + (idx - 14);
  else { const int k = idx - 16; src = (rank + (xcd & 1) * 4 + (k >> 2)) & 7; half = 1; tc = (xcd >> 1) * 4 + (k & 3); }
  const bool remote_x = src != rank;
  const int n0 = ((rank + drel) & 7) * NS + tc * BN;
  const int koff = half ? 0 : xcd * 8;
  auto early_index = [](int c) { return (c & 3) == 1 ? 12 + (c >> 2) : c - ((c + 2) >> 2); };
  const int fidx = early_index(tc);
  const bool early = drel != 0 && fidx < NF;
  const int tile = tc;
  unsigned char* myws = A.ws[0];
  LAS unsigned* red = (LAS unsigned*)lds;

  if (wid >= 4) {
    if (wid < 6) {
      const int j = wid - 4;
      LAS char* base = lds + X_OFF + j * 16384;
      if (!remote_x) {
        auto rs = __builtin_amdgcn_make_buffer_rsrc((void*)(A.x + (size_t)(half * BM + j * 128) * KG), 0, 128 * KG * 2, 0x00020000);
        const int v0 = (lane >> 3) * (KG * 2) + (((lane & 7) ^ (lane >> 4)) << 4);
        const int v1 = (lane >> 3) * (KG * 2) + (((lane & 7) ^ (4 | (lane >> 4))) << 4);
        auto issue = [&](int t, int slot) {
#pragma unroll
          for (int i = 0; i < 16; ++i)
            __builtin_amdgcn_raw_ptr_buffer_load_lds(rs, (LAS void*)(base + slot * XB + i * 1024), 16, (i & 1) ? v1 : v0, i * 8 * (KG * 2) + ((t + koff) & 63) * 128, 0, 0);
        };
        issue(0, 0); issue(1, 1);
        int slot = 2;
        WAIT_V(16); BAR();
        for (int t = 0; t < NT - 1; ++t) {
          if (t + 2 < NT) { issue(t + 2, slot); WAIT_V(16); } else WAIT_V(0);
          slot = slot == NXS - 1 ? 0 : slot + 1;
          BAR();
        }
      } else {
        const unsigned char* sb = myws + WS_XS + (size_t)src * XS_SLOT + j * 16 * PCS;
        const unsigned vo = lane * 16 + (PADX ? (lane < 56 ? 512 : 640) : 0);
        auto issue = [&](int t, int slot) {
#pragma unroll
          for (int i = 0; i < 16; ++i)
            __builtin_amdgcn_global_load_lds((const GAS void*)(sb + (size_t)((t * 32 + i) * PCS) + vo), (LAS void*)(base + slot * XB + i * 1024), 16, 0, 17);
        };
        BAR();
        issue(0, 0); issue(1, 1);
        int slot = 2;
        WAIT_V(16); BAR();
        for (int t = 0; t < NT - 1; ++t) {
          if (t + 2 < NT) { issue(t + 2, slot); WAIT_V(16); } else WAIT_V(0);
          slot = slot == NXS - 1 ? 0 : slot + 1;
          BAR();
        }
      }
      return;
    }
    if (wid == 6) {
      auto rs = __builtin_amdgcn_make_buffer_rsrc((void*)(A.w + n0), 0, (unsigned)((size_t)KG * NG * 2 - (size_t)n0 * 2), 0x00020000);
      const int v0 = (lane >> 4) * (NG * 2) + (((lane & 15) ^ ((lane >> 4) << 1)) << 4);
      const int v1 = (lane >> 4) * (NG * 2) + (((lane & 15) ^ ((4 | (lane >> 4)) << 1)) << 4);
      auto issue = [&](int t, int slot, bool stream) {
        if (stream) {
#pragma unroll
          for (int i = 0; i < 16; ++i)
            __builtin_amdgcn_raw_ptr_buffer_load_lds(rs, (LAS void*)(lds + slot * WB + i * 1024), 16, (i & 2) ? v1 : v0, (((t + koff) & 63) * 64 + i * 4) * (NG * 2), 0, 2);
        } else {
#pragma unroll
          for (int i = 0; i < 16; ++i)
            __builtin_amdgcn_raw_ptr_buffer_load_lds(rs, (LAS void*)(lds + slot * WB + i * 1024), 16, (i & 2) ? v1 : v0, (((t + koff) & 63) * 64 + i * 4) * (NG * 2), 0, 0);
        }
      };
      if (remote_x) BAR();
      issue(0, 0, false); issue(1, 1, false);
      unsigned last;
      asm volatile("global_load_dword %0, %1, off sc1" : "=v"(last) : "v"(myws + WS_LAST) : "memory");
      WAIT_V(16); BAR();
      __builtin_amdgcn_s_sleep(WSL);
      issue(2, 2, false); issue(3, 3, false);
      asm volatile("s_waitcnt vmcnt(32)" : "+v"(last) :: "memory");
      BAR();
      const bool stream = drel != 0 && (unsigned)NOW() - (unsigned)__builtin_amdgcn_readfirstlane(last) > 30000u;
      WAIT_V(16); BAR();
      auto run = [&](bool st) {
        for (int t = 2; t < NT - 2; t += 2) {
          if (fidx >= 11) __builtin_amdgcn_s_sleep(WSLS); else if (early) __builtin_amdgcn_s_sleep(WSLF); else __builtin_amdgcn_s_sleep(WSL);
          issue(t + 2, (t + 2) & 3, st); issue(t + 3, (t + 3) & 3, st);
          WAIT_V(32); BAR();
          WAIT_V(16); BAR();
        }
      };
      if (stream) run(true); else run(false);
      WAIT_V(0); BAR();
      return;
    }
    STAMP(0);
    const u64 t0 = NOW();
    GAS unsigned* cnt = (GAS unsigned*)(myws + WS_CNT) + bid;
    GAS unsigned* tmo = (GAS unsigned*)(myws + WS_TMO);
    const GAS void* tab = (const GAS char*)(const GAS void*)__builtin_amdgcn_kernarg_segment_ptr() + (lane >> 3) * 8;
    const int first = half ? POFFB : POFF, last = early ? TF0 + fidx * TFD : drel != 0 ? TEND - SLK : TEND;
    const int pace = (last - first) / (NT - 1);
    const int soft = (early || (fidx >= 11 && drel != 0)) ? 0 : half ? SOFT : SOFTA;
    const int phase = (half || DSA) ? ((bid * 37) & 63) * DESYNC / (64 * DRAMP) : 0;
    auto due_at = [&](int t) { return t0 + (unsigned)(first + t * pace + (t < NT - DRAMP ? DRAMP : NT - t) * phase + (t < SP ? t * soft / SP : (63 - t) * soft / (63 - SP))) / 10u; };
    unsigned call;
    unsigned char* pws;
    bool bad = false;
    if (!half) {
      if (XT) {
        const char* tp = (const char*)A.x + (size_t)(idx * 16 + (lane & 7)) * (KG * 2) + (xcd * 8 + (lane >> 3)) * 128;
        asm volatile("global_load_dword v253, %0, off\n\tglobal_load_dword v253, %1, off" ::"v"(tp), "v"(tp + 8 * KG * 2) : "memory", "v253");
      }
      for (int t = 0; t < NT; ++t) {
        const u64 due = t ? due_at(t) : 0;
        while (NOW() < due) __builtin_amdgcn_s_sleep(1);
        BAR();
        if (t == 0) {
          __builtin_amdgcn_s_sleep(8);
          asm volatile("global_load_dwordx2 v[250:251], %0, off\n\tglobal_load_dword v252, %1, off sc1" ::"v"(tab), "v"(cnt) : "memory", "v250", "v251", "v252");
        }
      }
      unsigned plo, phi, c0;
      asm volatile("s_waitcnt vmcnt(0)\n\tv_mov_b32 %0, v250\n\tv_mov_b32 %1, v251\n\tv_mov_b32 %2, v252\n\ts_nop 0" : "=v"(plo), "=v"(phi), "=v"(c0) :: "memory");
      call = __builtin_amdgcn_readfirstlane(c0) + 1u;
      pws = (unsigned char*)(((u64)phi << 32) | plo);
    } else {
      const int l = ((idx - 4) & 15) * 8 + xcd, pg = l & 15, pj = l >> 4;
      const char* s0 = (const char*)A.x + (size_t)(BM + pg * 16 + (lane >> 3)) * (KG * 2) + (((lane & 7) ^ (lane >> 4)) << 4) + pj * 128;
      const char* s1 = (const char*)A.x + (size_t)(BM + pg * 16 + 8 + (lane >> 3)) * (KG * 2) + (((lane & 7) ^ (4 | (lane >> 4))) << 4) + pj * 128;
      u32x4 r[16];
      unsigned c0;
      asm volatile("global_load_dwordx4 %0, %4, off\n\tglobal_load_dwordx4 %1, %5, off\n\tglobal_load_dword %2, %6, off sc1\n\tglobal_load_dwordx2 %3, %7, off\n\ts_waitcnt vmcnt(0)"
                   : "=&v"(r[0]), "=&v"(r[1]), "=&v"(c0), "=&v"(pws) : "v"(s0), "v"(s1), "v"(cnt), "v"(tab) : "memory");
      call = __builtin_amdgcn_readfirstlane(c0) + 1u;
      const bool flagger = lane >= 8 && (lane & 7) == 0;
      const unsigned xs_mine = (unsigned)(WS_XS + (size_t)rank * XS_SLOT) + lane * 16 + (PADX ? (lane < 56 ? 512 : 640) : 0) + (pj * 32 + pg * 2) * PCS;
      GAS unsigned* fp = (GAS unsigned*)(pws + WS_XF + rank * 512) + pj * 16 + pg;
      const GAS unsigned char* xf = (const GAS unsigned char*)(myws + WS_XF + src * 512) + (lane & 7) * 8;
      int nb = 0;
      bool stored = false, polled = false;
      unsigned pbase = 0, ready = remote_x ? 0 : NT;
      auto burst = [&]() {
        const unsigned vo = xs_mine + nb * 8 * 32 * PCS;
#pragma unroll
        for (int d = 1; d < NR; ++d) {
          asm volatile("global_store_dwordx4 %0, %1, off sc0 sc1\n\ts_nop 1" ::"v"(A.ws[d] + vo), "v"(r[0]) : "memory");
          asm volatile("global_store_dwordx4 %0, %1, off offset:%c2 sc0 sc1\n\ts_nop 1" ::"v"(A.ws[d] + vo), "v"(r[1]), "i"(PCS) : "memory");
        }
        stored = true;
      };
      auto burst_due = [&]() { return NOW() >= t0 + (unsigned)(EOFF + (nb ? 8 * TAUE + (8 * nb + pj - 8) * TAU : pj * TAUE)) / 10u; };
      auto poll = [&]() {
        pbase = ready;
        asm volatile("global_load_dwordx2 v[250:251], %0, off sc0 sc1" :: "v"(xf + ((pbase + (lane >> 3)) & 7) * 64) : "memory", "v250", "v251");
        polled = true;
      };
      auto settle = [&]() {
        unsigned p0, p1;
        asm volatile("s_waitcnt vmcnt(0)\n\tv_mov_b32 %0, v250\n\tv_mov_b32 %1, v251" : "=v"(p0), "=v"(p1) :: "memory");
        if (stored) {
#pragma unroll
          for (int i = 0; i < 14; ++i) r[i] = r[i + 2];
          ++nb;
          if (flagger) __hip_atomic_store(fp, call * 16u + (unsigned)nb, RLX_SYS);
          stored = false;
        }
        if (polled) {
          const unsigned want = call * 16u + ((pbase + (lane >> 3)) >> 3) + 1u;
          const u64 miss = ~__ballot(p0 >= want && p1 >= want);
          ready = pbase + (miss ? (unsigned)__builtin_ctzll(miss) >> 3 : 8u);
          polled = false;
        }
      };
      auto gate = [&](unsigned need) {
        while (ready < need) {
          if (!stored && nb < 8 && burst_due()) burst();
          if (!polled) poll();
          settle();
          if (ready < need && (__hip_atomic_load(tmo, RLX_AGT) == call || NOW() - t0 > DEADLINE)) { bad = true; ready = NT; }
        }
      };
      auto load_rest = [&]() {
        asm volatile("global_load_dwordx4 %0, %6, off offset:1024\n\tglobal_load_dwordx4 %1, %7, off offset:1024\n\tglobal_load_dwordx4 %2, %6, off offset:2048\n\t"
                     "global_load_dwordx4 %3, %7, off offset:2048\n\tglobal_load_dwordx4 %4, %6, off offset:3072\n\tglobal_load_dwordx4 %5, %7, off offset:3072\n\ts_waitcnt vmcnt(0)"
                     : "=&v"(r[2]), "=&v"(r[3]), "=&v"(r[4]), "=&v"(r[5]), "=&v"(r[6]), "=&v"(r[7]) : "v"(s0), "v"(s1) : "memory");
        asm volatile("global_load_dwordx4 %0, %8, off\n\tglobal_load_dwordx4 %1, %9, off\n\tglobal_load_dwordx4 %2, %8, off offset:1024\n\tglobal_load_dwordx4 %3, %9, off offset:1024\n\t"
                     "global_load_dwordx4 %4, %8, off offset:2048\n\tglobal_load_dwordx4 %5, %9, off offset:2048\n\tglobal_load_dwordx4 %6, %8, off offset:3072\n\tglobal_load_dwordx4 %7, %9, off offset:3072\n\t"
                     "s_waitcnt vmcnt(0)"
                     : "=&v"(r[8]), "=&v"(r[9]), "=&v"(r[10]), "=&v"(r[11]), "=&v"(r[12]), "=&v"(r[13]), "=&v"(r[14]), "=&v"(r[15]) : "v"(s0 + 4096), "v"(s1 + 4096) : "memory");
      };
      if (pj < 3) {
        while (!burst_due()) __builtin_amdgcn_s_sleep(1);
        burst();
        WAIT_V(0);
        if (flagger) __hip_atomic_store(fp, call * 16u + 1u, RLX_SYS);
      }
      load_rest();
      settle();
      STAMP(8);
      if (remote_x) { gate(2); BAR(); }
      for (const u64 due = due_at(0); NOW() < due;) {
        if (!stored && nb == 0 && burst_due()) burst();
        if (stored && !vm_busy()) settle();
        __builtin_amdgcn_s_sleep(1);
      }
      STAMP(9);
      for (int t = 0; t < NT; ++t) {
        const unsigned need = t + 3 < NT ? t + 3 : NT;
        gate(need);
        const u64 due = due_at(t);
        while (NOW() < due) __builtin_amdgcn_s_sleep(1);
        BAR();
        const int c = 8 * nb + pj;
        if (nb < 8 || ready < (unsigned)NT) {
          __builtin_amdgcn_s_sleep(DSL);
          if ((stored || polled) && (!vm_busy() || (stored && t + 4 >= c))) settle();
          else if ((t & 1) == 0) {
            if (!stored && nb < 8 && (t + 7 >= c || burst_due())) burst();
            if (!polled && ready < (unsigned)NT && ready < need + LEAD) poll();
          }
        }
      }
      settle();
      while (nb < 8) { burst(); settle(); }
    }
    const unsigned par = call & 1u;
    if (bad && lane == 0) __hip_atomic_store(tmo, call, RLX_AGT);
    GAS u64* gp = (GAS u64*)(pws + WS_GLB + (par * 8 + (lane & 7)) * 128) + rank;
    const unsigned late = (unsigned)(NOW() - due_at(NT - 1));
    if (lane == 0) red[6] = par | (late > LT2 ? 4u : late > LT1 ? 2u : 0u);
    WAIT_L(0);
    BAR();
    STAMP(1);
    {
      const unsigned m = dpp_umax<0x4E>(dpp_umax<0xB1>(red[lane & 3]));
      if (lane == 0) __hip_atomic_store((GAS u64*)(myws + WS_LOC) + par * NWG + bid, ((u64)call << 32) | m, RLX_AGT);
    }
    if (early) {
      BAR();
      if (lane == 0) __hip_atomic_store((GAS unsigned*)(A.ws[drel] + WS_YF + (rank * TPP + tile) * 128), call, RLX_SYS);
      STAMP(6);
      GAS unsigned* f = lane == 1 ? tmo : (GAS unsigned*)(myws + WS_YF + (((rank - drel) & 7) * TPP + tile) * 128);
      for (;;) {
        const unsigned v = __hip_atomic_load(f, RLX_SYS);
        if ((unsigned)__builtin_amdgcn_readlane(v, 0) >= call) break;
        if ((unsigned)__builtin_amdgcn_readlane(v, 1) == call || NOW() - t0 > DEADLINE) { bad = true; break; }
        __builtin_amdgcn_s_sleep(2);
      }
      if (lane == 0) red[5] = bad;
      WAIT_L(0);
      BAR();
      STAMP(7);
    }
    if (bid == 14 * 8) {
      GAS u64* g = (GAS u64*)(myws + WS_LOC) + par * NWG + lane;
      unsigned m;
      for (;;) {
        const u64 a = __hip_atomic_load(g, RLX_AGT), b = __hip_atomic_load(g + 64, RLX_AGT), c = __hip_atomic_load(g + 128, RLX_AGT), d = __hip_atomic_load(g + 192, RLX_AGT);
        const bool ok = (unsigned)(a >> 32) == call && (unsigned)(b >> 32) == call && (unsigned)(c >> 32) == call && (unsigned)(d >> 32) == call;
        m = max(max((unsigned)a, (unsigned)b), max((unsigned)c, (unsigned)d));
        if (__all(ok)) break;
        if (NOW() - t0 > DEADLINE) { bad = true; break; }
      }
      m = wave_umax(m);
      if (!bad) __hip_atomic_store(gp, ((u64)call << 32) | m, RLX_SYS);
      STAMP(2);
    }
    {
      GAS u64* g = lane == 8 ? (GAS u64*)tmo : (GAS u64*)(myws + WS_GLB + (par * 8 + xcd) * 128) + (lane & 7);
      unsigned gm;
      for (;;) {
        const u64 a = __hip_atomic_load(g, RLX_SYS);
        gm = lane == 8 ? 0u : (unsigned)a;
        if (__builtin_amdgcn_readlane((unsigned)a, 8) == call) { bad = true; break; }
        if (__all(lane == 8 || (unsigned)(a >> 32) == call)) break;
        if (bad || NOW() - t0 > DEADLINE) { bad = true; break; }
        __builtin_amdgcn_s_sleep(1);
      }
      gm = row_umax(gm);
      if (lane == 0) red[4] = bad ? 0x7fc00000u : __float_as_uint(448.0f / __uint_as_float(gm));
      if (bad && lane == 0) __hip_atomic_store(tmo, call, RLX_AGT);
    }
    WAIT_L(0);
    BAR();
    if (lane == 0) __hip_atomic_store(cnt, call, RLX_AGT);
    if (bid == 15 * 8 && lane == 0) __hip_atomic_store((GAS unsigned*)(myws + WS_LAST), (unsigned)NOW(), RLX_AGT);
    STAMP(3);
    BAR();
    STAMP(4);
    if (drel != 0 && !early && lane == 0)
      __hip_atomic_store((GAS unsigned*)(A.ws[drel] + WS_DONE) + rank * TPP + tile, call, RLX_SYS);
    if (bid == 15 * 8) {
      const bool mine = (lane >> 3) == rank;
      GAS u64* f = mine ? (GAS u64*)tmo : (GAS u64*)(myws + WS_DONE) + lane;
      const bool e0 = early_index(2 * (lane & 7)) < NF, e1 = early_index(2 * (lane & 7) + 1) < NF;
      for (;;) {
        const u64 a = __hip_atomic_load(f, RLX_SYS);
        const bool ok = mine || ((e0 || (unsigned)a >= call) && (e1 || (unsigned)(a >> 32) >= call));
        if (__all(ok)) break;
        if (__any(mine && (unsigned)a == call) || NOW() - t0 > DEADLINE) {
          if (lane == 0) { __hip_atomic_store(tmo, call, RLX_AGT); A.out[0][(size_t)(rank * MS + half * BM) * NS + tc * BN] = 0x7f; }
          break;
        }
      }
    }
    STAMP(5);
    return;
  }

  const int wm = wid >> 1, wn = wid & 1;
  const int fr = lane & 15, g = lane >> 4;
  const int xa0 = X_OFF + (wm * 128 + fr) * 128 + ((g ^ ((fr >> 1) & 7)) << 4);
  const int xa1 = xa0 ^ 64;
  const int q = fr >> 2, p = fr & 3, h = q | ((g & 1) << 2);
  int wa[4];
#pragma unroll
  for (int nt = 0; nt < 4; ++nt) wa[nt] = (8 * g + q) * 256 + (((4 * wn + nt) ^ h) << 5) + p * 8;
  f32x4 acc[8][4];
#pragma unroll
  for (int a = 0; a < 8; ++a)
#pragma unroll
    for (int b = 0; b < 4; ++b) { acc[a][b] = (f32x4){0.f, 0.f, 0.f, 0.f}; asm volatile("" : "+v"(acc[a][b])); }
  s16x4 WA[4][4], WC[4][4];
  bf16x8 X[8][2];
#define MF(a, b, Wf, ks) do { if (!skip) { const bf16x8 wv = __builtin_bit_cast(bf16x8, __builtin_shufflevector(Wf[b][2 * ks], Wf[b][2 * ks + 1], 0, 1, 2, 3, 4, 5, 6, 7)); \
    asm volatile("v_mfma_f32_16x16x32_bf16 %0, %1, %2, %0" : "+v"(acc[a][b]) : "v"(wv), "v"(X[a][ks])); } SCHED(); } while (0)
#define RW(Wf, b, i) do { Wf[b][i] = __builtin_amdgcn_ds_read_tr16_b64_v4i16((LAS s16x4*)(lds + wp[b] + wimm + (i >> 1) * 8192 + (i & 1) * 1024)); SCHED(); } while (0)
#define RX(a, ks) do { X[a][ks] = *(LAS bf16x8*)(lds + (ks ? xp1 : xp0) + a * 2048); SCHED(); } while (0)
#define GRP(a, r0, r1, r2, r3) MF(a, 0, Wc, 0); r0; MF(a, 0, Wc, 1); MF(a, 1, Wc, 0); r1; MF(a, 1, Wc, 1); MF(a, 2, Wc, 0); r2; MF(a, 2, Wc, 1); MF(a, 3, Wc, 0); r3; MF(a, 3, Wc, 1);
  auto step = [&](s16x4 (&Wp)[4][4], s16x4 (&Wc)[4][4], int xoff, const int (&wp)[4], int wimm, bool first) {
    bool skip = first;
    const int xp0 = xa0 + xoff, xp1 = xa1 + xoff;
    MF(5, 0, Wp, 0); RW(Wc, 0, 0); MF(5, 0, Wp, 1); RW(Wc, 0, 1); MF(5, 1, Wp, 0); RW(Wc, 0, 2); MF(5, 1, Wp, 1); RW(Wc, 0, 3);
    MF(5, 2, Wp, 0); RW(Wc, 1, 0); MF(5, 2, Wp, 1); RW(Wc, 1, 1); MF(5, 3, Wp, 0); RW(Wc, 1, 2); MF(5, 3, Wp, 1); RW(Wc, 1, 3);
    MF(6, 0, Wp, 0); RW(Wc, 2, 0); MF(6, 0, Wp, 1); RW(Wc, 2, 1); MF(6, 1, Wp, 0); RW(Wc, 2, 2); MF(6, 1, Wp, 1); RW(Wc, 2, 3);
    MF(6, 2, Wp, 0); RW(Wc, 3, 0); MF(6, 2, Wp, 1); RW(Wc, 3, 1); MF(6, 3, Wp, 0); RW(Wc, 3, 2); MF(6, 3, Wp, 1); RW(Wc, 3, 3);
    MF(7, 0, Wp, 0); RX(0, 0); MF(7, 0, Wp, 1); RX(0, 1); MF(7, 1, Wp, 0); MF(7, 1, Wp, 1); RX(1, 0);
    MF(7, 2, Wp, 0); MF(7, 2, Wp, 1); RX(1, 1); MF(7, 3, Wp, 0); MF(7, 3, Wp, 1);
    skip = false;
    GRP(0, RX(2, 0), RX(2, 1), , )
    GRP(1, RX(3, 0), RX(3, 1), , )
    GRP(2, RX(4, 0), RX(4, 1), RX(5, 0), RX(5, 1))
    GRP(3, RX(6, 0), RX(6, 1), RX(7, 0), RX(7, 1))
    GRP(4, , , , )
  };
  if (remote_x) BAR();
  BAR();
  SCHED();
  step(WC, WA, 0, wa, 0, true);
  int xoff = 0;
  for (int t = 1; t < NT - 1; t += 2) {
    xoff = xoff == (NXS - 1) * XB ? 0 : xoff + XB;
    WAIT_L(0); BAR(); SCHED();
    step(WA, WC, xoff, wa, WB, false);
    xoff = xoff == (NXS - 1) * XB ? 0 : xoff + XB;
#pragma unroll
    for (int b = 0; b < 4; ++b) wa[b] ^= 2 * WB;
    WAIT_L(0); BAR(); SCHED();
    step(WC, WA, xoff, wa, 0, false);
  }
  xoff = xoff == (NXS - 1) * XB ? 0 : xoff + XB;
  WAIT_L(0); BAR(); SCHED();
  step(WA, WC, xoff, wa, WB, false);
  float mx = 0.f;
#define MX(a, b) do { asm volatile("v_max3_f32 %0, |%1|, |%2|, %0\n\tv_max3_f32 %0, |%3|, |%4|, %0" : "+v"(mx) : "v"(acc[a][b][0]), "v"(acc[a][b][1]), "v"(acc[a][b][2]), "v"(acc[a][b][3])); SCHED(); } while (0)
#define MX2(a, b) MX(a, b); MX(a + 1, b)
  {
    const bool skip = false;
    auto& Wc = WC;
    GRP(5, MX2(0, 0), MX2(0, 1), MX2(0, 2), MX2(0, 3))
    GRP(6, MX2(2, 0), MX2(2, 1), MX2(2, 2), MX2(2, 3))
    GRP(7, MX2(4, 0), MX2(4, 1), MX2(4, 2), MX2(4, 3))
    MX(6, 0); MX(6, 1); MX(6, 2); MX(6, 3);
    asm volatile("s_nop 7" ::: "memory");
    MX(7, 0); MX(7, 1); MX(7, 2); MX(7, 3);
    asm volatile("s_nop 1" : "+v"(mx));
  }
  {
    const unsigned m = wave_umax(__float_as_uint(mx));
    if (lane == 0) red[wid] = m;
  }
  WAIT_L(0);
  BAR();
  const unsigned lo = (unsigned)((wm * 128 + fr) * NS + wn * 64 + g * 16);
  unsigned char* op = A.out[drel] + (size_t)(src * MS + half * BM) * NS + tc * BN + lo;
  if (early) {
    const unsigned lvl = __builtin_amdgcn_readfirstlane(red[6]) >> 1;
    const size_t slot = WS_YS + (size_t)((red[6] & 1u) * (NR * TPP) + tile) * YS_TILE + wid * YS_WAVE + lane * 16;
    unsigned char* sp = A.ws[drel] + slot + (size_t)rank * TPP * YS_TILE;
#define PUT(off, val) do { const u32x4 o_ = val; asm volatile("global_store_dwordx4 %0, %1, off offset:%c2 sc0 sc1\n\ts_nop 1" ::"v"(sp + c * 4096), "v"(o_), "i"(off) : "memory"); if (lvl == 0) WAIT_V(FTH); else if (lvl == 1) WAIT_V(FTH1); else WAIT_V(FTH2); } while (0)
    unsigned head = (__builtin_bit_cast(u16, (__bf16)mx) >> 8) & 0x7fu;
    const unsigned brep = head * 0x01010101u | 0x80808080u;
#pragma unroll
    for (int c = 0; c < 4; ++c) {
      unsigned L[8], H[8], N[4], ovf = 0;
#pragma unroll
      for (int j = 0; j < 8; ++j) {
        const f32x4 v = acc[2 * c + (j >> 2)][j & 3];
        typedef __bf16 bf16x2 __attribute__((ext_vector_type(2)));
        const unsigned p0 = __builtin_bit_cast(unsigned, (bf16x2){(__bf16)v[0], (__bf16)v[1]}), p1 = __builtin_bit_cast(unsigned, (bf16x2){(__bf16)v[2], (__bf16)v[3]});
        L[j] = __builtin_amdgcn_perm(p1, p0, 0x06040200u);
        H[j] = __builtin_amdgcn_perm(p1, p0, 0x07050301u);
      }
      PUT(0, ((u32x4){L[0], L[1], L[2], L[3]}));
      PUT(1024, ((u32x4){L[4], L[5], L[6], L[7]}));
#pragma unroll
      for (int k = 0; k < 4; ++k) {
        const unsigned d0 = brep - (H[2 * k] & 0x7f7f7f7fu), d1 = brep - (H[2 * k + 1] & 0x7f7f7f7fu);
        ovf |= ((d0 ^ 0x80808080u) | (d1 ^ 0x80808080u)) & 0xf8f8f8f8u;
        N[k] = (d0 & 0x07070707u) | ((H[2 * k] & 0x80808080u) >> 4) | ((d1 & 0x07070707u) << 4) | (H[2 * k + 1] & 0x80808080u);
      }
      if (__any(ovf != 0)) {
        PUT(2048, ((u32x4){H[0], H[1], H[2], H[3]}));
        PUT(3072, ((u32x4){H[4], H[5], H[6], H[7]}));
        head |= 256u << c;
      } else PUT(2048, ((u32x4){N[0], N[1], N[2], N[3]}));
    }
    asm volatile("global_store_dword %0, %1, off sc0 sc1" ::"v"(sp + 16384 - lane * 12), "v"(head) : "memory");
    WAIT_V(0);
    BAR();
    BAR();
    const int from = (rank - drel) & 7;
    op = A.out[0] + (size_t)(from * MS + half * BM) * NS + tc * BN + lo;
    if (__builtin_amdgcn_readfirstlane(red[5]) == 0) {
      const unsigned char* lp = myws + slot + (size_t)from * TPP * YS_TILE;
      u32x4 r[12];
      asm volatile("global_load_dword %0, %13, off sc0 sc1\n\t"
                   "global_load_dwordx4 %1, %14, off offset:-4096 sc0 sc1\n\tglobal_load_dwordx4 %2, %14, off offset:-3072 sc0 sc1\n\tglobal_load_dwordx4 %3, %14, off offset:-2048 sc0 sc1\n\t"
                   "global_load_dwordx4 %4, %14, off sc0 sc1\n\tglobal_load_dwordx4 %5, %14, off offset:1024 sc0 sc1\n\tglobal_load_dwordx4 %6, %14, off offset:2048 sc0 sc1\n\t"
                   "global_load_dwordx4 %7, %15, off offset:-4096 sc0 sc1\n\tglobal_load_dwordx4 %8, %15, off offset:-3072 sc0 sc1\n\tglobal_load_dwordx4 %9, %15, off offset:-2048 sc0 sc1\n\t"
                   "global_load_dwordx4 %10, %15, off sc0 sc1\n\tglobal_load_dwordx4 %11, %15, off offset:1024 sc0 sc1\n\tglobal_load_dwordx4 %12, %15, off offset:2048 sc0 sc1\n\ts_waitcnt vmcnt(0)"
                   : "=&v"(head), "=&v"(r[0]), "=&v"(r[1]), "=&v"(r[2]), "=&v"(r[3]), "=&v"(r[4]), "=&v"(r[5]), "=&v"(r[6]), "=&v"(r[7]), "=&v"(r[8]), "=&v"(r[9]), "=&v"(r[10]), "=&v"(r[11])
                   : "v"(lp + 16384 - lane * 12), "v"(lp + 4096), "v"(lp + 12288) : "memory");
      const unsigned raw = __builtin_amdgcn_readfirstlane(head) >> 8, base = (head & 0x7fu) * 0x01010101u;
#pragma unroll
      for (int c = 0; c < 4; ++c) {
        unsigned H[8];
        const u32x4 n4 = r[3 * c + 2];
        if ((raw >> c) & 1u) {
          u32x4 h2;
          asm volatile("global_load_dwordx4 %0, %1, off offset:3072 sc0 sc1\n\ts_waitcnt vmcnt(0)" : "=&v"(h2) : "v"(lp + c * 4096) : "memory");
#pragma unroll
          for (int k = 0; k < 4; ++k) { H[k] = n4[k]; H[4 + k] = h2[k]; }
        } else {
#pragma unroll
          for (int k = 0; k < 4; ++k) {
            H[2 * k] = (base - (n4[k] & 0x07070707u)) | ((n4[k] & 0x08080808u) << 4);
            H[2 * k + 1] = (base - ((n4[k] >> 4) & 0x07070707u)) | (n4[k] & 0x80808080u);
          }
        }
#pragma unroll
        for (int j = 0; j < 8; ++j) {
          const unsigned l = r[3 * c + (j >> 2)][j & 3];
          acc[2 * c + (j >> 2)][j & 3] = (f32x4){__uint_as_float(__builtin_amdgcn_perm(H[j], l, 0x04000c0cu)), __uint_as_float(__builtin_amdgcn_perm(H[j], l, 0x05010c0cu)),
                                                 __uint_as_float(__builtin_amdgcn_perm(H[j], l, 0x06020c0cu)), __uint_as_float(__builtin_amdgcn_perm(H[j], l, 0x07030c0cu))};
        }
      }
    }
  }
  BAR();
  const float inv = __uint_as_float(red[4]);
  const bool throttle = drel != 0 && !early;
#pragma unroll
  for (int a = 0; a < 8; ++a) {
    unsigned d[4];
#pragma unroll
    for (int b = 0; b < 4; ++b) {
      const f32x4 v = acc[a][b] * inv;
      int pk = __builtin_amdgcn_cvt_pk_fp8_f32(v[0], v[1], 0, false);
      pk = __builtin_amdgcn_cvt_pk_fp8_f32(v[2], v[3], pk, true);
      d[b] = pk;
    }
    { auto r = __builtin_amdgcn_permlane32_swap(d[0], d[2], false, false); d[0] = r[0]; d[2] = r[1]; }
    { auto r = __builtin_amdgcn_permlane32_swap(d[1], d[3], false, false); d[1] = r[0]; d[3] = r[1]; }
    { auto r = __builtin_amdgcn_permlane16_swap(d[0], d[1], false, false); d[0] = r[0]; d[1] = r[1]; }
    { auto r = __builtin_amdgcn_permlane16_swap(d[2], d[3], false, false); d[2] = r[0]; d[3] = r[1]; }
    const u32x4 o = {d[0], d[1], d[2], d[3]};
    asm volatile("global_store_dwordx4 %0, %1, off sc0 sc1\n\ts_nop 1" ::"v"(op + (size_t)a * 16 * NS), "v"(o) : "memory");
    if (throttle) WAIT_V(QTH);
  }
  WAIT_V(0);
  BAR();
}

extern "C" void kernel_launch(int rank, int nranks, ncclComm_t comm, hipStream_t stream,
                              void* const* d_in, const int* in_sizes, int n_in,
                              void* d_out, int out_size, const int* in_dtypes, int out_dtype,
                              void* const* d_out_all, void* const* d_ws_all, size_t ws_bytes) {
  (void)comm; (void)in_sizes; (void)n_in; (void)d_out; (void)out_size; (void)in_dtypes; (void)out_dtype;
  if (nranks != NR || (unsigned)rank >= (unsigned)NR || ws_bytes < WS_END) { fprintf(stderr, "unexpected configuration\n"); return; }
  Args a;
  a.x = (const u16*)d_in[0]; a.w = (const u16*)d_in[1];
  for (int i = 0; i < NR; ++i) { a.out[i] = (unsigned char*)d_out_all[(rank + i) & 7]; a.ws[i] = (unsigned char*)d_ws_all[(rank + i) & 7]; }
  a.rank = rank; a.pad = 0;
  hipLaunchKernelGGL(fused_gemm_a2a, dim3(NWG), dim3(512), 0, stream, a);
  HIP_CHECK(hipGetLastError());
}
```

```cpp
#include <hip/hip_runtime.h>
#include <rccl/rccl.h>
#include <cstdio>

#define HIP_CHECK(expr) do { hipError_t _e = (expr); if (_e != hipSuccess) { \
    fprintf(stderr, "HIP error %s at %s:%d\n", hipGetErrorString(_e), __FILE__, __LINE__); \
    return; } } while (0)

#define LAS __attribute__((address_space(3)))
#define GAS __attribute__((address_space(1)))
typedef __bf16 bf16x8 __attribute__((ext_vector_type(8)));
typedef short s16x4 __attribute__((ext_vector_type(4)));
typedef float f32x4 __attribute__((ext_vector_type(4)));
typedef unsigned u32x4 __attribute__((ext_vector_type(4)));
typedef unsigned short u16;
typedef unsigned long long u64;

namespace {
constexpr int NR = 8;
constexpr int MG = 4096, NG = 16384, KG = 4096;
constexpr int MS = MG / NR;
constexpr int NS = NG / NR;
constexpr int BM = 256, BN = 128, BK = 64, NT = KG / BK;
constexpr int XB = BM * BK * 2, WB = BK * BN * 2;
constexpr int NXS = 3, NWS = 4;
constexpr int X_OFF = NWS * WB;
constexpr int LDS_BYTES = NXS * XB + NWS * WB;
constexpr int NWG = 256;
constexpr int TPP = 16;
constexpr u64 DEADLINE = 150000000ull;
constexpr int NF = 12;
constexpr int POFF = 1000, POFFB = 7000;
constexpr int TF0 = 48500, TFD = 960;
constexpr int TEND = 59800;
constexpr int SP = 44;
constexpr int SOFTF = 200, SOFTS = 1000, SOFTA = 5000, SOFT = 2400;
constexpr int DESYNC = 1600, DRAMP = 32;
constexpr int EOFF = 600, TAUE = 640, TAU = 730;
constexpr int LEAD = 6;
constexpr int WSL = 5, WSLS = 7, WSLF = 4, DSL = 9;
constexpr int FTH = 12, FTH1 = 3, FTH2 = 1, QTH = 3;
constexpr unsigned LT1 = 40, LT2 = 120;

constexpr size_t WS_TMO = 0;
constexpr size_t WS_LAST = 256;
constexpr size_t WS_DONE = 1024;
constexpr size_t WS_GLB = 4096;
constexpr size_t WS_LOC = 8192;
constexpr size_t WS_XF = 16384;
constexpr size_t WS_YF = 32768;
constexpr size_t WS_CNT = 65536;
constexpr size_t WS_XS = 2u << 20;
constexpr int PCS = 2048;
constexpr size_t XS_SLOT = (size_t)NT * 32 * PCS;
constexpr size_t WS_YS = WS_XS + NR * XS_SLOT;
constexpr int YS_WAVE = 17 * 1024;
constexpr size_t YS_TILE = 4 * YS_WAVE;
constexpr size_t WS_END = WS_YS + (size_t)2 * NR * TPP * YS_TILE;
static_assert(NF <= TPP && WS_YF + NR * TPP * 256 <= WS_CNT && WS_CNT + NWG * 256 <= (1u << 20), "layout");

struct Args {
  unsigned char* ws[NR];
  unsigned char* out[NR];
  const u16* x; const u16* w;
  int rank; int pad;
};
}

#define BAR() do { asm volatile("" ::: "memory"); __builtin_amdgcn_s_barrier(); asm volatile("" ::: "memory"); } while (0)
#define WAIT_V(n) asm volatile("s_waitcnt vmcnt(%0)" ::"n"(n) : "memory")
#define WAIT_L(n) asm volatile("s_waitcnt lgkmcnt(%0)" ::"n"(n) : "memory")
#define SCHED() __builtin_amdgcn_sched_barrier(0)
#define RLX_SYS __ATOMIC_RELAXED, __HIP_MEMORY_SCOPE_SYSTEM
#define RLX_AGT __ATOMIC_RELAXED, __HIP_MEMORY_SCOPE_AGENT
#define NOW() __builtin_amdgcn_s_memrealtime()

template <int CTRL> __device__ __forceinline__ unsigned dpp_umax(unsigned v) {
  const unsigned t = __builtin_amdgcn_update_dpp(v, v, CTRL, 0xf, 0xf, false);
  return t > v ? t : v;
}
__device__ __forceinline__ unsigned row_umax(unsigned v) {
  v = dpp_umax<0xB1>(v); v = dpp_umax<0x4E>(v); v = dpp_umax<0x141>(v); return dpp_umax<0x140>(v);
}
__device__ __forceinline__ unsigned wave_umax(unsigned v) {
  v = row_umax(v);
  const unsigned a = __builtin_amdgcn_readlane(v, 0), b = __builtin_amdgcn_readlane(v, 16), c = __builtin_amdgcn_readlane(v, 32), d = __builtin_amdgcn_readlane(v, 48);
  return max(max(a, b), max(c, d));
}

__device__ __forceinline__ unsigned vm_busy() { return __builtin_amdgcn_s_getreg(7 | (31 << 11)) & 0x00C0000Fu; }

__global__ __launch_bounds__(512, 2) void fused_gemm_a2a(Args A) {
  __shared__ __attribute__((aligned(16))) char smem[LDS_BYTES];
  LAS char* lds = (LAS char*)smem;
  const int tid = threadIdx.x, wid = __builtin_amdgcn_readfirstlane(tid >> 6), lane = tid & 63;
  const int bid = blockIdx.x, xcd = bid & 7, idx = (bid >> 3) & 31;
  const int rank = A.rank;
  int src = rank, half = 0, drel = 0, tc;
  if (idx < 14) { const int a = xcd * 14 + idx; drel = 1 + (a >> 4); tc = a & 15; }
  else if (idx < 16) tc = (xcd >> 1) * 4 + (xcd & 1) * 2 + (idx - 14);
  else { const int k = idx - 16; src = (rank + (xcd & 1) * 4 + (k >> 2)) & 7; half = 1; tc = (xcd >> 1) * 4 + (k & 3); }
  const bool remote_x = src != rank;
  const int n0 = ((rank + drel) & 7) * NS + tc * BN;
  const int koff = half ? 0 : xcd * 8;
  auto early_index = [](int c) { return (c & 3) == 1 ? 12 + (c >> 2) : c - ((c + 2) >> 2); };
  const int fidx = early_index(tc);
  const bool early = drel != 0 && fidx < NF;
  const int tile = tc;
  unsigned char* myws = A.ws[0];
  LAS unsigned* red = (LAS unsigned*)lds;

  if (wid >= 4) {
    if (wid < 6) {
      const int j = wid - 4;
      LAS char* base = lds + X_OFF + j * 16384;
      if (!remote_x) {
        auto rs = __builtin_amdgcn_make_buffer_rsrc((void*)(A.x + (size_t)(half * BM + j * 128) * KG), 0, 128 * KG * 2, 0x00020000);
        const int v0 = (lane >> 3) * (KG * 2) + (((lane & 7) ^ (lane >> 4)) << 4);
        const int v1 = (lane >> 3) * (KG * 2) + (((lane & 7) ^ (4 | (lane >> 4))) << 4);
        auto issue = [&](int t, int slot) {
#pragma unroll
          for (int i = 0; i < 16; ++i)
            __builtin_amdgcn_raw_ptr_buffer_load_lds(rs, (LAS void*)(base + slot * XB + i * 1024), 16, (i & 1) ? v1 : v0, i * 8 * (KG * 2) + ((t + koff) & 63) * 128, 0, 0);
        };
        issue(0, 0); issue(1, 1);
        int slot = 2;
        WAIT_V(16); BAR();
        for (int t = 0; t < NT - 1; ++t) {
          if (t + 2 < NT) { issue(t + 2, slot); WAIT_V(16); } else WAIT_V(0);
          slot = slot == NXS - 1 ? 0 : slot + 1;
          BAR();
        }
      } else {
        const unsigned char* sb = myws + WS_XS + (size_t)src * XS_SLOT + j * 16 * PCS;
        const unsigned vo = lane * 16 + (lane < 56 ? 512 : 640);
        auto issue = [&](int t, int slot) {
#pragma unroll
          for (int i = 0; i < 16; ++i)
            __builtin_amdgcn_global_load_lds((const GAS void*)(sb + (size_t)((t * 32 + i) * PCS) + vo), (LAS void*)(base + slot * XB + i * 1024), 16, 0, 17);
        };
        BAR();
        issue(0, 0); issue(1, 1);
        int slot = 2;
        WAIT_V(16); BAR();
        for (int t = 0; t < NT - 1; ++t) {
          if (t + 2 < NT) { issue(t + 2, slot); WAIT_V(16); } else WAIT_V(0);
          slot = slot == NXS - 1 ? 0 : slot + 1;
          BAR();
        }
      }
      return;
    }
    if (wid == 6) {
      auto rs = __builtin_amdgcn_make_buffer_rsrc((void*)(A.w + n0), 0, (unsigned)((size_t)KG * NG * 2 - (size_t)n0 * 2), 0x00020000);
      const int v0 = (lane >> 4) * (NG * 2) + (((lane & 15) ^ ((lane >> 4) << 1)) << 4);
      const int v1 = (lane >> 4) * (NG * 2) + (((lane & 15) ^ ((4 | (lane >> 4)) << 1)) << 4);
      auto issue = [&](int t, int slot, bool stream) {
        if (stream) {
#pragma unroll
          for (int i = 0; i < 16; ++i)
            __builtin_amdgcn_raw_ptr_buffer_load_lds(rs, (LAS void*)(lds + slot * WB + i * 1024), 16, (i & 2) ? v1 : v0, (((t + koff) & 63) * 64 + i * 4) * (NG * 2), 0, 2);
        } else {
#pragma unroll
          for (int i = 0; i < 16; ++i)
            __builtin_amdgcn_raw_ptr_buffer_load_lds(rs, (LAS void*)(lds + slot * WB + i * 1024), 16, (i & 2) ? v1 : v0, (((t + koff) & 63) * 64 + i * 4) * (NG * 2), 0, 0);
        }
      };
      if (remote_x) BAR();
      issue(0, 0, false); issue(1, 1, false);
      unsigned last;
      asm volatile("global_load_dword %0, %1, off sc1" : "=v"(last) : "v"(myws + WS_LAST) : "memory");
      WAIT_V(16); BAR();
      __builtin_amdgcn_s_sleep(WSL);
      issue(2, 2, false); issue(3, 3, false);
      asm volatile("s_waitcnt vmcnt(32)" : "+v"(last) :: "memory");
      BAR();
      const bool stream = drel != 0 && (unsigned)NOW() - (unsigned)__builtin_amdgcn_readfirstlane(last) > 30000u;
      WAIT_V(16); BAR();
      auto run = [&](bool st) {
        for (int t = 2; t < NT - 2; t += 2) {
          if (fidx >= 11) __builtin_amdgcn_s_sleep(WSLS); else if (early) __builtin_amdgcn_s_sleep(WSLF); else __builtin_amdgcn_s_sleep(WSL);
          issue(t + 2, (t + 2) & 3, st); issue(t + 3, (t + 3) & 3, st);
          WAIT_V(32); BAR();
          WAIT_V(16); BAR();
        }
      };
      if (stream) run(true); else run(false);
      WAIT_V(0); BAR();
      return;
    }
    const u64 t0 = NOW();
    GAS unsigned* cnt = (GAS unsigned*)(myws + WS_CNT + bid * 256);
    const unsigned one = 1;
    u64 ex;
    GAS unsigned* tmo = (GAS unsigned*)(myws + WS_TMO);
    const GAS void* tab = (const GAS char*)(const GAS void*)__builtin_amdgcn_kernarg_segment_ptr() + (lane >> 3) * 8;
    const int first = half ? POFFB : POFF, last = early ? TF0 + fidx * TFD : TEND;
    const int pace = (last - first) / (NT - 1);
    const int soft = drel != 0 ? (fidx >= 11 ? SOFTS : fidx * SOFTF) : half ? SOFT : SOFTA;
    const int phase = ((bid * 37) & 63) * DESYNC / (64 * DRAMP);
    auto due_at = [&](int t) { return t0 + (unsigned)(first + t * pace + (t < NT - DRAMP ? DRAMP : NT - t) * phase + (t < SP ? t * soft / SP : (63 - t) * soft / (63 - SP))) / 10u; };
    unsigned call;
    unsigned char* pws;
    bool bad = false;
    if (!half) {
      {
        const char* tp = (const char*)A.x + (size_t)(idx * 16 + (lane & 7)) * (KG * 2) + (xcd * 8 + (lane >> 3)) * 128;
        asm volatile("global_load_dword v253, %0, off\n\tglobal_load_dword v253, %1, off" ::"v"(tp), "v"(tp + 8 * KG * 2) : "memory", "v253");
      }
      for (int t = 0; t < NT; ++t) {
        const u64 due = t ? due_at(t) : 0;
        while (NOW() < due) __builtin_amdgcn_s_sleep(1);
        BAR();
        if (t == 0) {
          __builtin_amdgcn_s_sleep(8);
          asm volatile("global_load_dwordx2 v[250:251], %1, off\n\ts_mov_b64 %0, exec\n\ts_mov_b64 exec, 1\n\tglobal_atomic_add v252, %2, %3, off sc0\n\ts_mov_b64 exec, %0"
                       : "=&s"(ex) : "v"(tab), "v"(cnt), "v"(one) : "memory", "v250", "v251", "v252");
        }
      }
      unsigned plo, phi, c0;
      asm volatile("s_waitcnt vmcnt(0)\n\tv_mov_b32 %0, v250\n\tv_mov_b32 %1, v251\n\tv_mov_b32 %2, v252\n\ts_nop 0" : "=v"(plo), "=v"(phi), "=v"(c0) :: "memory");
      call = __builtin_amdgcn_readfirstlane(c0) + 1u;
      pws = (unsigned char*)(((u64)phi << 32) | plo);
    } else {
      const int l = ((idx - 4) & 15) * 8 + xcd, pg = l & 15, pj = l >> 4;
      const char* s0 = (const char*)A.x + (size_t)(BM + pg * 16 + (lane >> 3)) * (KG * 2) + (((lane & 7) ^ (lane >> 4)) << 4) + pj * 128;
      const char* s1 = (const char*)A.x + (size_t)(BM + pg * 16 + 8 + (lane >> 3)) * (KG * 2) + (((lane & 7) ^ (4 | (lane >> 4))) << 4) + pj * 128;
      u32x4 r[16];
      unsigned c0;
      asm volatile("global_load_dwordx4 %0, %5, off\n\tglobal_load_dwordx4 %1, %6, off\n\tglobal_load_dwordx2 %3, %8, off\n\t"
                   "s_mov_b64 %4, exec\n\ts_mov_b64 exec, 1\n\tglobal_atomic_add %2, %7, %9, off sc0\n\ts_mov_b64 exec, %4\n\ts_waitcnt vmcnt(0)"
                   : "=&v"(r[0]), "=&v"(r[1]), "=&v"(c0), "=&v"(pws), "=&s"(ex) : "v"(s0), "v"(s1), "v"(cnt), "v"(tab), "v"(one) : "memory");
      call = __builtin_amdgcn_readfirstlane(c0) + 1u;
      const bool flagger = lane >= 8 && (lane & 7) == 0;
      const unsigned xs_mine = (unsigned)(WS_XS + (size_t)rank * XS_SLOT) + lane * 16 + (lane < 56 ? 512 : 640) + (pj * 32 + pg * 2) * PCS;
      GAS unsigned* fp = (GAS unsigned*)(pws + WS_XF + rank * 1024 + (pj >> 1) * 256 + (pj & 1) * 64) + pg;
      const GAS unsigned char* xf = (const GAS unsigned char*)(myws + WS_XF + src * 1024) + (lane & 7) * 8;
      int nb = 0;
      bool stored = false, polled = false;
      unsigned pbase = 0, ready = remote_x ? 0 : NT;
      auto burst = [&]() {
        const unsigned vo = xs_mine + nb * 8 * 32 * PCS;
#pragma unroll
        for (int d = 1; d < NR; ++d) {
          asm volatile("global_store_dwordx4 %0, %1, off sc0 sc1\n\ts_nop 1" ::"v"(A.ws[d] + vo), "v"(r[0]) : "memory");
          asm volatile("global_store_dwordx4 %0, %1, off offset:%c2 sc0 sc1\n\ts_nop 1" ::"v"(A.ws[d] + vo), "v"(r[1]), "i"(PCS) : "memory");
        }
        stored = true;
      };
      auto burst_due = [&]() { return NOW() >= t0 + (unsigned)(EOFF + (nb ? 8 * TAUE + (8 * nb + pj - 8) * TAU : pj * TAUE)) / 10u; };
      auto poll = [&]() {
        pbase = ready;
        asm volatile("global_load_dwordx2 v[250:251], %0, off sc0 sc1" :: "v"(xf + ((pbase + (lane >> 3)) & 6) * 128 + ((pbase + (lane >> 3)) & 1) * 64) : "memory", "v250", "v251");
        polled = true;
      };
      auto settle = [&]() {
        unsigned p0, p1;
        asm volatile("s_waitcnt vmcnt(0)\n\tv_mov_b32 %0, v250\n\tv_mov_b32 %1, v251" : "=v"(p0), "=v"(p1) :: "memory");
        if (stored) {
#pragma unroll
          for (int i = 0; i < 14; ++i) r[i] = r[i + 2];
          ++nb;
          if (flagger) __hip_atomic_store(fp, call * 16u + (unsigned)nb, RLX_SYS);
          stored = false;
        }
        if (polled) {
          const unsigned want = call * 16u + ((pbase + (lane >> 3)) >> 3) + 1u;
          const u64 miss = ~__ballot(p0 >= want && p1 >= want);
          ready = pbase + (miss ? (unsigned)__builtin_ctzll(miss) >> 3 : 8u);
          polled = false;
        }
      };
      auto gate = [&](unsigned need) {
        while (ready < need) {
          if (!stored && nb < 8 && burst_due()) burst();
          if (!polled) poll();
          settle();
          if (ready < need && (__hip_atomic_load(tmo, RLX_AGT) == call || NOW() - t0 > DEADLINE)) { bad = true; ready = NT; }
        }
      };
      auto load_rest = [&]() {
        asm volatile("global_load_dwordx4 %0, %6, off offset:1024\n\tglobal_load_dwordx4 %1, %7, off offset:1024\n\tglobal_load_dwordx4 %2, %6, off offset:2048\n\t"
                     "global_load_dwordx4 %3, %7, off offset:2048\n\tglobal_load_dwordx4 %4, %6, off offset:3072\n\tglobal_load_dwordx4 %5, %7, off offset:3072\n\ts_waitcnt vmcnt(0)"
                     : "=&v"(r[2]), "=&v"(r[3]), "=&v"(r[4]), "=&v"(r[5]), "=&v"(r[6]), "=&v"(r[7]) : "v"(s0), "v"(s1) : "memory");
        asm volatile("global_load_dwordx4 %0, %8, off\n\tglobal_load_dwordx4 %1, %9, off\n\tglobal_load_dwordx4 %2, %8, off offset:1024\n\tglobal_load_dwordx4 %3, %9, off offset:1024\n\t"
                     "global_load_dwordx4 %4, %8, off offset:2048\n\tglobal_load_dwordx4 %5, %9, off offset:2048\n\tglobal_load_dwordx4 %6, %8, off offset:3072\n\tglobal_load_dwordx4 %7, %9, off offset:3072\n\t"
                     "s_waitcnt vmcnt(0)"
                     : "=&v"(r[8]), "=&v"(r[9]), "=&v"(r[10]), "=&v"(r[11]), "=&v"(r[12]), "=&v"(r[13]), "=&v"(r[14]), "=&v"(r[15]) : "v"(s0 + 4096), "v"(s1 + 4096) : "memory");
      };
      if (pj < 3) {
        while (!burst_due()) __builtin_amdgcn_s_sleep(1);
        burst();
        WAIT_V(0);
        if (flagger) __hip_atomic_store(fp, call * 16u + 1u, RLX_SYS);
      }
      load_rest();
      settle();
      if (remote_x) { gate(2); BAR(); }
      for (const u64 due = due_at(0); NOW() < due;) {
        if (!stored && nb == 0 && burst_due()) burst();
        if (stored && !vm_busy()) settle();
        __builtin_amdgcn_s_sleep(1);
      }
      for (int t = 0; t < NT; ++t) {
        const unsigned need = t + 3 < NT ? t + 3 : NT;
        gate(need);
        const u64 due = due_at(t);
        while (NOW() < due) __builtin_amdgcn_s_sleep(1);
        BAR();
        const int c = 8 * nb + pj;
        if (nb < 8 || ready < (unsigned)NT) {
          __builtin_amdgcn_s_sleep(DSL);
          if ((stored || polled) && (!vm_busy() || (stored && t + 4 >= c))) settle();
          else if ((t & 1) == 0) {
            if (!stored && nb < 8 && (t + 7 >= c || burst_due())) burst();
            if (!polled && ready < (unsigned)NT && ready < need + LEAD) poll();
          }
        }
      }
      settle();
      while (nb < 8) { burst(); settle(); }
    }
    const unsigned par = call & 1u;
    if (bad && lane == 0) __hip_atomic_store(tmo, call, RLX_AGT);
    GAS u64* gp = (GAS u64*)(pws + WS_GLB + (par * 8 + (lane & 7)) * 256) + rank;
    const unsigned late = (unsigned)(NOW() - due_at(NT - 1));
    if (lane == 0) red[6] = par | (late > LT2 ? 4u : late > LT1 ? 2u : 0u);
    WAIT_L(0);
    BAR();
    {
      const unsigned m = dpp_umax<0x4E>(dpp_umax<0xB1>(red[lane & 3]));
      if (lane == 0) __hip_atomic_store((GAS u64*)(myws + WS_LOC + (par * 16 + (bid >> 4)) * 256) + (bid & 15), ((u64)call << 32) | m, RLX_AGT);
    }
    if (early) {
      BAR();
      if (lane == 0) __hip_atomic_store((GAS unsigned*)(A.ws[drel] + WS_YF + (rank * TPP + tile) * 256), call, RLX_SYS);
      GAS unsigned* f = lane == 1 ? tmo : (GAS unsigned*)(myws + WS_YF + (((rank - drel) & 7) * TPP + tile) * 256);
      for (;;) {
        const unsigned v = __hip_atomic_load(f, RLX_SYS);
        if ((unsigned)__builtin_amdgcn_readlane(v, 0) >= call) break;
        if ((unsigned)__builtin_amdgcn_readlane(v, 1) == call || NOW() - t0 > DEADLINE) { bad = true; break; }
        __builtin_amdgcn_s_sleep(2);
      }
      if (lane == 0) red[5] = bad;
      WAIT_L(0);
      BAR();
    }
    if (bid == 14 * 8) {
      GAS u64* g = (GAS u64*)(myws + WS_LOC + (par * 16 + (lane >> 4)) * 256) + (lane & 15);
      unsigned m;
      for (;;) {
        const u64 a = __hip_atomic_load(g, RLX_AGT), b = __hip_atomic_load(g + 128, RLX_AGT), c = __hip_atomic_load(g + 256, RLX_AGT), d = __hip_atomic_load(g + 384, RLX_AGT);
        const bool ok = (unsigned)(a >> 32) == call && (unsigned)(b >> 32) == call && (unsigned)(c >> 32) == call && (unsigned)(d >> 32) == call;
        m = max(max((unsigned)a, (unsigned)b), max((unsigned)c, (unsigned)d));
        if (__all(ok)) break;
        if (NOW() - t0 > DEADLINE) { bad = true; break; }
      }
      m = wave_umax(m);
      __hip_atomic_store(gp, ((u64)call << 32) | (bad ? 0x7fc00000u : m), RLX_SYS);
    }
    {
      GAS u64* g = lane == 8 ? (GAS u64*)tmo : (GAS u64*)(myws + WS_GLB + (par * 8 + xcd) * 256) + (lane & 7);
      unsigned gm;
      for (;;) {
        const u64 a = __hip_atomic_load(g, RLX_SYS);
        gm = lane == 8 ? 0u : (unsigned)a;
        if (__builtin_amdgcn_readlane((unsigned)a, 8) == call) { bad = true; break; }
        if (__all(lane == 8 || (unsigned)(a >> 32) >= call)) {
          bad = bad || __any(lane != 8 && (unsigned)(a >> 32) != call);
          break;
        }
        if (bad || NOW() - t0 > DEADLINE) { bad = true; break; }
        __builtin_amdgcn_s_sleep(1);
      }
      gm = row_umax(gm);
      if (lane == 0) red[4] = bad ? 0x7fc00000u : __float_as_uint(448.0f / __uint_as_float(gm));
      if (bad && lane == 0) __hip_atomic_store(tmo, call, RLX_AGT);
    }
    WAIT_L(0);
    BAR();
    if (bid == 15 * 8 && lane == 0) __hip_atomic_store((GAS unsigned*)(myws + WS_LAST), (unsigned)NOW(), RLX_AGT);
    BAR();
    if (drel != 0 && !early && lane == 0)
      __hip_atomic_store((GAS unsigned*)(A.ws[drel] + WS_DONE + (rank >> 1) * 256 + (rank & 1) * 64) + tile, call, RLX_SYS);
    if (bid == 15 * 8) {
      const bool mine = (lane >> 3) == rank;
      GAS u64* f = mine ? (GAS u64*)tmo : (GAS u64*)(myws + WS_DONE + (lane >> 4) * 256) + (lane & 15);
      const bool e0 = early_index(2 * (lane & 7)) < NF, e1 = early_index(2 * (lane & 7) + 1) < NF;
      for (;;) {
        const u64 a = __hip_atomic_load(f, RLX_SYS);
        const bool ok = mine || ((e0 || (unsigned)a >= call) && (e1 || (unsigned)(a >> 32) >= call));
        if (__all(ok)) break;
        if (__any(mine && (unsigned)a == call) || NOW() - t0 > DEADLINE) {
          if (lane == 0) { __hip_atomic_store(tmo, call, RLX_AGT); A.out[0][(size_t)(rank * MS + half * BM) * NS + tc * BN] = 0x7f; }
          break;
        }
      }
    }
    return;
  }

  const int wm = wid >> 1, wn = wid & 1;
  const int fr = lane & 15, g = lane >> 4;
  const int xa0 = X_OFF + (wm * 128 + fr) * 128 + ((g ^ ((fr >> 1) & 7)) << 4);
  const int xa1 = xa0 ^ 64;
  const int q = fr >> 2, p = fr & 3, h = q | ((g & 1) << 2);
  int wa[4];
#pragma unroll
  for (int nt = 0; nt < 4; ++nt) wa[nt] = (8 * g + q) * 256 + (((4 * wn + nt) ^ h) << 5) + p * 8;
  f32x4 acc[8][4];
#pragma unroll
  for (int a = 0; a < 8; ++a)
#pragma unroll
    for (int b = 0; b < 4; ++b) { acc[a][b] = (f32x4){0.f, 0.f, 0.f, 0.f}; asm volatile("" : "+v"(acc[a][b])); }
  s16x4 WA[4][4], WC[4][4];
  bf16x8 X[8][2];
#define MF(a, b, Wf, ks) do { if (!skip) { const bf16x8 wv = __builtin_bit_cast(bf16x8, __builtin_shufflevector(Wf[b][2 * ks], Wf[b][2 * ks + 1], 0, 1, 2, 3, 4, 5, 6, 7)); \
    asm volatile("v_mfma_f32_16x16x32_bf16 %0, %1, %2, %0" : "+v"(acc[a][b]) : "v"(wv), "v"(X[a][ks])); } SCHED(); } while (0)
#define RW(Wf, b, i) do { Wf[b][i] = __builtin_amdgcn_ds_read_tr16_b64_v4i16((LAS s16x4*)(lds + wp[b] + wimm + (i >> 1) * 8192 + (i & 1) * 1024)); SCHED(); } while (0)
#define RX(a, ks) do { X[a][ks] = *(LAS bf16x8*)(lds + (ks ? xp1 : xp0) + a * 2048); SCHED(); } while (0)
#define GRP(a, r0, r1, r2, r3) MF(a, 0, Wc, 0); r0; MF(a, 0, Wc, 1); MF(a, 1, Wc, 0); r1; MF(a, 1, Wc, 1); MF(a, 2, Wc, 0); r2; MF(a, 2, Wc, 1); MF(a, 3, Wc, 0); r3; MF(a, 3, Wc, 1);
  auto step = [&](s16x4 (&Wp)[4][4], s16x4 (&Wc)[4][4], int xoff, const int (&wp)[4], int wimm, bool first) {
    bool skip = first;
    const int xp0 = xa0 + xoff, xp1 = xa1 + xoff;
    MF(5, 0, Wp, 0); RW(Wc, 0, 0); MF(5, 0, Wp, 1); RW(Wc, 0, 1); MF(5, 1, Wp, 0); RW(Wc, 0, 2); MF(5, 1, Wp, 1); RW(Wc, 0, 3);
    MF(5, 2, Wp, 0); RW(Wc, 1, 0); MF(5, 2, Wp, 1); RW(Wc, 1, 1); MF(5, 3, Wp, 0); RW(Wc, 1, 2); MF(5, 3, Wp, 1); RW(Wc, 1, 3);
    MF(6, 0, Wp, 0); RW(Wc, 2, 0); MF(6, 0, Wp, 1); RW(Wc, 2, 1); MF(6, 1, Wp, 0); RW(Wc, 2, 2); MF(6, 1, Wp, 1); RW(Wc, 2, 3);
    MF(6, 2, Wp, 0); RW(Wc, 3, 0); MF(6, 2, Wp, 1); RW(Wc, 3, 1); MF(6, 3, Wp, 0); RW(Wc, 3, 2); MF(6, 3, Wp, 1); RW(Wc, 3, 3);
    MF(7, 0, Wp, 0); RX(0, 0); MF(7, 0, Wp, 1); RX(0, 1); MF(7, 1, Wp, 0); MF(7, 1, Wp, 1); RX(1, 0);
    MF(7, 2, Wp, 0); MF(7, 2, Wp, 1); RX(1, 1); MF(7, 3, Wp, 0); MF(7, 3, Wp, 1);
    skip = false;
    GRP(0, RX(2, 0), RX(2, 1), , )
    GRP(1, RX(3, 0), RX(3, 1), , )
    GRP(2, RX(4, 0), RX(4, 1), RX(5, 0), RX(5, 1))
    GRP(3, RX(6, 0), RX(6, 1), RX(7, 0), RX(7, 1))
    GRP(4, , , , )
  };
  if (remote_x) BAR();
  BAR();
  SCHED();
  step(WC, WA, 0, wa, 0, true);
  int xoff = 0;
  for (int t = 1; t < NT - 1; t += 2) {
    xoff = xoff == (NXS - 1) * XB ? 0 : xoff + XB;
    WAIT_L(0); BAR(); SCHED();
    step(WA, WC, xoff, wa, WB, false);
    xoff = xoff == (NXS - 1) * XB ? 0 : xoff + XB;
#pragma unroll
    for (int b = 0; b < 4; ++b) wa[b] ^= 2 * WB;
    WAIT_L(0); BAR(); SCHED();
    step(WC, WA, xoff, wa, 0, false);
  }
  xoff = xoff == (NXS - 1) * XB ? 0 : xoff + XB;
  WAIT_L(0); BAR(); SCHED();
  step(WA, WC, xoff, wa, WB, false);
  float mx = 0.f;
#define MX(a, b) do { asm volatile("v_max3_f32 %0, |%1|, |%2|, %0\n\tv_max3_f32 %0, |%3|, |%4|, %0" : "+v"(mx) : "v"(acc[a][b][0]), "v"(acc[a][b][1]), "v"(acc[a][b][2]), "v"(acc[a][b][3])); SCHED(); } while (0)
#define MX2(a, b) MX(a, b); MX(a + 1, b)
  {
    const bool skip = false;
    auto& Wc = WC;
    GRP(5, MX2(0, 0), MX2(0, 1), MX2(0, 2), MX2(0, 3))
    GRP(6, MX2(2, 0), MX2(2, 1), MX2(2, 2), MX2(2, 3))
    GRP(7, MX2(4, 0), MX2(4, 1), MX2(4, 2), MX2(4, 3))
    MX(6, 0); MX(6, 1); MX(6, 2); MX(6, 3);
    asm volatile("s_nop 7" ::: "memory");
    MX(7, 0); MX(7, 1); MX(7, 2); MX(7, 3);
    asm volatile("s_nop 1" : "+v"(mx));
  }
  {
    const unsigned m = wave_umax(__float_as_uint(mx));
    if (lane == 0) red[wid] = m;
  }
  WAIT_L(0);
  BAR();
  const unsigned lo = (unsigned)((wm * 128 + fr) * NS + wn * 64 + g * 16);
  unsigned char* op = A.out[drel] + (size_t)(src * MS + half * BM) * NS + tc * BN + lo;
  if (early) {
    const unsigned lvl = __builtin_amdgcn_readfirstlane(red[6]) >> 1;
    const size_t slot = WS_YS + (size_t)((red[6] & 1u) * (NR * TPP) + tile) * YS_TILE + wid * YS_WAVE + lane * 16;
    unsigned char* sp = A.ws[drel] + slot + (size_t)rank * TPP * YS_TILE;
#define PUT(off, val) do { const u32x4 o_ = val; asm volatile("global_store_dwordx4 %0, %1, off offset:%c2 sc0 sc1\n\ts_nop 1" ::"v"(sp + c * 4096), "v"(o_), "i"(off) : "memory"); if (lvl == 0) WAIT_V(FTH); else if (lvl == 1) WAIT_V(FTH1); else WAIT_V(FTH2); } while (0)
    unsigned head = (__builtin_bit_cast(u16, (__bf16)mx) >> 8) & 0x7fu;
    const unsigned brep = head * 0x01010101u | 0x80808080u;
#pragma unroll
    for (int c = 0; c < 4; ++c) {
      unsigned L[8], H[8], N[4], ovf = 0;
#pragma unroll
      for (int j = 0; j < 8; ++j) {
        const f32x4 v = acc[2 * c + (j >> 2)][j & 3];
        typedef __bf16 bf16x2 __attribute__((ext_vector_type(2)));
        const unsigned p0 = __builtin_bit_cast(unsigned, (bf16x2){(__bf16)v[0], (__bf16)v[1]}), p1 = __builtin_bit_cast(unsigned, (bf16x2){(__bf16)v[2], (__bf16)v[3]});
        L[j] = __builtin_amdgcn_perm(p1, p0, 0x06040200u);
        H[j] = __builtin_amdgcn_perm(p1, p0, 0x07050301u);
      }
      PUT(0, ((u32x4){L[0], L[1], L[2], L[3]}));
      PUT(1024, ((u32x4){L[4], L[5], L[6], L[7]}));
#pragma unroll
      for (int k = 0; k < 4; ++k) {
        const unsigned d0 = brep - (H[2 * k] & 0x7f7f7f7fu), d1 = brep - (H[2 * k + 1] & 0x7f7f7f7fu);
        ovf |= ((d0 ^ 0x80808080u) | (d1 ^ 0x80808080u)) & 0xf8f8f8f8u;
        N[k] = (d0 & 0x07070707u) | ((H[2 * k] & 0x80808080u) >> 4) | ((d1 & 0x07070707u) << 4) | (H[2 * k + 1] & 0x80808080u);
      }
      if (__any(ovf != 0)) {
        PUT(2048, ((u32x4){H[0], H[1], H[2], H[3]}));
        PUT(3072, ((u32x4){H[4], H[5], H[6], H[7]}));
        head |= 256u << c;
      } else PUT(2048, ((u32x4){N[0], N[1], N[2], N[3]}));
    }
    asm volatile("global_store_dword %0, %1, off sc0 sc1" ::"v"(sp + 16384 - lane * 12), "v"(head) : "memory");
    WAIT_V(0);
    BAR();
    BAR();
    const int from = (rank - drel) & 7;
    op = A.out[0] + (size_t)(from * MS + half * BM) * NS + tc * BN + lo;
    if (__builtin_amdgcn_readfirstlane(red[5]) == 0) {
      const unsigned char* lp = myws + slot + (size_t)from * TPP * YS_TILE;
      u32x4 r[12];
      asm volatile("global_load_dword %0, %13, off sc0 sc1\n\t"
                   "global_load_dwordx4 %1, %14, off offset:-4096 sc0 sc1\n\tglobal_load_dwordx4 %2, %14, off offset:-3072 sc0 sc1\n\tglobal_load_dwordx4 %3, %14, off offset:-2048 sc0 sc1\n\t"
                   "global_load_dwordx4 %4, %14, off sc0 sc1\n\tglobal_load_dwordx4 %5, %14, off offset:1024 sc0 sc1\n\tglobal_load_dwordx4 %6, %14, off offset:2048 sc0 sc1\n\t"
                   "global_load_dwordx4 %7, %15, off offset:-4096 sc0 sc1\n\tglobal_load_dwordx4 %8, %15, off offset:-3072 sc0 sc1\n\tglobal_load_dwordx4 %9, %15, off offset:-2048 sc0 sc1\n\t"
                   "global_load_dwordx4 %10, %15, off sc0 sc1\n\tglobal_load_dwordx4 %11, %15, off offset:1024 sc0 sc1\n\tglobal_load_dwordx4 %12, %15, off offset:2048 sc0 sc1\n\ts_waitcnt vmcnt(0)"
                   : "=&v"(head), "=&v"(r[0]), "=&v"(r[1]), "=&v"(r[2]), "=&v"(r[3]), "=&v"(r[4]), "=&v"(r[5]), "=&v"(r[6]), "=&v"(r[7]), "=&v"(r[8]), "=&v"(r[9]), "=&v"(r[10]), "=&v"(r[11])
                   : "v"(lp + 16384 - lane * 12), "v"(lp + 4096), "v"(lp + 12288) : "memory");
      const unsigned raw = __builtin_amdgcn_readfirstlane(head) >> 8, base = (head & 0x7fu) * 0x01010101u;
#pragma unroll
      for (int c = 0; c < 4; ++c) {
        unsigned H[8];
        const u32x4 n4 = r[3 * c + 2];
        if ((raw >> c) & 1u) {
          u32x4 h2;
          asm volatile("global_load_dwordx4 %0, %1, off offset:3072 sc0 sc1\n\ts_waitcnt vmcnt(0)" : "=&v"(h2) : "v"(lp + c * 4096) : "memory");
#pragma unroll
          for (int k = 0; k < 4; ++k) { H[k] = n4[k]; H[4 + k] = h2[k]; }
        } else {
#pragma unroll
          for (int k = 0; k < 4; ++k) {
            H[2 * k] = (base - (n4[k] & 0x07070707u)) | ((n4[k] & 0x08080808u) << 4);
            H[2 * k + 1] = (base - ((n4[k] >> 4) & 0x07070707u)) | (n4[k] & 0x80808080u);
          }
        }
#pragma unroll
        for (int j = 0; j < 8; ++j) {
          const unsigned l = r[3 * c + (j >> 2)][j & 3];
          acc[2 * c + (j >> 2)][j & 3] = (f32x4){__uint_as_float(__builtin_amdgcn_perm(H[j], l, 0x04000c0cu)), __uint_as_float(__builtin_amdgcn_perm(H[j], l, 0x05010c0cu)),
                                                 __uint_as_float(__builtin_amdgcn_perm(H[j], l, 0x06020c0cu)), __uint_as_float(__builtin_amdgcn_perm(H[j], l, 0x07030c0cu))};
        }
      }
    }
  }
  BAR();
  const float inv = __uint_as_float(red[4]);
  const bool throttle = drel != 0 && !early;
#pragma unroll
  for (int a = 0; a < 8; ++a) {
    unsigned d[4];
#pragma unroll
    for (int b = 0; b < 4; ++b) {
      const f32x4 v = acc[a][b] * inv;
      int pk = __builtin_amdgcn_cvt_pk_fp8_f32(v[0], v[1], 0, false);
      pk = __builtin_amdgcn_cvt_pk_fp8_f32(v[2], v[3], pk, true);
      d[b] = pk;
    }
    { auto r = __builtin_amdgcn_permlane32_swap(d[0], d[2], false, false); d[0] = r[0]; d[2] = r[1]; }
    { auto r = __builtin_amdgcn_permlane32_swap(d[1], d[3], false, false); d[1] = r[0]; d[3] = r[1]; }
    { auto r = __builtin_amdgcn_permlane16_swap(d[0], d[1], false, false); d[0] = r[0]; d[1] = r[1]; }
    { auto r = __builtin_amdgcn_permlane16_swap(d[2], d[3], false, false); d[2] = r[0]; d[3] = r[1]; }
    const u32x4 o = {d[0], d[1], d[2], d[3]};
    asm volatile("global_store_dwordx4 %0, %1, off sc0 sc1\n\ts_nop 1" ::"v"(op + (size_t)a * 16 * NS), "v"(o) : "memory");
    if (throttle) WAIT_V(QTH);
  }
  WAIT_V(0);
  BAR();
}

extern "C" void kernel_launch(int rank, int nranks, ncclComm_t comm, hipStream_t stream,
                              void* const* d_in, const int* in_sizes, int n_in,
                              void* d_out, int out_size, const int* in_dtypes, int out_dtype,
                              void* const* d_out_all, void* const* d_ws_all, size_t ws_bytes) {
  (void)comm; (void)in_sizes; (void)n_in; (void)d_out; (void)out_size; (void)in_dtypes; (void)out_dtype;
  if (nranks != NR || (unsigned)rank >= (unsigned)NR || ws_bytes < WS_END) { fprintf(stderr, "unexpected configuration\n"); return; }
  Args a;
  a.x = (const u16*)d_in[0]; a.w = (const u16*)d_in[1];
  for (int i = 0; i < NR; ++i) { a.out[i] = (unsigned char*)d_out_all[(rank + i) & 7]; a.ws[i] = (unsigned char*)d_ws_all[(rank + i) & 7]; }
  a.rank = rank; a.pad = 0;
  hipLaunchKernelGGL(fused_gemm_a2a, dim3(NWG), dim3(512), 0, stream, a);
  HIP_CHECK(hipGetLastError());
}
```

```cpp
#include <hip/hip_runtime.h>
#include <rccl/rccl.h>
#include <cstdio>

#define HIP_CHECK(expr) do { hipError_t _e = (expr); if (_e != hipSuccess) { \
    fprintf(stderr, "HIP error %s at %s:%d\n", hipGetErrorString(_e), __FILE__, __LINE__); \
    return; } } while (0)

#define LAS __attribute__((address_space(3)))
#define GAS __attribute__((address_space(1)))
typedef __bf16 bf16x8 __attribute__((ext_vector_type(8)));
typedef short s16x4 __attribute__((ext_vector_type(4)));
typedef float f32x4 __attribute__((ext_vector_type(4)));
typedef unsigned u32x4 __attribute__((ext_vector_type(4)));
typedef unsigned short u16;
typedef unsigned long long u64;

namespace {
constexpr int NR = 8;
constexpr int MG = 4096, NG = 16384, KG = 4096;
constexpr int MS = MG / NR;
constexpr int NS = NG / NR;
constexpr int BM = 256, BN = 128, BK = 64, NT = KG / BK;
constexpr int XB = BM * BK * 2, WB = BK * BN * 2;
constexpr int NXS = 3, NWS = 4;
constexpr int X_OFF = NWS * WB;
constexpr int LDS_BYTES = NXS * XB + NWS * WB;
constexpr int NWG = 256;
constexpr int TPP = 16;
constexpr u64 DEADLINE = 150000000ull;
constexpr int NF = 12;
constexpr int POFF = 1000, POFFB = 6800, SBL = 2500;
constexpr int TF0 = 47800, TFD = 950;
constexpr int TEND = 58900;
constexpr int SP = 42;
constexpr int SOFTF = 260, SOFTS = 1500, SOFT = 2900;
constexpr int EOFF = 600, NE = 11, TAUE = 620, TAU = 728, STG = 45;
constexpr int LEAD = 6;
constexpr int WSL = 5, WSLS = 7, WSLF = 4, DSL = 9;
constexpr int FTH = 12, FTH1 = 3, FTH2 = 1, QTH = 3;
constexpr unsigned LT1 = 40, LT2 = 120;

constexpr size_t WS_TMO = 0;
constexpr size_t WS_LAST = 256;
constexpr size_t WS_DONE = 1024;
constexpr size_t WS_GLB = 4096;
constexpr size_t WS_LOC = 8192;
constexpr size_t WS_XF = 16384;
constexpr size_t WS_YF = 32768;
constexpr size_t WS_CNT = 65536;
constexpr size_t WS_XS = 2u << 20;
constexpr int PCS = 2048;
constexpr size_t XS_SLOT = (size_t)NT * 32 * PCS;
constexpr size_t WS_YS = WS_XS + NR * XS_SLOT;
constexpr int YS_WAVE = 17 * 1024;
constexpr size_t YS_TILE = 4 * YS_WAVE;
constexpr size_t WS_END = WS_YS + (size_t)2 * NR * TPP * YS_TILE;
static_assert(NF <= TPP && WS_YF + NR * TPP * 256 <= WS_CNT && WS_CNT + NWG * 256 <= (1u << 20), "layout");

struct Args {
  unsigned char* ws[NR];
  unsigned char* out[NR];
  const u16* x; const u16* w;
  int rank; int pad;
};
}

#define BAR() do { asm volatile("" ::: "memory"); __builtin_amdgcn_s_barrier(); asm volatile("" ::: "memory"); } while (0)
#define WAIT_V(n) asm volatile("s_waitcnt vmcnt(%0)" ::"n"(n) : "memory")
#define WAIT_L(n) asm volatile("s_waitcnt lgkmcnt(%0)" ::"n"(n) : "memory")
#define SCHED() __builtin_amdgcn_sched_barrier(0)
#define RLX_SYS __ATOMIC_RELAXED, __HIP_MEMORY_SCOPE_SYSTEM
#define RLX_AGT __ATOMIC_RELAXED, __HIP_MEMORY_SCOPE_AGENT
#define NOW() __builtin_amdgcn_s_memrealtime()

template <int CTRL> __device__ __forceinline__ unsigned dpp_umax(unsigned v) {
  const unsigned t = __builtin_amdgcn_update_dpp(v, v, CTRL, 0xf, 0xf, false);
  return t > v ? t : v;
}
__device__ __forceinline__ unsigned row_umax(unsigned v) {
  v = dpp_umax<0xB1>(v); v = dpp_umax<0x4E>(v); v = dpp_umax<0x141>(v); return dpp_umax<0x140>(v);
}
__device__ __forceinline__ unsigned wave_umax(unsigned v) {
  v = row_umax(v);
  const unsigned a = __builtin_amdgcn_readlane(v, 0), b = __builtin_amdgcn_readlane(v, 16), c = __builtin_amdgcn_readlane(v, 32), d = __builtin_amdgcn_readlane(v, 48);
  return max(max(a, b), max(c, d));
}

__device__ __forceinline__ unsigned vm_busy() { return __builtin_amdgcn_s_getreg(7 | (31 << 11)) & 0x00C0000Fu; }

__global__ __launch_bounds__(512, 2) void fused_gemm_a2a(Args A) {
  __shared__ __attribute__((aligned(16))) char smem[LDS_BYTES];
  LAS char* lds = (LAS char*)smem;
  const int tid = threadIdx.x, wid = __builtin_amdgcn_readfirstlane(tid >> 6), lane = tid & 63;
  const int bid = blockIdx.x, xcd = bid & 7, idx = (bid >> 3) & 31;
  const int rank = A.rank;
  int src = rank, half = 0, drel = 0, tc;
  if (idx < 14) { const int a = xcd * 14 + idx; drel = 1 + (a >> 4); tc = a & 15; }
  else if (idx < 16) tc = xcd * 2 + (idx - 14);
  else { const int k = idx - 16; src = (rank + (xcd & 1) * 4 + (k >> 2)) & 7; half = 1; tc = (xcd >> 1) * 4 + (k & 3); }
  const bool remote_x = src != rank;
  const int n0 = ((rank + drel) & 7) * NS + tc * BN;
  const int koff = drel ? xcd * 8 : 0;
  auto early_index = [](int c) { return (c & 3) == 1 ? 12 + (c >> 2) : c - ((c + 2) >> 2); };
  const int fidx = early_index(tc);
  const bool early = drel != 0 && fidx < NF;
  const int tile = tc;
  unsigned char* myws = A.ws[0];
  LAS unsigned* red = (LAS unsigned*)lds;

  if (wid >= 4) {
    if (wid < 6) {
      const int j = wid - 4;
      LAS char* base = lds + X_OFF + j * 16384;
      if (!remote_x) {
        auto rs = __builtin_amdgcn_make_buffer_rsrc((void*)(A.x + (size_t)(half * BM + j * 128) * KG), 0, 128 * KG * 2, 0x00020000);
        const int v0 = (lane >> 3) * (KG * 2) + (((lane & 7) ^ (lane >> 4)) << 4);
        const int v1 = (lane >> 3) * (KG * 2) + (((lane & 7) ^ (4 | (lane >> 4))) << 4);
        auto issue = [&](int t, int slot) {
#pragma unroll
          for (int i = 0; i < 16; ++i)
            __builtin_amdgcn_raw_ptr_buffer_load_lds(rs, (LAS void*)(base + slot * XB + i * 1024), 16, (i & 1) ? v1 : v0, i * 8 * (KG * 2) + ((t + koff) & 63) * 128, 0, 0);
        };
        issue(0, 0); issue(1, 1);
        int slot = 2;
        WAIT_V(16); BAR();
        for (int t = 0; t < NT - 1; ++t) {
          if (t + 2 < NT) { issue(t + 2, slot); WAIT_V(16); } else WAIT_V(0);
          slot = slot == NXS - 1 ? 0 : slot + 1;
          BAR();
        }
      } else {
        const unsigned char* sb = myws + WS_XS + (size_t)src * XS_SLOT + j * 16 * PCS;
        const unsigned vo = lane * 16 + (lane < 56 ? 512 : 640);
        auto issue = [&](int t, int slot) {
#pragma unroll
          for (int i = 0; i < 16; ++i)
            __builtin_amdgcn_global_load_lds((const GAS void*)(sb + (size_t)((t * 32 + i) * PCS) + vo), (LAS void*)(base + slot * XB + i * 1024), 16, 0, 17);
        };
        BAR();
        issue(0, 0); issue(1, 1);
        int slot = 2;
        WAIT_V(16); BAR();
        for (int t = 0; t < NT - 1; ++t) {
          if (t + 2 < NT) { issue(t + 2, slot); WAIT_V(16); } else WAIT_V(0);
          slot = slot == NXS - 1 ? 0 : slot + 1;
          BAR();
        }
      }
      return;
    }
    if (wid == 6) {
      auto rs = __builtin_amdgcn_make_buffer_rsrc((void*)(A.w + n0), 0, (unsigned)((size_t)KG * NG * 2 - (size_t)n0 * 2), 0x00020000);
      const int v0 = (lane >> 4) * (NG * 2) + (((lane & 15) ^ ((lane >> 4) << 1)) << 4);
      const int v1 = (lane >> 4) * (NG * 2) + (((lane & 15) ^ ((4 | (lane >> 4)) << 1)) << 4);
      auto issue = [&](int t, int slot, bool stream) {
        if (stream) {
#pragma unroll
          for (int i = 0; i < 16; ++i)
            __builtin_amdgcn_raw_ptr_buffer_load_lds(rs, (LAS void*)(lds + slot * WB + i * 1024), 16, (i & 2) ? v1 : v0, (((t + koff) & 63) * 64 + i * 4) * (NG * 2), 0, 2);
        } else {
#pragma unroll
          for (int i = 0; i < 16; ++i)
            __builtin_amdgcn_raw_ptr_buffer_load_lds(rs, (LAS void*)(lds + slot * WB + i * 1024), 16, (i & 2) ? v1 : v0, (((t + koff) & 63) * 64 + i * 4) * (NG * 2), 0, 0);
        }
      };
      if (remote_x) BAR();
      issue(0, 0, false); issue(1, 1, false);
      unsigned last;
      asm volatile("global_load_dword %0, %1, off sc1" : "=v"(last) : "v"(myws + WS_LAST) : "memory");
      WAIT_V(16); BAR();
      __builtin_amdgcn_s_sleep(WSL);
      issue(2, 2, false); issue(3, 3, false);
      asm volatile("s_waitcnt vmcnt(32)" : "+v"(last) :: "memory");
      BAR();
      const bool stream = drel != 0 && (unsigned)NOW() - (unsigned)__builtin_amdgcn_readfirstlane(last) > 30000u;
      WAIT_V(16); BAR();
      auto run = [&](bool st) {
        for (int t = 2; t < NT - 2; t += 2) {
          if (fidx >= 11) __builtin_amdgcn_s_sleep(WSLS); else if (early) __builtin_amdgcn_s_sleep(WSLF); else __builtin_amdgcn_s_sleep(WSL);
          issue(t + 2, (t + 2) & 3, st); issue(t + 3, (t + 3) & 3, st);
          WAIT_V(32); BAR();
          WAIT_V(16); BAR();
        }
      };
      if (stream) run(true); else run(false);
      WAIT_V(0); BAR();
      return;
    }
    const u64 t0 = NOW();
    GAS unsigned* cnt = (GAS unsigned*)(myws + WS_CNT + bid * 256);
    const unsigned one = 1;
    u64 ex;
    GAS unsigned* tmo = (GAS unsigned*)(myws + WS_TMO);
    const GAS void* tab = (const GAS char*)(const GAS void*)__builtin_amdgcn_kernarg_segment_ptr() + (lane >> 3) * 8;
    const int first = drel ? POFF : half ? POFFB : POFFB - SBL, last = early ? TF0 + fidx * TFD : TEND;
    const int pace = (last - first) / (NT - 1);
    const int soft = drel != 0 ? (fidx >= 11 ? SOFTS : fidx * SOFTF) : SOFT;
    auto due_at = [&](int t) { return t0 + (unsigned)(first + t * pace + (t < SP ? t * soft / SP : (63 - t) * soft / (63 - SP))) / 10u; };
    unsigned call;
    unsigned char* pws;
    bool bad = false;
    if (!half) {
      {
        const char* tp = (const char*)A.x + (size_t)(idx * 16 + (lane & 7)) * (KG * 2) + (xcd * 8 + (lane >> 3)) * 128;
        asm volatile("global_load_dword v253, %0, off\n\tglobal_load_dword v253, %1, off" ::"v"(tp), "v"(tp + 8 * KG * 2) : "memory", "v253");
      }
      for (int t = 0; t < NT; ++t) {
        const u64 due = t ? due_at(t) : 0;
        while (NOW() < due) __builtin_amdgcn_s_sleep(1);
        BAR();
        if (t == 0) {
          __builtin_amdgcn_s_sleep(8);
          asm volatile("global_load_dwordx2 v[250:251], %1, off\n\ts_mov_b64 %0, exec\n\ts_mov_b64 exec, 1\n\tglobal_atomic_add v252, %2, %3, off sc0\n\ts_mov_b64 exec, %0"
                       : "=&s"(ex) : "v"(tab), "v"(cnt), "v"(one) : "memory", "v250", "v251", "v252");
        }
      }
      unsigned plo, phi, c0;
      asm volatile("s_waitcnt vmcnt(0)\n\tv_mov_b32 %0, v250\n\tv_mov_b32 %1, v251\n\tv_mov_b32 %2, v252\n\ts_nop 0" : "=v"(plo), "=v"(phi), "=v"(c0) :: "memory");
      call = __builtin_amdgcn_readfirstlane(c0) + 1u;
      pws = (unsigned char*)(((u64)phi << 32) | plo);
    } else {
      const int l = ((idx - 4) & 15) * 8 + xcd, pg = l & 15, pj = l >> 4;
      const char* s0 = (const char*)A.x + (size_t)(BM + pg * 16 + (lane >> 3)) * (KG * 2) + (((lane & 7) ^ (lane >> 4)) << 4) + pj * 128;
      const char* s1 = (const char*)A.x + (size_t)(BM + pg * 16 + 8 + (lane >> 3)) * (KG * 2) + (((lane & 7) ^ (4 | (lane >> 4))) << 4) + pj * 128;
      u32x4 r[16];
      unsigned c0;
      asm volatile("global_load_dwordx4 %0, %5, off\n\tglobal_load_dwordx4 %1, %6, off\n\tglobal_load_dwordx2 %3, %8, off\n\t"
                   "s_mov_b64 %4, exec\n\ts_mov_b64 exec, 1\n\tglobal_atomic_add %2, %7, %9, off sc0\n\ts_mov_b64 exec, %4\n\ts_waitcnt vmcnt(0)"
                   : "=&v"(r[0]), "=&v"(r[1]), "=&v"(c0), "=&v"(pws), "=&s"(ex) : "v"(s0), "v"(s1), "v"(cnt), "v"(tab), "v"(one) : "memory");
      call = __builtin_amdgcn_readfirstlane(c0) + 1u;
      const bool flagger = lane >= 8 && (lane & 7) == 0;
      const unsigned xs_mine = (unsigned)(WS_XS + (size_t)rank * XS_SLOT) + lane * 16 + (lane < 56 ? 512 : 640) + (pj * 32 + pg * 2) * PCS;
      GAS unsigned* fp = (GAS unsigned*)(pws + WS_XF + rank * 1024 + (pj >> 1) * 256 + (pj & 1) * 64) + pg;
      const GAS unsigned char* xf = (const GAS unsigned char*)(myws + WS_XF + src * 1024) + (lane & 7) * 8;
      int nb = 0;
      bool stored = false, polled = false;
      unsigned pbase = 0, ready = remote_x ? 0 : NT;
      auto burst = [&]() {
        const unsigned vo = xs_mine + nb * 8 * 32 * PCS;
#pragma unroll
        for (int d = 1; d < NR; ++d) {
          asm volatile("global_store_dwordx4 %0, %1, off sc0 sc1\n\ts_nop 1" ::"v"(A.ws[d] + vo), "v"(r[0]) : "memory");
          asm volatile("global_store_dwordx4 %0, %1, off offset:%c2 sc0 sc1\n\ts_nop 1" ::"v"(A.ws[d] + vo), "v"(r[1]), "i"(PCS) : "memory");
        }
        stored = true;
      };
      auto burst_due = [&]() { const int c = 8 * nb + pj; return NOW() >= t0 + (unsigned)(EOFF + (c > NE ? NE * TAUE + (c - NE) * TAU + pg * STG : c * TAUE + (c > 2 ? pg * STG : 0))) / 10u; };
      auto poll = [&]() {
        pbase = ready;
        asm volatile("global_load_dwordx2 v[250:251], %0, off sc0 sc1" :: "v"(xf + ((pbase + (lane >> 3)) & 6) * 128 + ((pbase + (lane >> 3)) & 1) * 64) : "memory", "v250", "v251");
        polled = true;
      };
      auto settle = [&]() {
        unsigned p0, p1;
        asm volatile("s_waitcnt vmcnt(0)\n\tv_mov_b32 %0, v250\n\tv_mov_b32 %1, v251" : "=v"(p0), "=v"(p1) :: "memory");
        if (stored) {
#pragma unroll
          for (int i = 0; i < 14; ++i) r[i] = r[i + 2];
          ++nb;
          if (flagger) __hip_atomic_store(fp, call * 16u + (unsigned)nb, RLX_SYS);
          stored = false;
        }
        if (polled) {
          const unsigned want = call * 16u + ((pbase + (lane >> 3)) >> 3) + 1u;
          const u64 miss = ~__ballot(p0 >= want && p1 >= want);
          ready = pbase + (miss ? (unsigned)__builtin_ctzll(miss) >> 3 : 8u);
          polled = false;
        }
      };
      auto gate = [&](unsigned need) {
        while (ready < need) {
          if (!stored && nb < 8 && burst_due()) burst();
          if (!polled) poll();
          settle();
          if (ready < need && (__hip_atomic_load(tmo, RLX_AGT) == call || NOW() - t0 > DEADLINE)) { bad = true; ready = NT; }
        }
      };
      auto load_rest = [&]() {
        asm volatile("global_load_dwordx4 %0, %6, off offset:1024\n\tglobal_load_dwordx4 %1, %7, off offset:1024\n\tglobal_load_dwordx4 %2, %6, off offset:2048\n\t"
                     "global_load_dwordx4 %3, %7, off offset:2048\n\tglobal_load_dwordx4 %4, %6, off offset:3072\n\tglobal_load_dwordx4 %5, %7, off offset:3072\n\ts_waitcnt vmcnt(0)"
                     : "=&v"(r[2]), "=&v"(r[3]), "=&v"(r[4]), "=&v"(r[5]), "=&v"(r[6]), "=&v"(r[7]) : "v"(s0), "v"(s1) : "memory");
        asm volatile("global_load_dwordx4 %0, %8, off\n\tglobal_load_dwordx4 %1, %9, off\n\tglobal_load_dwordx4 %2, %8, off offset:1024\n\tglobal_load_dwordx4 %3, %9, off offset:1024\n\t"
                     "global_load_dwordx4 %4, %8, off offset:2048\n\tglobal_load_dwordx4 %5, %9, off offset:2048\n\tglobal_load_dwordx4 %6, %8, off offset:3072\n\tglobal_load_dwordx4 %7, %9, off offset:3072\n\t"
                     "s_waitcnt vmcnt(0)"
                     : "=&v"(r[8]), "=&v"(r[9]), "=&v"(r[10]), "=&v"(r[11]), "=&v"(r[12]), "=&v"(r[13]), "=&v"(r[14]), "=&v"(r[15]) : "v"(s0 + 4096), "v"(s1 + 4096) : "memory");
      };
      if (pj < 3) {
        while (!burst_due()) __builtin_amdgcn_s_sleep(1);
        burst();
        WAIT_V(0);
        if (flagger) __hip_atomic_store(fp, call * 16u + 1u, RLX_SYS);
      }
      load_rest();
      settle();
      if (remote_x) { gate(2); BAR(); }
      for (int t = 0; t < NT; ++t) {
        const unsigned need = t + 3 < NT ? t + 3 : NT;
        gate(need);
        const u64 due = due_at(t);
        while (NOW() < due) {
          if (!stored && nb < 8 && burst_due()) burst();
          if (stored && !polled && !vm_busy()) settle();
          __builtin_amdgcn_s_sleep(1);
        }
        BAR();
        const int c = 8 * nb + pj;
        if (nb < 8 || ready < (unsigned)NT) {
          __builtin_amdgcn_s_sleep(DSL);
          if ((stored || polled) && (!vm_busy() || (stored && t + 4 >= c))) settle();
          else if ((t & 1) == 0) {
            if (!stored && nb < 8 && (t + 7 >= c || burst_due())) burst();
            if (!polled && ready < (unsigned)NT && ready < need + LEAD) poll();
          }
        }
      }
      settle();
      while (nb < 8) { burst(); settle(); }
    }
    const unsigned par = call & 1u;
    if (bad && lane == 0) __hip_atomic_store(tmo, call, RLX_AGT);
    GAS u64* gp = (GAS u64*)(pws + WS_GLB + (par * 8 + (lane & 7)) * 256) + rank;
    const unsigned late = (unsigned)(NOW() - due_at(NT - 1));
    if (lane == 0) red[6] = par | (late > LT2 ? 4u : late > LT1 ? 2u : 0u);
    WAIT_L(0);
    BAR();
    {
      const unsigned m = dpp_umax<0x4E>(dpp_umax<0xB1>(red[lane & 3]));
      if (lane == 0) __hip_atomic_store((GAS u64*)(myws + WS_LOC + (par * 16 + (bid >> 4)) * 256) + (bid & 15), ((u64)call << 32) | m, RLX_AGT);
    }
    if (early) {
      BAR();
      if (lane == 0) __hip_atomic_store((GAS unsigned*)(A.ws[drel] + WS_YF + (rank * TPP + tile) * 256), call, RLX_SYS);
      GAS unsigned* f = lane == 1 ? tmo : (GAS unsigned*)(myws + WS_YF + (((rank - drel) & 7) * TPP + tile) * 256);
      for (;;) {
        const unsigned v = __hip_atomic_load(f, RLX_SYS);
        if ((unsigned)__builtin_amdgcn_readlane(v, 0) >= call) break;
        if ((unsigned)__builtin_amdgcn_readlane(v, 1) == call || NOW() - t0 > DEADLINE) { bad = true; break; }
        __builtin_amdgcn_s_sleep(2);
      }
      if (lane == 0) red[5] = bad;
      WAIT_L(0);
      BAR();
    }
    if (bid == 14 * 8) {
      GAS u64* g = (GAS u64*)(myws + WS_LOC + (par * 16 + (lane >> 4)) * 256) + (lane & 15);
      unsigned m;
      for (;;) {
        const u64 a = __hip_atomic_load(g, RLX_AGT), b = __hip_atomic_load(g + 128, RLX_AGT), c = __hip_atomic_load(g + 256, RLX_AGT), d = __hip_atomic_load(g + 384, RLX_AGT);
        const bool ok = (unsigned)(a >> 32) == call && (unsigned)(b >> 32) == call && (unsigned)(c >> 32) == call && (unsigned)(d >> 32) == call;
        m = max(max((unsigned)a, (unsigned)b), max((unsigned)c, (unsigned)d));
        if (__all(ok)) break;
        if (NOW() - t0 > DEADLINE) { bad = true; break; }
      }
      m = wave_umax(m);
      __hip_atomic_store(gp, ((u64)call << 32) | (bad ? 0x7fc00000u : m), RLX_SYS);
    }
    {
      GAS u64* g = lane == 8 ? (GAS u64*)tmo : (GAS u64*)(myws + WS_GLB + (par * 8 + xcd) * 256) + (lane & 7);
      unsigned gm;
      for (;;) {
        const u64 a = __hip_atomic_load(g, RLX_SYS);
        gm = lane == 8 ? 0u : (unsigned)a;
        if (__builtin_amdgcn_readlane((unsigned)a, 8) == call) { bad = true; break; }
        if (__all(lane == 8 || (unsigned)(a >> 32) >= call)) {
          bad = bad || __any(lane != 8 && (unsigned)(a >> 32) != call);
          break;
        }
        if (bad || NOW() - t0 > DEADLINE) { bad = true; break; }
        __builtin_amdgcn_s_sleep(1);
      }
      gm = row_umax(gm);
      if (lane == 0) red[4] = bad ? 0x7fc00000u : __float_as_uint(448.0f / __uint_as_float(gm));
      if (bad && lane == 0) __hip_atomic_store(tmo, call, RLX_AGT);
    }
    WAIT_L(0);
    BAR();
    if (bid == 15 * 8 && lane == 0) __hip_atomic_store((GAS unsigned*)(myws + WS_LAST), (unsigned)NOW(), RLX_AGT);
    BAR();
    if (drel != 0 && !early && lane == 0)
      __hip_atomic_store((GAS unsigned*)(A.ws[drel] + WS_DONE + (rank >> 1) * 256 + (rank & 1) * 64) + tile, call, RLX_SYS);
    if (bid == 15 * 8) {
      const bool mine = (lane >> 3) == rank;
      GAS u64* f = mine ? (GAS u64*)tmo : (GAS u64*)(myws + WS_DONE + (lane >> 4) * 256) + (lane & 15);
      const bool e0 = early_index(2 * (lane & 7)) < NF, e1 = early_index(2 * (lane & 7) + 1) < NF;
      for (;;) {
        const u64 a = __hip_atomic_load(f, RLX_SYS);
        const bool ok = mine || ((e0 || (unsigned)a >= call) && (e1 || (unsigned)(a >> 32) >= call));
        if (__all(ok)) break;
        if (__any(mine && (unsigned)a == call) || NOW() - t0 > DEADLINE) {
          if (lane == 0) { __hip_atomic_store(tmo, call, RLX_AGT); A.out[0][(size_t)(rank * MS + half * BM) * NS + tc * BN] = 0x7f; }
          break;
        }
      }
    }
    return;
  }

  const int wm = wid >> 1, wn = wid & 1;
  const int fr = lane & 15, g = lane >> 4;
  const int xa0 = X_OFF + (wm * 128 + fr) * 128 + ((g ^ ((fr >> 1) & 7)) << 4);
  const int xa1 = xa0 ^ 64;
  const int q = fr >> 2, p = fr & 3, h = q | ((g & 1) << 2);
  int wa[4];
#pragma unroll
  for (int nt = 0; nt < 4; ++nt) wa[nt] = (8 * g + q) * 256 + (((4 * wn + nt) ^ h) << 5) + p * 8;
  f32x4 acc[8][4];
#pragma unroll
  for (int a = 0; a < 8; ++a)
#pragma unroll
    for (int b = 0; b < 4; ++b) { acc[a][b] = (f32x4){0.f, 0.f, 0.f, 0.f}; asm volatile("" : "+v"(acc[a][b])); }
  s16x4 WA[4][4], WC[4][4];
  bf16x8 X[8][2];
#define MF(a, b, Wf, ks) do { if (!skip) { const bf16x8 wv = __builtin_bit_cast(bf16x8, __builtin_shufflevector(Wf[b][2 * ks], Wf[b][2 * ks + 1], 0, 1, 2, 3, 4, 5, 6, 7)); \
    asm volatile("v_mfma_f32_16x16x32_bf16 %0, %1, %2, %0" : "+v"(acc[a][b]) : "v"(wv), "v"(X[a][ks])); } SCHED(); } while (0)
#define RW(Wf, b, i) do { Wf[b][i] = __builtin_amdgcn_ds_read_tr16_b64_v4i16((LAS s16x4*)(lds + wp[b] + wimm + (i >> 1) * 8192 + (i & 1) * 1024)); SCHED(); } while (0)
#define RX(a, ks) do { X[a][ks] = *(LAS bf16x8*)(lds + (ks ? xp1 : xp0) + a * 2048); SCHED(); } while (0)
#define GRP(a, r0, r1, r2, r3) MF(a, 0, Wc, 0); r0; MF(a, 0, Wc, 1); MF(a, 1, Wc, 0); r1; MF(a, 1, Wc, 1); MF(a, 2, Wc, 0); r2; MF(a, 2, Wc, 1); MF(a, 3, Wc, 0); r3; MF(a, 3, Wc, 1);
  auto step = [&](s16x4 (&Wp)[4][4], s16x4 (&Wc)[4][4], int xoff, const int (&wp)[4], int wimm, bool first) {
    bool skip = first;
    const int xp0 = xa0 + xoff, xp1 = xa1 + xoff;
    MF(5, 0, Wp, 0); RW(Wc, 0, 0); MF(5, 0, Wp, 1); RW(Wc, 0, 1); MF(5, 1, Wp, 0); RW(Wc, 0, 2); MF(5, 1, Wp, 1); RW(Wc, 0, 3);
    MF(5, 2, Wp, 0); RW(Wc, 1, 0); MF(5, 2, Wp, 1); RW(Wc, 1, 1); MF(5, 3, Wp, 0); RW(Wc, 1, 2); MF(5, 3, Wp, 1); RW(Wc, 1, 3);
    MF(6, 0, Wp, 0); RW(Wc, 2, 0); MF(6, 0, Wp, 1); RW(Wc, 2, 1); MF(6, 1, Wp, 0); RW(Wc, 2, 2); MF(6, 1, Wp, 1); RW(Wc, 2, 3);
    MF(6, 2, Wp, 0); RW(Wc, 3, 0); MF(6, 2, Wp, 1); RW(Wc, 3, 1); MF(6, 3, Wp, 0); RW(Wc, 3, 2); MF(6, 3, Wp, 1); RW(Wc, 3, 3);
    MF(7, 0, Wp, 0); RX(0, 0); MF(7, 0, Wp, 1); RX(0, 1); MF(7, 1, Wp, 0); MF(7, 1, Wp, 1); RX(1, 0);
    MF(7, 2, Wp, 0); MF(7, 2, Wp, 1); RX(1, 1); MF(7, 3, Wp, 0); MF(7, 3, Wp, 1);
    skip = false;
    GRP(0, RX(2, 0), RX(2, 1), , )
    GRP(1, RX(3, 0), RX(3, 1), , )
    GRP(2, RX(4, 0), RX(4, 1), RX(5, 0), RX(5, 1))
    GRP(3, RX(6, 0), RX(6, 1), RX(7, 0), RX(7, 1))
    GRP(4, , , , )
  };
  if (remote_x) BAR();
  BAR();
  SCHED();
  step(WC, WA, 0, wa, 0, true);
  int xoff = 0;
  for (int t = 1; t < NT - 1; t += 2) {
    xoff = xoff == (NXS - 1) * XB ? 0 : xoff + XB;
    WAIT_L(0); BAR(); SCHED();
    step(WA, WC, xoff, wa, WB, false);
    xoff = xoff == (NXS - 1) * XB ? 0 : xoff + XB;
#pragma unroll
    for (int b = 0; b < 4; ++b) wa[b] ^= 2 * WB;
    WAIT_L(0); BAR(); SCHED();
    step(WC, WA, xoff, wa, 0, false);
  }
  xoff = xoff == (NXS - 1) * XB ? 0 : xoff + XB;
  WAIT_L(0); BAR(); SCHED();
  step(WA, WC, xoff, wa, WB, false);
  float mx = 0.f;
#define MX(a, b) do { asm volatile("v_max3_f32 %0, |%1|, |%2|, %0\n\tv_max3_f32 %0, |%3|, |%4|, %0" : "+v"(mx) : "v"(acc[a][b][0]), "v"(acc[a][b][1]), "v"(acc[a][b][2]), "v"(acc[a][b][3])); SCHED(); } while (0)
#define MX2(a, b) MX(a, b); MX(a + 1, b)
  {
    const bool skip = false;
    auto& Wc = WC;
    GRP(5, MX2(0, 0), MX2(0, 1), MX2(0, 2), MX2(0, 3))
    GRP(6, MX2(2, 0), MX2(2, 1), MX2(2, 2), MX2(2, 3))
    GRP(7, MX2(4, 0), MX2(4, 1), MX2(4, 2), MX2(4, 3))
    MX(6, 0); MX(6, 1); MX(6, 2); MX(6, 3);
    asm volatile("s_nop 7" ::: "memory");
    MX(7, 0); MX(7, 1); MX(7, 2); MX(7, 3);
    asm volatile("s_nop 1" : "+v"(mx));
  }
  {
    const unsigned m = wave_umax(__float_as_uint(mx));
    if (lane == 0) red[wid] = m;
  }
  WAIT_L(0);
  BAR();
  const unsigned lo = (unsigned)((wm * 128 + fr) * NS + wn * 64 + g * 16);
  unsigned char* op = A.out[drel] + (size_t)(src * MS + half * BM) * NS + tc * BN + lo;
  if (early) {
    const unsigned lvl = __builtin_amdgcn_readfirstlane(red[6]) >> 1;
    const size_t slot = WS_YS + (size_t)((red[6] & 1u) * (NR * TPP) + tile) * YS_TILE + wid * YS_WAVE + lane * 16;
    unsigned char* sp = A.ws[drel] + slot + (size_t)rank * TPP * YS_TILE;
#define PUT(off, val) do { const u32x4 o_ = val; asm volatile("global_store_dwordx4 %0, %1, off offset:%c2 sc0 sc1\n\ts_nop 1" ::"v"(sp + c * 4096), "v"(o_), "i"(off) : "memory"); if (lvl == 0) WAIT_V(FTH); else if (lvl == 1) WAIT_V(FTH1); else WAIT_V(FTH2); } while (0)
    unsigned head = (__builtin_bit_cast(u16, (__bf16)mx) >> 8) & 0x7fu;
    const unsigned brep = head * 0x01010101u | 0x80808080u;
#pragma unroll
    for (int c = 0; c < 4; ++c) {
      unsigned L[8], H[8], N[4], ovf = 0;
#pragma unroll
      for (int j = 0; j < 8; ++j) {
        const f32x4 v = acc[2 * c + (j >> 2)][j & 3];
        typedef __bf16 bf16x2 __attribute__((ext_vector_type(2)));
        const unsigned p0 = __builtin_bit_cast(unsigned, (bf16x2){(__bf16)v[0], (__bf16)v[1]}), p1 = __builtin_bit_cast(unsigned, (bf16x2){(__bf16)v[2], (__bf16)v[3]});
        L[j] = __builtin_amdgcn_perm(p1, p0, 0x06040200u);
        H[j] = __builtin_amdgcn_perm(p1, p0, 0x07050301u);
      }
      PUT(0, ((u32x4){L[0], L[1], L[2], L[3]}));
      PUT(1024, ((u32x4){L[4], L[5], L[6], L[7]}));
#pragma unroll
      for (int k = 0; k < 4; ++k) {
        const unsigned d0 = brep - (H[2 * k] & 0x7f7f7f7fu), d1 = brep - (H[2 * k + 1] & 0x7f7f7f7fu);
        ovf |= ((d0 ^ 0x80808080u) | (d1 ^ 0x80808080u)) & 0xf8f8f8f8u;
        N[k] = (d0 & 0x07070707u) | ((H[2 * k] & 0x80808080u) >> 4) | ((d1 & 0x07070707u) << 4) | (H[2 * k + 1] & 0x80808080u);
      }
      if (__any(ovf != 0)) {
        PUT(2048, ((u32x4){H[0], H[1], H[2], H[3]}));
        PUT(3072, ((u32x4){H[4], H[5], H[6], H[7]}));
        head |= 256u << c;
      } else PUT(2048, ((u32x4){N[0], N[1], N[2], N[3]}));
    }
    asm volatile("global_store_dword %0, %1, off sc0 sc1" ::"v"(sp + 16384 - lane * 12), "v"(head) : "memory");
    WAIT_V(0);
    BAR();
    BAR();
    const int from = (rank - drel) & 7;
    op = A.out[0] + (size_t)(from * MS + half * BM) * NS + tc * BN + lo;
    if (__builtin_amdgcn_readfirstlane(red[5]) == 0) {
      const unsigned char* lp = myws + slot + (size_t)from * TPP * YS_TILE;
      u32x4 r[12];
      asm volatile("global_load_dword %0, %13, off sc0 sc1\n\t"
                   "global_load_dwordx4 %1, %14, off offset:-4096 sc0 sc1\n\tglobal_load_dwordx4 %2, %14, off offset:-3072 sc0 sc1\n\tglobal_load_dwordx4 %3, %14, off offset:-2048 sc0 sc1\n\t"
                   "global_load_dwordx4 %4, %14, off sc0 sc1\n\tglobal_load_dwordx4 %5, %14, off offset:1024 sc0 sc1\n\tglobal_load_dwordx4 %6, %14, off offset:2048 sc0 sc1\n\t"
                   "global_load_dwordx4 %7, %15, off offset:-4096 sc0 sc1\n\tglobal_load_dwordx4 %8, %15, off offset:-3072 sc0 sc1\n\tglobal_load_dwordx4 %9, %15, off offset:-2048 sc0 sc1\n\t"
                   "global_load_dwordx4 %10, %15, off sc0 sc1\n\tglobal_load_dwordx4 %11, %15, off offset:1024 sc0 sc1\n\tglobal_load_dwordx4 %12, %15, off offset:2048 sc0 sc1\n\ts_waitcnt vmcnt(0)"
                   : "=&v"(head), "=&v"(r[0]), "=&v"(r[1]), "=&v"(r[2]), "=&v"(r[3]), "=&v"(r[4]), "=&v"(r[5]), "=&v"(r[6]), "=&v"(r[7]), "=&v"(r[8]), "=&v"(r[9]), "=&v"(r[10]), "=&v"(r[11])
                   : "v"(lp + 16384 - lane * 12), "v"(lp + 4096), "v"(lp + 12288) : "memory");
      const unsigned raw = __builtin_amdgcn_readfirstlane(head) >> 8, base = (head & 0x7fu) * 0x01010101u;
#pragma unroll
      for (int c = 0; c < 4; ++c) {
        unsigned H[8];
        const u32x4 n4 = r[3 * c + 2];
        if ((raw >> c) & 1u) {
          u32x4 h2;
          asm volatile("global_load_dwordx4 %0, %1, off offset:3072 sc0 sc1\n\ts_waitcnt vmcnt(0)" : "=&v"(h2) : "v"(lp + c * 4096) : "memory");
#pragma unroll
          for (int k = 0; k < 4; ++k) { H[k] = n4[k]; H[4 + k] = h2[k]; }
        } else {
#pragma unroll
          for (int k = 0; k < 4; ++k) {
            H[2 * k] = (base - (n4[k] & 0x07070707u)) | ((n4[k] & 0x08080808u) << 4);
            H[2 * k + 1] = (base - ((n4[k] >> 4) & 0x07070707u)) | (n4[k] & 0x80808080u);
          }
        }
#pragma unroll
        for (int j = 0; j < 8; ++j) {
          const unsigned l = r[3 * c + (j >> 2)][j & 3];
          acc[2 * c + (j >> 2)][j & 3] = (f32x4){__uint_as_float(__builtin_amdgcn_perm(H[j], l, 0x04000c0cu)), __uint_as_float(__builtin_amdgcn_perm(H[j], l, 0x05010c0cu)),
                                                 __uint_as_float(__builtin_amdgcn_perm(H[j], l, 0x06020c0cu)), __uint_as_float(__builtin_amdgcn_perm(H[j], l, 0x07030c0cu))};
        }
      }
    }
  }
  BAR();
  const float inv = __uint_as_float(red[4]);
  const bool throttle = drel != 0 && !early;
#pragma unroll
  for (int a = 0; a < 8; ++a) {
    unsigned d[4];
#pragma unroll
    for (int b = 0; b < 4; ++b) {
      const f32x4 v = acc[a][b] * inv;
      int pk = __builtin_amdgcn_cvt_pk_fp8_f32(v[0], v[1], 0, false);
      pk = __builtin_amdgcn_cvt_pk_fp8_f32(v[2], v[3], pk, true);
      d[b] = pk;
    }
    { auto r = __builtin_amdgcn_permlane32_swap(d[0], d[2], false, false); d[0] = r[0]; d[2] = r[1]; }
    { auto r = __builtin_amdgcn_permlane32_swap(d[1], d[3], false, false); d[1] = r[0]; d[3] = r[1]; }
    { auto r = __builtin_amdgcn_permlane16_swap(d[0], d[1], false, false); d[0] = r[0]; d[1] = r[1]; }
    { auto r = __builtin_amdgcn_permlane16_swap(d[2], d[3], false, false); d[2] = r[0]; d[3] = r[1]; }
    const u32x4 o = {d[0], d[1], d[2], d[3]};
    asm volatile("global_store_dwordx4 %0, %1, off sc0 sc1\n\ts_nop 1" ::"v"(op + (size_t)a * 16 * NS), "v"(o) : "memory");
    if (throttle) WAIT_V(QTH);
  }
  WAIT_V(0);
  BAR();
}

extern "C" void kernel_launch(int rank, int nranks, ncclComm_t comm, hipStream_t stream,
                              void* const* d_in, const int* in_sizes, int n_in,
                              void* d_out, int out_size, const int* in_dtypes, int out_dtype,
                              void* const* d_out_all, void* const* d_ws_all, size_t ws_bytes) {
  (void)comm; (void)in_sizes; (void)n_in; (void)d_out; (void)out_size; (void)in_dtypes; (void)out_dtype;
  if (nranks != NR || (unsigned)rank >= (unsigned)NR || ws_bytes < WS_END) { fprintf(stderr, "unexpected configuration\n"); return; }
  Args a;
  a.x = (const u16*)d_in[0]; a.w = (const u16*)d_in[1];
  for (int i = 0; i < NR; ++i) { a.out[i] = (unsigned char*)d_out_all[(rank + i) & 7]; a.ws[i] = (unsigned char*)d_ws_all[(rank + i) & 7]; }
  a.rank = rank; a.pad = 0;
  hipLaunchKernelGGL(fused_gemm_a2a, dim3(NWG), dim3(512), 0, stream, a);
  HIP_CHECK(hipGetLastError());
}
```

```cpp
#include <hip/hip_runtime.h>
#include <rccl/rccl.h>
#include <cstdio>

#define HIP_CHECK(expr) do { hipError_t _e = (expr); if (_e != hipSuccess) { \
    fprintf(stderr, "HIP error %s at %s:%d\n", hipGetErrorString(_e), __FILE__, __LINE__); \
    return; } } while (0)

#define LAS __attribute__((address_space(3)))
#define GAS __attribute__((address_space(1)))
typedef __bf16 bf16x8 __attribute__((ext_vector_type(8)));
typedef short s16x4 __attribute__((ext_vector_type(4)));
typedef float f32x4 __attribute__((ext_vector_type(4)));
typedef unsigned u32x4 __attribute__((ext_vector_type(4)));
typedef unsigned short u16;
typedef unsigned long long u64;

namespace {
constexpr int NR = 8;
constexpr int MG = 4096, NG = 16384, KG = 4096;
constexpr int MS = MG / NR;
constexpr int NS = NG / NR;
constexpr int BM = 256, BN = 128, BK = 64, NT = KG / BK;
constexpr int XB = BM * BK * 2, WB = BK * BN * 2;
constexpr int NXS = 3, NWS = 4;
constexpr int X_OFF = NWS * WB;
constexpr int LDS_BYTES = NXS * XB + NWS * WB;
constexpr int NWG = 256;
constexpr int TPP = 16;
constexpr u64 DEADLINE = 150000000ull;
constexpr int NF = 12;
constexpr int POFF = 1000, POFFB = 6800, SBL = 2500;
constexpr int TF0 = 47500, TFD = 935;
constexpr int TEND = 58400;
constexpr int SP = 42;
constexpr int SOFTF = 260, SOFTS = 1500, SOFT = 2800;
constexpr int EOFF = 600, NE = 11, TAUE = 620, TAU = 722, STG = 45;
constexpr int LEAD = 3;
constexpr int WSL = 5, WSLS = 7, WSLF = 4, DSL = 9;
constexpr int FTH = 12, FTH1 = 3, FTH2 = 1, QTH = 3;
constexpr unsigned LT1 = 40, LT2 = 120;

constexpr size_t WS_TMO = 0;
constexpr size_t WS_LAST = 256;
constexpr size_t WS_DONE = 1024;
constexpr size_t WS_GLB = 4096;
constexpr size_t WS_LOC = 8192;
constexpr size_t WS_XF = 16384;
constexpr size_t WS_YF = 32768;
constexpr size_t WS_CNT = 65536;
constexpr size_t WS_XS = 2u << 20;
constexpr int PCS = 2048;
constexpr size_t XS_SLOT = (size_t)NT * 32 * PCS;
constexpr size_t WS_YS = WS_XS + NR * XS_SLOT;
constexpr int YS_WAVE = 17 * 1024;
constexpr size_t YS_TILE = 4 * YS_WAVE;
constexpr size_t WS_END = WS_YS + (size_t)2 * NR * TPP * YS_TILE;
static_assert(NF <= TPP && WS_YF + NR * TPP * 256 <= WS_CNT && WS_CNT + NWG * 256 <= (1u << 20), "layout");

struct Args {
  unsigned char* ws[NR];
  unsigned char* out[NR];
  const u16* x; const u16* w;
  int rank; int pad;
};
}

#define BAR() do { asm volatile("" ::: "memory"); __builtin_amdgcn_s_barrier(); asm volatile("" ::: "memory"); } while (0)
#define WAIT_V(n) asm volatile("s_waitcnt vmcnt(%0)" ::"n"(n) : "memory")
#define WAIT_L(n) asm volatile("s_waitcnt lgkmcnt(%0)" ::"n"(n) : "memory")
#define SCHED() __builtin_amdgcn_sched_barrier(0)
#define RLX_SYS __ATOMIC_RELAXED, __HIP_MEMORY_SCOPE_SYSTEM
#define RLX_AGT __ATOMIC_RELAXED, __HIP_MEMORY_SCOPE_AGENT
#define NOW() __builtin_amdgcn_s_memrealtime()

template <int CTRL> __device__ __forceinline__ unsigned dpp_umax(unsigned v) {
  const unsigned t = __builtin_amdgcn_update_dpp(v, v, CTRL, 0xf, 0xf, false);
  return t > v ? t : v;
}
__device__ __forceinline__ unsigned row_umax(unsigned v) {
  v = dpp_umax<0xB1>(v); v = dpp_umax<0x4E>(v); v = dpp_umax<0x141>(v); return dpp_umax<0x140>(v);
}
__device__ __forceinline__ unsigned wave_umax(unsigned v) {
  v = row_umax(v);
  const unsigned a = __builtin_amdgcn_readlane(v, 0), b = __builtin_amdgcn_readlane(v, 16), c = __builtin_amdgcn_readlane(v, 32), d = __builtin_amdgcn_readlane(v, 48);
  return max(max(a, b), max(c, d));
}

__device__ __forceinline__ unsigned vm_busy() { return __builtin_amdgcn_s_getreg(7 | (31 << 11)) & 0x00C0000Fu; }

__global__ __launch_bounds__(512, 2) void fused_gemm_a2a(Args A) {
  __shared__ __attribute__((aligned(16))) char smem[LDS_BYTES];
  LAS char* lds = (LAS char*)smem;
  const int tid = threadIdx.x, wid = __builtin_amdgcn_readfirstlane(tid >> 6), lane = tid & 63;
  const int bid = blockIdx.x, xcd = bid & 7, idx = (bid >> 3) & 31;
  const int rank = A.rank;
  int src = rank, half = 0, drel = 0, tc;
  if (idx < 14) { const int a = xcd * 14 + idx; drel = 1 + (a >> 4); tc = a & 15; }
  else if (idx < 16) tc = xcd * 2 + (idx - 14);
  else { const int k = idx - 16; src = (rank + (xcd & 1) * 4 + (k >> 2)) & 7; half = 1; tc = (xcd >> 1) * 4 + (k & 3); }
  const bool remote_x = src != rank;
  const int n0 = ((rank + drel) & 7) * NS + tc * BN;
  const int koff = drel ? xcd * 8 : 0;
  auto early_index = [](int c) { return (c & 3) == 1 ? 12 + (c >> 2) : c - ((c + 2) >> 2); };
  const int fidx = early_index(tc);
  const bool early = drel != 0 && fidx < NF;
  const int tile = tc;
  unsigned char* myws = A.ws[0];
  LAS unsigned* red = (LAS unsigned*)lds;

  if (wid >= 4) {
    if (wid < 6) {
      const int j = wid - 4;
      LAS char* base = lds + X_OFF + j * 16384;
      if (!remote_x) {
        auto rs = __builtin_amdgcn_make_buffer_rsrc((void*)(A.x + (size_t)(half * BM + j * 128) * KG), 0, 128 * KG * 2, 0x00020000);
        const int v0 = (lane >> 3) * (KG * 2) + (((lane & 7) ^ (lane >> 4)) << 4);
        const int v1 = (lane >> 3) * (KG * 2) + (((lane & 7) ^ (4 | (lane >> 4))) << 4);
        auto issue = [&](int t, int slot) {
#pragma unroll
          for (int i = 0; i < 16; ++i)
            __builtin_amdgcn_raw_ptr_buffer_load_lds(rs, (LAS void*)(base + slot * XB + i * 1024), 16, (i & 1) ? v1 : v0, i * 8 * (KG * 2) + ((t + koff) & 63) * 128, 0, 0);
        };
        issue(0, 0); issue(1, 1);
        int slot = 2;
        WAIT_V(16); BAR();
        for (int t = 0; t < NT - 1; ++t) {
          if (t + 2 < NT) { issue(t + 2, slot); WAIT_V(16); } else WAIT_V(0);
          slot = slot == NXS - 1 ? 0 : slot + 1;
          BAR();
        }
      } else {
        const unsigned char* sb = myws + WS_XS + (size_t)src * XS_SLOT + j * 16 * PCS;
        const unsigned vo = lane * 16 + (lane < 56 ? 512 : 640);
        auto issue = [&](int t, int slot) {
#pragma unroll
          for (int i = 0; i < 16; ++i)
            __builtin_amdgcn_global_load_lds((const GAS void*)(sb + (size_t)((t * 32 + i) * PCS) + vo), (LAS void*)(base + slot * XB + i * 1024), 16, 0, 17);
        };
        BAR();
        issue(0, 0); issue(1, 1);
        int slot = 2;
        WAIT_V(16); BAR();
        for (int t = 0; t < NT - 1; ++t) {
          if (t + 2 < NT) { issue(t + 2, slot); WAIT_V(16); } else WAIT_V(0);
          slot = slot == NXS - 1 ? 0 : slot + 1;
          BAR();
        }
      }
      return;
    }
    if (wid == 6) {
      auto rs = __builtin_amdgcn_make_buffer_rsrc((void*)(A.w + n0), 0, (unsigned)((size_t)KG * NG * 2 - (size_t)n0 * 2), 0x00020000);
      const int v0 = (lane >> 4) * (NG * 2) + (((lane & 15) ^ ((lane >> 4) << 1)) << 4);
      const int v1 = (lane >> 4) * (NG * 2) + (((lane & 15) ^ ((4 | (lane >> 4)) << 1)) << 4);
      auto issue = [&](int t, int slot, bool stream) {
        if (stream) {
#pragma unroll
          for (int i = 0; i < 16; ++i)
            __builtin_amdgcn_raw_ptr_buffer_load_lds(rs, (LAS void*)(lds + slot * WB + i * 1024), 16, (i & 2) ? v1 : v0, (((t + koff) & 63) * 64 + i * 4) * (NG * 2), 0, 2);
        } else {
#pragma unroll
          for (int i = 0; i < 16; ++i)
            __builtin_amdgcn_raw_ptr_buffer_load_lds(rs, (LAS void*)(lds + slot * WB + i * 1024), 16, (i & 2) ? v1 : v0, (((t + koff) & 63) * 64 + i * 4) * (NG * 2), 0, 0);
        }
      };
      if (remote_x) BAR();
      issue(0, 0, false); issue(1, 1, false);
      unsigned last;
      asm volatile("global_load_dword %0, %1, off sc1" : "=v"(last) : "v"(myws + WS_LAST) : "memory");
      WAIT_V(16); BAR();
      __builtin_amdgcn_s_sleep(WSL);
      issue(2, 2, false); issue(3, 3, false);
      asm volatile("s_waitcnt vmcnt(32)" : "+v"(last) :: "memory");
      BAR();
      const bool stream = drel != 0 && (unsigned)NOW() - (unsigned)__builtin_amdgcn_readfirstlane(last) > 30000u;
      WAIT_V(16); BAR();
      auto run = [&](bool st) {
        for (int t = 2; t < NT - 2; t += 2) {
          if (fidx >= 11) __builtin_amdgcn_s_sleep(WSLS); else if (early) __builtin_amdgcn_s_sleep(WSLF); else __builtin_amdgcn_s_sleep(WSL);
          issue(t + 2, (t + 2) & 3, st); issue(t + 3, (t + 3) & 3, st);
          WAIT_V(32); BAR();
          WAIT_V(16); BAR();
        }
      };
      if (stream) run(true); else run(false);
      WAIT_V(0); BAR();
      return;
    }
    const u64 t0 = NOW();
    GAS unsigned* cnt = (GAS unsigned*)(myws + WS_CNT + bid * 256);
    const unsigned one = 1;
    u64 ex;
    GAS unsigned* tmo = (GAS unsigned*)(myws + WS_TMO);
    const GAS void* tab = (const GAS char*)(const GAS void*)__builtin_amdgcn_kernarg_segment_ptr() + (lane >> 3) * 8;
    const int first = drel ? POFF : half ? POFFB : POFFB - SBL, last = early ? TF0 + fidx * TFD : TEND;
    const int pace = (last - first) / (NT - 1);
    const int soft = drel != 0 ? (fidx >= 11 ? SOFTS : fidx * SOFTF) : SOFT;
    auto due_at = [&](int t) { return t0 + (unsigned)(first + t * pace + (t < SP ? t * soft / SP : (63 - t) * soft / (63 - SP))) / 10u; };
    unsigned call;
    unsigned char* pws;
    bool bad = false;
    if (!half) {
      {
        const char* tp = (const char*)A.x + (size_t)(idx * 16 + (lane & 7)) * (KG * 2) + (xcd * 8 + (lane >> 3)) * 128;
        asm volatile("global_load_dword v253, %0, off\n\tglobal_load_dword v253, %1, off" ::"v"(tp), "v"(tp + 8 * KG * 2) : "memory", "v253");
      }
      for (int t = 0; t < NT; ++t) {
        const u64 due = t ? due_at(t) : 0;
        while (NOW() < due) __builtin_amdgcn_s_sleep(1);
        BAR();
        if (t == 0) {
          __builtin_amdgcn_s_sleep(8);
          asm volatile("global_load_dwordx2 v[250:251], %1, off\n\ts_mov_b64 %0, exec\n\ts_mov_b64 exec, 1\n\tglobal_atomic_add v252, %2, %3, off sc0\n\ts_mov_b64 exec, %0"
                       : "=&s"(ex) : "v"(tab), "v"(cnt), "v"(one) : "memory", "v250", "v251", "v252");
        }
      }
      unsigned plo, phi, c0;
      asm volatile("s_waitcnt vmcnt(0)\n\tv_mov_b32 %0, v250\n\tv_mov_b32 %1, v251\n\tv_mov_b32 %2, v252\n\ts_nop 0" : "=v"(plo), "=v"(phi), "=v"(c0) :: "memory");
      call = __builtin_amdgcn_readfirstlane(c0) + 1u;
      pws = (unsigned char*)(((u64)phi << 32) | plo);
    } else {
      const int l = ((idx - 4) & 15) * 8 + xcd, pg = l & 15, pj = l >> 4;
      const char* s0 = (const char*)A.x + (size_t)(BM + pg * 16 + (lane >> 3)) * (KG * 2) + (((lane & 7) ^ (lane >> 4)) << 4) + pj * 128;
      const char* s1 = (const char*)A.x + (size_t)(BM + pg * 16 + 8 + (lane >> 3)) * (KG * 2) + (((lane & 7) ^ (4 | (lane >> 4))) << 4) + pj * 128;
      u32x4 r[16];
      unsigned c0;
      asm volatile("global_load_dwordx4 %0, %5, off\n\tglobal_load_dwordx4 %1, %6, off\n\tglobal_load_dwordx2 %3, %8, off\n\t"
                   "s_mov_b64 %4, exec\n\ts_mov_b64 exec, 1\n\tglobal_atomic_add %2, %7, %9, off sc0\n\ts_mov_b64 exec, %4\n\ts_waitcnt vmcnt(0)"
                   : "=&v"(r[0]), "=&v"(r[1]), "=&v"(c0), "=&v"(pws), "=&s"(ex) : "v"(s0), "v"(s1), "v"(cnt), "v"(tab), "v"(one) : "memory");
      call = __builtin_amdgcn_readfirstlane(c0) + 1u;
      const bool flagger = lane >= 8 && (lane & 7) == 0;
      const unsigned xs_mine = (unsigned)(WS_XS + (size_t)rank * XS_SLOT) + lane * 16 + (lane < 56 ? 512 : 640) + (pj * 32 + pg * 2) * PCS;
      GAS unsigned* fp = (GAS unsigned*)(pws + WS_XF + rank * 1024 + (pj >> 1) * 256 + (pj & 1) * 64) + pg;
      const GAS unsigned char* xf = (const GAS unsigned char*)(myws + WS_XF + src * 1024) + (lane & 7) * 8;
      int nb = 0;
      bool stored = false, polled = false;
      unsigned pbase = 0, ready = remote_x ? 0 : NT;
      auto burst = [&]() {
        const unsigned vo = xs_mine + nb * 8 * 32 * PCS;
#pragma unroll
        for (int d = 1; d < NR; ++d) {
          asm volatile("global_store_dwordx4 %0, %1, off sc0 sc1\n\ts_nop 1" ::"v"(A.ws[d] + vo), "v"(r[0]) : "memory");
          asm volatile("global_store_dwordx4 %0, %1, off offset:%c2 sc0 sc1\n\ts_nop 1" ::"v"(A.ws[d] + vo), "v"(r[1]), "i"(PCS) : "memory");
        }
        stored = true;
      };
      auto burst_due = [&]() { const int c = 8 * nb + pj; return NOW() >= t0 + (unsigned)(EOFF + (c > NE ? NE * TAUE + (c - NE) * TAU + pg * STG : c * TAUE + (c > 2 ? pg * STG : 0))) / 10u; };
      auto poll = [&]() {
        pbase = ready;
        asm volatile("global_load_dwordx2 v[250:251], %0, off sc0 sc1" :: "v"(xf + ((pbase + (lane >> 3)) & 6) * 128 + ((pbase + (lane >> 3)) & 1) * 64) : "memory", "v250", "v251");
        polled = true;
      };
      auto settle = [&]() {
        unsigned p0, p1;
        asm volatile("s_waitcnt vmcnt(0)\n\tv_mov_b32 %0, v250\n\tv_mov_b32 %1, v251" : "=v"(p0), "=v"(p1) :: "memory");
        if (stored) {
#pragma unroll
          for (int i = 0; i < 14; ++i) r[i] = r[i + 2];
          ++nb;
          if (flagger) __hip_atomic_store(fp, call * 16u + (unsigned)nb, RLX_SYS);
          stored = false;
        }
        if (polled) {
          const unsigned want = call * 16u + ((pbase + (lane >> 3)) >> 3) + 1u;
          const u64 miss = ~__ballot(p0 >= want && p1 >= want);
          ready = pbase + (miss ? (unsigned)__builtin_ctzll(miss) >> 3 : 8u);
          polled = false;
        }
      };
      auto gate = [&](unsigned need) {
        while (ready < need) {
          if (!stored && nb < 8 && burst_due()) burst();
          if (!polled) poll();
          settle();
          if (ready < need && (__hip_atomic_load(tmo, RLX_AGT) == call || NOW() - t0 > DEADLINE)) { bad = true; ready = NT; }
        }
      };
      auto load_rest = [&]() {
        asm volatile("global_load_dwordx4 %0, %6, off offset:1024\n\tglobal_load_dwordx4 %1, %7, off offset:1024\n\tglobal_load_dwordx4 %2, %6, off offset:2048\n\t"
                     "global_load_dwordx4 %3, %7, off offset:2048\n\tglobal_load_dwordx4 %4, %6, off offset:3072\n\tglobal_load_dwordx4 %5, %7, off offset:3072\n\ts_waitcnt vmcnt(0)"
                     : "=&v"(r[2]), "=&v"(r[3]), "=&v"(r[4]), "=&v"(r[5]), "=&v"(r[6]), "=&v"(r[7]) : "v"(s0), "v"(s1) : "memory");
        asm volatile("global_load_dwordx4 %0, %8, off\n\tglobal_load_dwordx4 %1, %9, off\n\tglobal_load_dwordx4 %2, %8, off offset:1024\n\tglobal_load_dwordx4 %3, %9, off offset:1024\n\t"
                     "global_load_dwordx4 %4, %8, off offset:2048\n\tglobal_load_dwordx4 %5, %9, off offset:2048\n\tglobal_load_dwordx4 %6, %8, off offset:3072\n\tglobal_load_dwordx4 %7, %9, off offset:3072\n\t"
                     "s_waitcnt vmcnt(0)"
                     : "=&v"(r[8]), "=&v"(r[9]), "=&v"(r[10]), "=&v"(r[11]), "=&v"(r[12]), "=&v"(r[13]), "=&v"(r[14]), "=&v"(r[15]) : "v"(s0 + 4096), "v"(s1 + 4096) : "memory");
      };
      auto wait_for = [&](int t) {
        for (const u64 due = due_at(t); NOW() < due;) {
          if (!stored && nb < 8 && burst_due()) burst();
          if (stored && !polled && !vm_busy()) settle();
          __builtin_amdgcn_s_sleep(1);
        }
      };
      if (pj < 3) {
        while (!burst_due()) __builtin_amdgcn_s_sleep(1);
        burst();
        WAIT_V(0);
        if (flagger) __hip_atomic_store(fp, call * 16u + 1u, RLX_SYS);
      }
      load_rest();
      settle();
      if (remote_x) { gate(2); BAR(); }
      wait_for(0);
      for (int t = 0; t < NT; ++t) {
        const unsigned need = t + 3 < NT ? t + 3 : NT;
        gate(need);
        wait_for(t);
        BAR();
        const int c = 8 * nb + pj;
        if (nb < 8 || ready < (unsigned)NT) {
          __builtin_amdgcn_s_sleep(DSL);
          if ((stored || polled) && (!vm_busy() || (stored && t + 4 >= c))) settle();
          else if ((t & 1) == 0) {
            if (!stored && nb < 8 && (t + 7 >= c || burst_due())) burst();
            if (!polled && ready < (unsigned)NT && ready < need + LEAD) poll();
          }
        }
      }
      settle();
      while (nb < 8) { burst(); settle(); }
    }
    const unsigned par = call & 1u;
    if (bad && lane == 0) __hip_atomic_store(tmo, call, RLX_AGT);
    GAS u64* gp = (GAS u64*)(pws + WS_GLB + (par * 8 + (lane & 7)) * 256) + rank;
    const unsigned late = (unsigned)(NOW() - due_at(NT - 1));
    if (lane == 0) red[6] = par | (late > LT2 ? 4u : late > LT1 ? 2u : 0u);
    WAIT_L(0);
    BAR();
    {
      const unsigned m = dpp_umax<0x4E>(dpp_umax<0xB1>(red[lane & 3]));
      if (lane == 0) __hip_atomic_store((GAS u64*)(myws + WS_LOC + (par * 16 + (bid >> 4)) * 256) + (bid & 15), ((u64)call << 32) | m, RLX_AGT);
    }
    if (early) {
      BAR();
      if (lane == 0) __hip_atomic_store((GAS unsigned*)(A.ws[drel] + WS_YF + (rank * TPP + tile) * 256), call, RLX_SYS);
      GAS unsigned* f = lane == 1 ? tmo : (GAS unsigned*)(myws + WS_YF + (((rank - drel) & 7) * TPP + tile) * 256);
      for (;;) {
        const unsigned v = __hip_atomic_load(f, RLX_SYS);
        if ((unsigned)__builtin_amdgcn_readlane(v, 0) >= call) break;
        if ((unsigned)__builtin_amdgcn_readlane(v, 1) == call || NOW() - t0 > DEADLINE) { bad = true; break; }
        __builtin_amdgcn_s_sleep(2);
      }
      if (lane == 0) red[5] = bad;
      WAIT_L(0);
      BAR();
    }
    if (bid == 14 * 8) {
      GAS u64* g = (GAS u64*)(myws + WS_LOC + (par * 16 + (lane >> 4)) * 256) + (lane & 15);
      unsigned m;
      for (;;) {
        const u64 a = __hip_atomic_load(g, RLX_AGT), b = __hip_atomic_load(g + 128, RLX_AGT), c = __hip_atomic_load(g + 256, RLX_AGT), d = __hip_atomic_load(g + 384, RLX_AGT);
        const bool ok = (unsigned)(a >> 32) == call && (unsigned)(b >> 32) == call && (unsigned)(c >> 32) == call && (unsigned)(d >> 32) == call;
        m = max(max((unsigned)a, (unsigned)b), max((unsigned)c, (unsigned)d));
        if (__all(ok)) break;
        if (NOW() - t0 > DEADLINE) { bad = true; break; }
      }
      m = wave_umax(m);
      __hip_atomic_store(gp, ((u64)call << 32) | (bad ? 0x7fc00000u : m), RLX_SYS);
    }
    {
      GAS u64* g = lane == 8 ? (GAS u64*)tmo : (GAS u64*)(myws + WS_GLB + (par * 8 + xcd) * 256) + (lane & 7);
      unsigned gm;
      for (;;) {
        const u64 a = __hip_atomic_load(g, RLX_SYS);
        gm = lane == 8 ? 0u : (unsigned)a;
        if (__builtin_amdgcn_readlane((unsigned)a, 8) == call) { bad = true; break; }
        if (__all(lane == 8 || (unsigned)(a >> 32) >= call)) {
          bad = bad || __any(lane != 8 && (unsigned)(a >> 32) != call);
          break;
        }
        if (bad || NOW() - t0 > DEADLINE) { bad = true; break; }
        __builtin_amdgcn_s_sleep(1);
      }
      gm = row_umax(gm);
      if (lane == 0) red[4] = bad ? 0x7fc00000u : __float_as_uint(448.0f / __uint_as_float(gm));
      if (bad && lane == 0) __hip_atomic_store(tmo, call, RLX_AGT);
    }
    WAIT_L(0);
    BAR();
    if (bid == 15 * 8 && lane == 0) __hip_atomic_store((GAS unsigned*)(myws + WS_LAST), (unsigned)NOW(), RLX_AGT);
    BAR();
    if (drel != 0 && !early && lane == 0)
      __hip_atomic_store((GAS unsigned*)(A.ws[drel] + WS_DONE + (rank >> 1) * 256 + (rank & 1) * 64) + tile, call, RLX_SYS);
    if (bid == 15 * 8) {
      const bool mine = (lane >> 3) == rank;
      GAS u64* f = mine ? (GAS u64*)tmo : (GAS u64*)(myws + WS_DONE + (lane >> 4) * 256) + (lane & 15);
      const bool e0 = early_index(2 * (lane & 7)) < NF, e1 = early_index(2 * (lane & 7) + 1) < NF;
      for (;;) {
        const u64 a = __hip_atomic_load(f, RLX_SYS);
        const bool ok = mine || ((e0 || (unsigned)a >= call) && (e1 || (unsigned)(a >> 32) >= call));
        if (__all(ok)) break;
        if (__any(mine && (unsigned)a == call) || NOW() - t0 > DEADLINE) {
          if (lane == 0) { __hip_atomic_store(tmo, call, RLX_AGT); A.out[0][(size_t)(rank * MS + half * BM) * NS + tc * BN] = 0x7f; }
          break;
        }
      }
    }
    return;
  }

  const int wm = wid >> 1, wn = wid & 1;
  const int fr = lane & 15, g = lane >> 4;
  const int xa0 = X_OFF + (wm * 128 + fr) * 128 + ((g ^ ((fr >> 1) & 7)) << 4);
  const int xa1 = xa0 ^ 64;
  const int q = fr >> 2, p = fr & 3, h = q | ((g & 1) << 2);
  int wa[4];
#pragma unroll
  for (int nt = 0; nt < 4; ++nt) wa[nt] = (8 * g + q) * 256 + (((4 * wn + nt) ^ h) << 5) + p * 8;
  f32x4 acc[8][4];
#pragma unroll
  for (int a = 0; a < 8; ++a)
#pragma unroll
    for (int b = 0; b < 4; ++b) { acc[a][b] = (f32x4){0.f, 0.f, 0.f, 0.f}; asm volatile("" : "+v"(acc[a][b])); }
  s16x4 WA[4][4], WC[4][4];
  bf16x8 X[8][2];
#define MF(a, b, Wf, ks) do { if (!skip) { const bf16x8 wv = __builtin_bit_cast(bf16x8, __builtin_shufflevector(Wf[b][2 * ks], Wf[b][2 * ks + 1], 0, 1, 2, 3, 4, 5, 6, 7)); \
    asm volatile("v_mfma_f32_16x16x32_bf16 %0, %1, %2, %0" : "+v"(acc[a][b]) : "v"(wv), "v"(X[a][ks])); } SCHED(); } while (0)
#define RW(Wf, b, i) do { Wf[b][i] = __builtin_amdgcn_ds_read_tr16_b64_v4i16((LAS s16x4*)(lds + wp[b] + wimm + (i >> 1) * 8192 + (i & 1) * 1024)); SCHED(); } while (0)
#define RX(a, ks) do { X[a][ks] = *(LAS bf16x8*)(lds + (ks ? xp1 : xp0) + a * 2048); SCHED(); } while (0)
#define GRP(a, r0, r1, r2, r3) MF(a, 0, Wc, 0); r0; MF(a, 0, Wc, 1); MF(a, 1, Wc, 0); r1; MF(a, 1, Wc, 1); MF(a, 2, Wc, 0); r2; MF(a, 2, Wc, 1); MF(a, 3, Wc, 0); r3; MF(a, 3, Wc, 1);
  auto step = [&](s16x4 (&Wp)[4][4], s16x4 (&Wc)[4][4], int xoff, const int (&wp)[4], int wimm, bool first) {
    bool skip = first;
    const int xp0 = xa0 + xoff, xp1 = xa1 + xoff;
    MF(5, 0, Wp, 0); RW(Wc, 0, 0); MF(5, 0, Wp, 1); RW(Wc, 0, 1); MF(5, 1, Wp, 0); RW(Wc, 0, 2); MF(5, 1, Wp, 1); RW(Wc, 0, 3);
    MF(5, 2, Wp, 0); RW(Wc, 1, 0); MF(5, 2, Wp, 1); RW(Wc, 1, 1); MF(5, 3, Wp, 0); RW(Wc, 1, 2); MF(5, 3, Wp, 1); RW(Wc, 1, 3);
    MF(6, 0, Wp, 0); RW(Wc, 2, 0); MF(6, 0, Wp, 1); RW(Wc, 2, 1); MF(6, 1, Wp, 0); RW(Wc, 2, 2); MF(6, 1, Wp, 1); RW(Wc, 2, 3);
    MF(6, 2, Wp, 0); RW(Wc, 3, 0); MF(6, 2, Wp, 1); RW(Wc, 3, 1); MF(6, 3, Wp, 0); RW(Wc, 3, 2); MF(6, 3, Wp, 1); RW(Wc, 3, 3);
    MF(7, 0, Wp, 0); RX(0, 0); MF(7, 0, Wp, 1); RX(0, 1); MF(7, 1, Wp, 0); MF(7, 1, Wp, 1); RX(1, 0);
    MF(7, 2, Wp, 0); MF(7, 2, Wp, 1); RX(1, 1); MF(7, 3, Wp, 0); MF(7, 3, Wp, 1);
    skip = false;
    GRP(0, RX(2, 0), RX(2, 1), , )
    GRP(1, RX(3, 0), RX(3, 1), , )
    GRP(2, RX(4, 0), RX(4, 1), RX(5, 0), RX(5, 1))
    GRP(3, RX(6, 0), RX(6, 1), RX(7, 0), RX(7, 1))
    GRP(4, , , , )
  };
  if (remote_x) BAR();
  BAR();
  SCHED();
  step(WC, WA, 0, wa, 0, true);
  int xoff = 0;
  for (int t = 1; t < NT - 1; t += 2) {
    xoff = xoff == (NXS - 1) * XB ? 0 : xoff + XB;
    WAIT_L(0); BAR(); SCHED();
    step(WA, WC, xoff, wa, WB, false);
    xoff = xoff == (NXS - 1) * XB ? 0 : xoff + XB;
#pragma unroll
    for (int b = 0; b < 4; ++b) wa[b] ^= 2 * WB;
    WAIT_L(0); BAR(); SCHED();
    step(WC, WA, xoff, wa, 0, false);
  }
  xoff = xoff == (NXS - 1) * XB ? 0 : xoff + XB;
  WAIT_L(0); BAR(); SCHED();
  step(WA, WC, xoff, wa, WB, false);
  float mx = 0.f;
#define MX(a, b) do { asm volatile("v_max3_f32 %0, |%1|, |%2|, %0\n\tv_max3_f32 %0, |%3|, |%4|, %0" : "+v"(mx) : "v"(acc[a][b][0]), "v"(acc[a][b][1]), "v"(acc[a][b][2]), "v"(acc[a][b][3])); SCHED(); } while (0)
#define MX2(a, b) MX(a, b); MX(a + 1, b)
  {
    const bool skip = false;
    auto& Wc = WC;
    GRP(5, MX2(0, 0), MX2(0, 1), MX2(0, 2), MX2(0, 3))
    GRP(6, MX2(2, 0), MX2(2, 1), MX2(2, 2), MX2(2, 3))
    GRP(7, MX2(4, 0), MX2(4, 1), MX2(4, 2), MX2(4, 3))
    MX(6, 0); MX(6, 1); MX(6, 2); MX(6, 3);
    asm volatile("s_nop 7" ::: "memory");
    MX(7, 0); MX(7, 1); MX(7, 2); MX(7, 3);
    asm volatile("s_nop 1" : "+v"(mx));
  }
  {
    const unsigned m = wave_umax(__float_as_uint(mx));
    if (lane == 0) red[wid] = m;
  }
  WAIT_L(0);
  BAR();
  const unsigned lo = (unsigned)((wm * 128 + fr) * NS + wn * 64 + g * 16);
  unsigned char* op = A.out[drel] + (size_t)(src * MS + half * BM) * NS + tc * BN + lo;
  if (early) {
    const unsigned lvl = __builtin_amdgcn_readfirstlane(red[6]) >> 1;
    const size_t slot = WS_YS + (size_t)((red[6] & 1u) * (NR * TPP) + tile) * YS_TILE + wid * YS_WAVE + lane * 16;
    unsigned char* sp = A.ws[drel] + slot + (size_t)rank * TPP * YS_TILE;
#define PUT(off, val) do { const u32x4 o_ = val; asm volatile("global_store_dwordx4 %0, %1, off offset:%c2 sc0 sc1\n\ts_nop 1" ::"v"(sp + c * 4096), "v"(o_), "i"(off) : "memory"); if (lvl == 0) WAIT_V(FTH); else if (lvl == 1) WAIT_V(FTH1); else WAIT_V(FTH2); } while (0)
    unsigned head = (__builtin_bit_cast(u16, (__bf16)mx) >> 8) & 0x7fu;
    const unsigned brep = head * 0x01010101u | 0x80808080u;
#pragma unroll
    for (int c = 0; c < 4; ++c) {
      unsigned L[8], H[8], N[4], ovf = 0;
#pragma unroll
      for (int j = 0; j < 8; ++j) {
        const f32x4 v = acc[2 * c + (j >> 2)][j & 3];
        typedef __bf16 bf16x2 __attribute__((ext_vector_type(2)));
        const unsigned p0 = __builtin_bit_cast(unsigned, (bf16x2){(__bf16)v[0], (__bf16)v[1]}), p1 = __builtin_bit_cast(unsigned, (bf16x2){(__bf16)v[2], (__bf16)v[3]});
        L[j] = __builtin_amdgcn_perm(p1, p0, 0x06040200u);
        H[j] = __builtin_amdgcn_perm(p1, p0, 0x07050301u);
      }
      PUT(0, ((u32x4){L[0], L[1], L[2], L[3]}));
      PUT(1024, ((u32x4){L[4], L[5], L[6], L[7]}));
#pragma unroll
      for (int k = 0; k < 4; ++k) {
        const unsigned d0 = brep - (H[2 * k] & 0x7f7f7f7fu), d1 = brep - (H[2 * k + 1] & 0x7f7f7f7fu);
        ovf |= ((d0 ^ 0x80808080u) | (d1 ^ 0x80808080u)) & 0xf8f8f8f8u;
        N[k] = (d0 & 0x07070707u) | ((H[2 * k] & 0x80808080u) >> 4) | ((d1 & 0x07070707u) << 4) | (H[2 * k + 1] & 0x80808080u);
      }
      if (__any(ovf != 0)) {
        PUT(2048, ((u32x4){H[0], H[1], H[2], H[3]}));
        PUT(3072, ((u32x4){H[4], H[5], H[6], H[7]}));
        head |= 256u << c;
      } else PUT(2048, ((u32x4){N[0], N[1], N[2], N[3]}));
    }
    asm volatile("global_store_dword %0, %1, off sc0 sc1" ::"v"(sp + 16384 - lane * 12), "v"(head) : "memory");
    WAIT_V(0);
    BAR();
    BAR();
    const int from = (rank - drel) & 7;
    op = A.out[0] + (size_t)(from * MS + half * BM) * NS + tc * BN + lo;
    if (__builtin_amdgcn_readfirstlane(red[5]) == 0) {
      const unsigned char* lp = myws + slot + (size_t)from * TPP * YS_TILE;
      u32x4 r[12];
      asm volatile("global_load_dword %0, %13, off sc0 sc1\n\t"
                   "global_load_dwordx4 %1, %14, off offset:-4096 sc0 sc1\n\tglobal_load_dwordx4 %2, %14, off offset:-3072 sc0 sc1\n\tglobal_load_dwordx4 %3, %14, off offset:-2048 sc0 sc1\n\t"
                   "global_load_dwordx4 %4, %14, off sc0 sc1\n\tglobal_load_dwordx4 %5, %14, off offset:1024 sc0 sc1\n\tglobal_load_dwordx4 %6, %14, off offset:2048 sc0 sc1\n\t"
                   "global_load_dwordx4 %7, %15, off offset:-4096 sc0 sc1\n\tglobal_load_dwordx4 %8, %15, off offset:-3072 sc0 sc1\n\tglobal_load_dwordx4 %9, %15, off offset:-2048 sc0 sc1\n\t"
                   "global_load_dwordx4 %10, %15, off sc0 sc1\n\tglobal_load_dwordx4 %11, %15, off offset:1024 sc0 sc1\n\tglobal_load_dwordx4 %12, %15, off offset:2048 sc0 sc1\n\ts_waitcnt vmcnt(0)"
                   : "=&v"(head), "=&v"(r[0]), "=&v"(r[1]), "=&v"(r[2]), "=&v"(r[3]), "=&v"(r[4]), "=&v"(r[5]), "=&v"(r[6]), "=&v"(r[7]), "=&v"(r[8]), "=&v"(r[9]), "=&v"(r[10]), "=&v"(r[11])
                   : "v"(lp + 16384 - lane * 12), "v"(lp + 4096), "v"(lp + 12288) : "memory");
      const unsigned raw = __builtin_amdgcn_readfirstlane(head) >> 8, base = (head & 0x7fu) * 0x01010101u;
#pragma unroll
      for (int c = 0; c < 4; ++c) {
        unsigned H[8];
        const u32x4 n4 = r[3 * c + 2];
        if ((raw >> c) & 1u) {
          u32x4 h2;
          asm volatile("global_load_dwordx4 %0, %1, off offset:3072 sc0 sc1\n\ts_waitcnt vmcnt(0)" : "=&v"(h2) : "v"(lp + c * 4096) : "memory");
#pragma unroll
          for (int k = 0; k < 4; ++k) { H[k] = n4[k]; H[4 + k] = h2[k]; }
        } else {
#pragma unroll
          for (int k = 0; k < 4; ++k) {
            H[2 * k] = (base - (n4[k] & 0x07070707u)) | ((n4[k] & 0x08080808u) << 4);
            H[2 * k + 1] = (base - ((n4[k] >> 4) & 0x07070707u)) | (n4[k] & 0x80808080u);
          }
        }
#pragma unroll
        for (int j = 0; j < 8; ++j) {
          const unsigned l = r[3 * c + (j >> 2)][j & 3];
          acc[2 * c + (j >> 2)][j & 3] = (f32x4){__uint_as_float(__builtin_amdgcn_perm(H[j], l, 0x04000c0cu)), __uint_as_float(__builtin_amdgcn_perm(H[j], l, 0x05010c0cu)),
                                                 __uint_as_float(__builtin_amdgcn_perm(H[j], l, 0x06020c0cu)), __uint_as_float(__builtin_amdgcn_perm(H[j], l, 0x07030c0cu))};
        }
      }
    }
  }
  BAR();
  const float inv = __uint_as_float(red[4]);
  const bool throttle = drel != 0 && !early;
#pragma unroll
  for (int a = 0; a < 8; ++a) {
    unsigned d[4];
#pragma unroll
    for (int b = 0; b < 4; ++b) {
      const f32x4 v = acc[a][b] * inv;
      int pk = __builtin_amdgcn_cvt_pk_fp8_f32(v[0], v[1], 0, false);
      pk = __builtin_amdgcn_cvt_pk_fp8_f32(v[2], v[3], pk, true);
      d[b] = pk;
    }
    { auto r = __builtin_amdgcn_permlane32_swap(d[0], d[2], false, false); d[0] = r[0]; d[2] = r[1]; }
    { auto r = __builtin_amdgcn_permlane32_swap(d[1], d[3], false, false); d[1] = r[0]; d[3] = r[1]; }
    { auto r = __builtin_amdgcn_permlane16_swap(d[0], d[1], false, false); d[0] = r[0]; d[1] = r[1]; }
    { auto r = __builtin_amdgcn_permlane16_swap(d[2], d[3], false, false); d[2] = r[0]; d[3] = r[1]; }
    const u32x4 o = {d[0], d[1], d[2], d[3]};
    asm volatile("global_store_dwordx4 %0, %1, off sc0 sc1\n\ts_nop 1" ::"v"(op + (size_t)a * 16 * NS), "v"(o) : "memory");
    if (throttle) WAIT_V(QTH);
  }
  WAIT_V(0);
  BAR();
}

extern "C" void kernel_launch(int rank, int nranks, ncclComm_t comm, hipStream_t stream,
                              void* const* d_in, const int* in_sizes, int n_in,
                              void* d_out, int out_size, const int* in_dtypes, int out_dtype,
                              void* const* d_out_all, void* const* d_ws_all, size_t ws_bytes) {
  (void)comm; (void)in_sizes; (void)n_in; (void)d_out; (void)out_size; (void)in_dtypes; (void)out_dtype;
  if (nranks != NR || (unsigned)rank >= (unsigned)NR || ws_bytes < WS_END) { fprintf(stderr, "unexpected configuration\n"); return; }
  Args a;
  a.x = (const u16*)d_in[0]; a.w = (const u16*)d_in[1];
  for (int i = 0; i < NR; ++i) { a.out[i] = (unsigned char*)d_out_all[(rank + i) & 7]; a.ws[i] = (unsigned char*)d_ws_all[(rank + i) & 7]; }
  a.rank = rank; a.pad = 0;
  hipLaunchKernelGGL(fused_gemm_a2a, dim3(NWG), dim3(512), 0, stream, a);
  HIP_CHECK(hipGetLastError());
}
```

```cpp
#include <hip/hip_runtime.h>
#include <rccl/rccl.h>
#include <cstdio>

#define HIP_CHECK(expr) do { hipError_t _e = (expr); if (_e != hipSuccess) { \
    fprintf(stderr, "HIP error %s at %s:%d\n", hipGetErrorString(_e), __FILE__, __LINE__); \
    return; } } while (0)

#define LAS __attribute__((address_space(3)))
#define GAS __attribute__((address_space(1)))
typedef __bf16 bf16x8 __attribute__((ext_vector_type(8)));
typedef short s16x4 __attribute__((ext_vector_type(4)));
typedef float f32x4 __attribute__((ext_vector_type(4)));
typedef unsigned u32x4 __attribute__((ext_vector_type(4)));
typedef unsigned short u16;
typedef unsigned long long u64;

namespace {
constexpr int NR = 8;
constexpr int MG = 4096, NG = 16384, KG = 4096;
constexpr int MS = MG / NR;
constexpr int NS = NG / NR;
constexpr int BM = 256, BN = 128, BK = 64, NT = KG / BK;
constexpr int XB = BM * BK * 2, WB = BK * BN * 2;
constexpr int NXS = 3, NWS = 4;
constexpr int X_OFF = NWS * WB;
constexpr int LDS_BYTES = NXS * XB + NWS * WB;
constexpr int NWG = 256;
constexpr int TPP = 16;
constexpr u64 DEADLINE = 150000000ull;
constexpr int NF = 12;
constexpr int POFF = 1000, POFFB = 6800, SBL = 2500;
constexpr int TF0 = 47500, TFD = 935;
constexpr int TEND = 58400;
constexpr int SP = 42;
constexpr int SOFTF = 260, SOFTS = 1500, SOFT = 2800;
constexpr int EOFF = 600, NE = 11, TAUE = 620, TAU = 722, STG = 45;
constexpr int LEAD = 3;
constexpr int WSL = 5, WSLS = 7, WSLF = 4, DSL = 9;
constexpr int FTH = 12, FTH1 = 3, FTH2 = 1, QTH = 3;
constexpr unsigned LT1 = 40, LT2 = 120;

constexpr size_t WS_TMO = 0;
constexpr size_t WS_LAST = 256;
constexpr size_t WS_DONE = 1024;
constexpr size_t WS_GLB = 4096;
constexpr size_t WS_LOC = 8192;
constexpr size_t WS_XF = 16384;
constexpr size_t WS_YF = 32768;
constexpr size_t WS_CNT = 65536;
constexpr size_t WS_XS = 2u << 20;
constexpr int PCS = 2048;
constexpr size_t XS_SLOT = (size_t)NT * 32 * PCS;
constexpr size_t WS_YS = WS_XS + NR * XS_SLOT;
constexpr int YS_WAVE = 17 * 1024;
constexpr size_t YS_TILE = 4 * YS_WAVE;
constexpr size_t WS_END = WS_YS + (size_t)2 * NR * TPP * YS_TILE;
static_assert(NF <= TPP && WS_YF + NR * TPP * 256 <= WS_CNT && WS_CNT + NWG * 256 <= (1u << 20), "layout");

struct Args {
  unsigned char* ws[NR];
  unsigned char* out[NR];
  const u16* x; const u16* w;
  int rank; int pad;
};
}

#define BAR() do { asm volatile("" ::: "memory"); __builtin_amdgcn_s_barrier(); asm volatile("" ::: "memory"); } while (0)
#define WAIT_V(n) asm volatile("s_waitcnt vmcnt(%0)" ::"n"(n) : "memory")
#define WAIT_L(n) asm volatile("s_waitcnt lgkmcnt(%0)" ::"n"(n) : "memory")
#define SCHED() __builtin_amdgcn_sched_barrier(0)
#define RLX_SYS __ATOMIC_RELAXED, __HIP_MEMORY_SCOPE_SYSTEM
#define RLX_AGT __ATOMIC_RELAXED, __HIP_MEMORY_SCOPE_AGENT
#define NOW() __builtin_amdgcn_s_memrealtime()

template <int CTRL> __device__ __forceinline__ unsigned dpp_umax(unsigned v) {
  const unsigned t = __builtin_amdgcn_update_dpp(v, v, CTRL, 0xf, 0xf, false);
  return t > v ? t : v;
}
__device__ __forceinline__ unsigned row_umax(unsigned v) {
  v = dpp_umax<0xB1>(v); v = dpp_umax<0x4E>(v); v = dpp_umax<0x141>(v); return dpp_umax<0x140>(v);
}
__device__ __forceinline__ unsigned wave_umax(unsigned v) {
  v = row_umax(v);
  const unsigned a = __builtin_amdgcn_readlane(v, 0), b = __builtin_amdgcn_readlane(v, 16), c = __builtin_amdgcn_readlane(v, 32), d = __builtin_amdgcn_readlane(v, 48);
  return max(max(a, b), max(c, d));
}

__device__ __forceinline__ unsigned vm_busy() { return __builtin_amdgcn_s_getreg(7 | (31 << 11)) & 0x00C0000Fu; }

__global__ __launch_bounds__(512, 2) void fused_gemm_a2a(Args A) {
  __shared__ __attribute__((aligned(16))) char smem[LDS_BYTES];
  LAS char* lds = (LAS char*)smem;
  const int tid = threadIdx.x, wid = __builtin_amdgcn_readfirstlane(tid >> 6), lane = tid & 63;
  const int bid = blockIdx.x, xcd = bid & 7, idx = (bid >> 3) & 31;
  const int rank = A.rank;
  int src = rank, half = 0, drel = 0, tc;
  if (idx < 14) { const int a = xcd * 14 + idx; drel = 1 + (a >> 4); tc = a & 15; }
  else if (idx < 16) tc = xcd * 2 + (idx - 14);
  else { const int k = idx - 16; src = (rank + (xcd & 1) * 4 + (k >> 2)) & 7; half = 1; tc = (xcd >> 1) * 4 + (k & 3); }
  const bool remote_x = src != rank;
  const int n0 = ((rank + drel) & 7) * NS + tc * BN;
  const int koff = drel ? xcd * 8 : 0;
  auto early_index = [](int c) { return (c & 3) == 1 ? 12 + (c >> 2) : c - ((c + 2) >> 2); };
  const int fidx = early_index(tc);
  const bool early = drel != 0 && fidx < NF;
  const int tile = tc;
  unsigned char* myws = A.ws[0];
  LAS unsigned* red = (LAS unsigned*)lds;

  if (wid >= 4) {
    if (wid < 6) {
      const int j = wid - 4;
      LAS char* base = lds + X_OFF + j * 16384;
      if (!remote_x) {
        auto rs = __builtin_amdgcn_make_buffer_rsrc((void*)(A.x + (size_t)(half * BM + j * 128) * KG), 0, 128 * KG * 2, 0x00020000);
        const int v0 = (lane >> 3) * (KG * 2) + (((lane & 7) ^ (lane >> 4)) << 4);
        const int v1 = (lane >> 3) * (KG * 2) + (((lane & 7) ^ (4 | (lane >> 4))) << 4);
        auto issue = [&](int t, int slot) {
#pragma unroll
          for (int i = 0; i < 16; ++i)
            __builtin_amdgcn_raw_ptr_buffer_load_lds(rs, (LAS void*)(base + slot * XB + i * 1024), 16, (i & 1) ? v1 : v0, i * 8 * (KG * 2) + ((t + koff) & 63) * 128, 0, 0);
        };
        issue(0, 0); issue(1, 1);
        int slot = 2;
        WAIT_V(16); BAR();
        for (int t = 0; t < NT - 1; ++t) {
          if (t + 2 < NT) { issue(t + 2, slot); WAIT_V(16); } else WAIT_V(0);
          slot = slot == NXS - 1 ? 0 : slot + 1;
          BAR();
        }
      } else {
        const unsigned char* sb = myws + WS_XS + (size_t)src * XS_SLOT + j * 16 * PCS;
        const unsigned vo = lane * 16 + (lane < 56 ? 512 : 640);
        auto issue = [&](int t, int slot) {
#pragma unroll
          for (int i = 0; i < 16; ++i)
            __builtin_amdgcn_global_load_lds((const GAS void*)(sb + (size_t)((t * 32 + i) * PCS) + vo), (LAS void*)(base + slot * XB + i * 1024), 16, 0, 17);
        };
        BAR();
        issue(0, 0); issue(1, 1);
        int slot = 2;
        WAIT_V(16); BAR();
        for (int t = 0; t < NT - 1; ++t) {
          if (t + 2 < NT) { issue(t + 2, slot); WAIT_V(16); } else WAIT_V(0);
          slot = slot == NXS - 1 ? 0 : slot + 1;
          BAR();
        }
      }
      return;
    }
    if (wid == 6) {
      auto rs = __builtin_amdgcn_make_buffer_rsrc((void*)(A.w + n0), 0, (unsigned)((size_t)KG * NG * 2 - (size_t)n0 * 2), 0x00020000);
      const int v0 = (lane >> 4) * (NG * 2) + (((lane & 15) ^ ((lane >> 4) << 1)) << 4);
      const int v1 = (lane >> 4) * (NG * 2) + (((lane & 15) ^ ((4 | (lane >> 4)) << 1)) << 4);
      auto issue = [&](int t, int slot, bool stream) {
        if (stream) {
#pragma unroll
          for (int i = 0; i < 16; ++i)
            __builtin_amdgcn_raw_ptr_buffer_load_lds(rs, (LAS void*)(lds + slot * WB + i * 1024), 16, (i & 2) ? v1 : v0, (((t + koff) & 63) * 64 + i * 4) * (NG * 2), 0, 2);
        } else {
#pragma unroll
          for (int i = 0; i < 16; ++i)
            __builtin_amdgcn_raw_ptr_buffer_load_lds(rs, (LAS void*)(lds + slot * WB + i * 1024), 16, (i & 2) ? v1 : v0, (((t + koff) & 63) * 64 + i * 4) * (NG * 2), 0, 0);
        }
      };
      if (half && ((idx + 4) & 15) < NR) {
        const int job = (idx + 4) & 7;
        const unsigned j = lane & 7;
        if (job != 0) {
          const size_t ro = j == 0 ? 0 : j < 4 ? WS_XS + rank * XS_SLOT + (j - 1) * (XS_SLOT / 2 - 64) : WS_YS + (size_t)(((j >> 1) & 1) * NR + rank) * TPP * YS_TILE + (j & 1) * (TPP * YS_TILE - 64);
          asm volatile("global_load_dword v253, %0, off sc0 sc1" ::"v"(lane & 8 ? A.out[job] + (size_t)rank * MS * NS + (lane & 1) * (BM * NS - 64) : A.ws[job] + ro) : "memory", "v253");
        } else {
          asm volatile("global_load_dword v253, %0, off sc0 sc1" ::"v"(lane < 56 ? myws + (size_t)lane * (WS_END / 56 & ~(size_t)63) : A.out[0] + (size_t)(lane - 56) * (1u << 20)) : "memory", "v253");
          asm volatile("global_load_dword v253, %0, off" ::"v"((const char*)A.w + (size_t)lane * (2u << 20)) : "memory", "v253");
        }
      }
      if (remote_x) BAR();
      issue(0, 0, false); issue(1, 1, false);
      unsigned last;
      asm volatile("global_load_dword %0, %1, off sc1" : "=v"(last) : "v"(myws + WS_LAST) : "memory");
      WAIT_V(16); BAR();
      __builtin_amdgcn_s_sleep(WSL);
      issue(2, 2, false); issue(3, 3, false);
      asm volatile("s_waitcnt vmcnt(32)" : "+v"(last) :: "memory");
      BAR();
      const bool stream = drel != 0 && (unsigned)NOW() - (unsigned)__builtin_amdgcn_readfirstlane(last) > 30000u;
      WAIT_V(16); BAR();
      auto run = [&](bool st) {
        for (int t = 2; t < NT - 2; t += 2) {
          if (fidx >= 11) __builtin_amdgcn_s_sleep(WSLS); else if (early) __builtin_amdgcn_s_sleep(WSLF); else __builtin_amdgcn_s_sleep(WSL);
          issue(t + 2, (t + 2) & 3, st); issue(t + 3, (t + 3) & 3, st);
          WAIT_V(32); BAR();
          WAIT_V(16); BAR();
        }
      };
      if (stream) run(true); else run(false);
      WAIT_V(0); BAR();
      return;
    }
    const u64 t0 = NOW();
    GAS unsigned* cnt = (GAS unsigned*)(myws + WS_CNT + bid * 256);
    const unsigned one = 1;
    u64 ex;
    GAS unsigned* tmo = (GAS unsigned*)(myws + WS_TMO);
    const GAS void* tab = (const GAS char*)(const GAS void*)__builtin_amdgcn_kernarg_segment_ptr() + (lane >> 3) * 8;
    const int first = drel ? POFF : half ? POFFB : POFFB - SBL, last = early ? TF0 + fidx * TFD : TEND;
    const int pace = (last - first) / (NT - 1);
    const int soft = drel != 0 ? (fidx >= 11 ? SOFTS : fidx * SOFTF) : SOFT;
    auto due_at = [&](int t) { return t0 + (unsigned)(first + t * pace + (t < SP ? t * soft / SP : (63 - t) * soft / (63 - SP))) / 10u; };
    unsigned call;
    unsigned char* pws;
    bool bad = false;
    if (!half) {
      {
        const char* tp = (const char*)A.x + (size_t)(idx * 16 + (lane & 7)) * (KG * 2) + (xcd * 8 + (lane >> 3)) * 128;
        asm volatile("global_load_dword v253, %0, off\n\tglobal_load_dword v253, %1, off" ::"v"(tp), "v"(tp + 8 * KG * 2) : "memory", "v253");
      }
      for (int t = 0; t < NT; ++t) {
        const u64 due = t ? due_at(t) : 0;
        while (NOW() < due) __builtin_amdgcn_s_sleep(1);
        BAR();
        if (t == 0) {
          __builtin_amdgcn_s_sleep(8);
          asm volatile("global_load_dwordx2 v[250:251], %1, off\n\ts_mov_b64 %0, exec\n\ts_mov_b64 exec, 1\n\tglobal_atomic_add v252, %2, %3, off sc0\n\ts_mov_b64 exec, %0"
                       : "=&s"(ex) : "v"(tab), "v"(cnt), "v"(one) : "memory", "v250", "v251", "v252");
        }
      }
      unsigned plo, phi, c0;
      asm volatile("s_waitcnt vmcnt(0)\n\tv_mov_b32 %0, v250\n\tv_mov_b32 %1, v251\n\tv_mov_b32 %2, v252\n\ts_nop 0" : "=v"(plo), "=v"(phi), "=v"(c0) :: "memory");
      call = __builtin_amdgcn_readfirstlane(c0) + 1u;
      pws = (unsigned char*)(((u64)phi << 32) | plo);
    } else {
      const int l = ((idx - 4) & 15) * 8 + xcd, pg = l & 15, pj = l >> 4;
      const char* s0 = (const char*)A.x + (size_t)(BM + pg * 16 + (lane >> 3)) * (KG * 2) + (((lane & 7) ^ (lane >> 4)) << 4) + pj * 128;
      const char* s1 = (const char*)A.x + (size_t)(BM + pg * 16 + 8 + (lane >> 3)) * (KG * 2) + (((lane & 7) ^ (4 | (lane >> 4))) << 4) + pj * 128;
      u32x4 r[16];
      unsigned c0;
      asm volatile("global_load_dwordx4 %0, %5, off\n\tglobal_load_dwordx4 %1, %6, off\n\tglobal_load_dwordx2 %3, %8, off\n\t"
                   "s_mov_b64 %4, exec\n\ts_mov_b64 exec, 1\n\tglobal_atomic_add %2, %7, %9, off sc0\n\ts_mov_b64 exec, %4\n\ts_waitcnt vmcnt(0)"
                   : "=&v"(r[0]), "=&v"(r[1]), "=&v"(c0), "=&v"(pws), "=&s"(ex) : "v"(s0), "v"(s1), "v"(cnt), "v"(tab), "v"(one) : "memory");
      call = __builtin_amdgcn_readfirstlane(c0) + 1u;
      const bool flagger = lane >= 8 && (lane & 7) == 0;
      const unsigned xs_mine = (unsigned)(WS_XS + (size_t)rank * XS_SLOT) + lane * 16 + (lane < 56 ? 512 : 640) + (pj * 32 + pg * 2) * PCS;
      GAS unsigned* fp = (GAS unsigned*)(pws + WS_XF + rank * 1024 + (pj >> 1) * 256 + (pj & 1) * 64) + pg;
      const GAS unsigned char* xf = (const GAS unsigned char*)(myws + WS_XF + src * 1024) + (lane & 7) * 8;
      int nb = 0;
      bool stored = false, polled = false;
      unsigned pbase = 0, ready = remote_x ? 0 : NT;
      auto burst = [&]() {
        const unsigned vo = xs_mine + nb * 8 * 32 * PCS;
#pragma unroll
        for (int d = 1; d < NR; ++d) {
          asm volatile("global_store_dwordx4 %0, %1, off sc0 sc1\n\ts_nop 1" ::"v"(A.ws[d] + vo), "v"(r[0]) : "memory");
          asm volatile("global_store_dwordx4 %0, %1, off offset:%c2 sc0 sc1\n\ts_nop 1" ::"v"(A.ws[d] + vo), "v"(r[1]), "i"(PCS) : "memory");
        }
        stored = true;
      };
      auto burst_due = [&]() { const int c = 8 * nb + pj; return NOW() >= t0 + (unsigned)(EOFF + (c > NE ? NE * TAUE + (c - NE) * TAU + pg * STG : c * TAUE + (c > 2 ? pg * STG : 0))) / 10u; };
      auto poll = [&]() {
        pbase = ready;
        asm volatile("global_load_dwordx2 v[250:251], %0, off sc0 sc1" :: "v"(xf + ((pbase + (lane >> 3)) & 6) * 128 + ((pbase + (lane >> 3)) & 1) * 64) : "memory", "v250", "v251");
        polled = true;
      };
      auto settle = [&]() {
        unsigned p0, p1;
        asm volatile("s_waitcnt vmcnt(0)\n\tv_mov_b32 %0, v250\n\tv_mov_b32 %1, v251" : "=v"(p0), "=v"(p1) :: "memory");
        if (stored) {
#pragma unroll
          for (int i = 0; i < 14; ++i) r[i] = r[i + 2];
          ++nb;
          if (flagger) __hip_atomic_store(fp, call * 16u + (unsigned)nb, RLX_SYS);
          stored = false;
        }
        if (polled) {
          const unsigned want = call * 16u + ((pbase + (lane >> 3)) >> 3) + 1u;
          const u64 miss = ~__ballot(p0 >= want && p1 >= want);
          ready = pbase + (miss ? (unsigned)__builtin_ctzll(miss) >> 3 : 8u);
          polled = false;
        }
      };
      auto gate = [&](unsigned need) {
        while (ready < need) {
          if (!stored && nb < 8 && burst_due()) burst();
          if (!polled) poll();
          settle();
          if (ready < need && (__hip_atomic_load(tmo, RLX_AGT) == call || NOW() - t0 > DEADLINE)) { bad = true; ready = NT; }
        }
      };
      auto load_rest = [&]() {
        asm volatile("global_load_dwordx4 %0, %6, off offset:1024\n\tglobal_load_dwordx4 %1, %7, off offset:1024\n\tglobal_load_dwordx4 %2, %6, off offset:2048\n\t"
                     "global_load_dwordx4 %3, %7, off offset:2048\n\tglobal_load_dwordx4 %4, %6, off offset:3072\n\tglobal_load_dwordx4 %5, %7, off offset:3072\n\ts_waitcnt vmcnt(0)"
                     : "=&v"(r[2]), "=&v"(r[3]), "=&v"(r[4]), "=&v"(r[5]), "=&v"(r[6]), "=&v"(r[7]) : "v"(s0), "v"(s1) : "memory");
        asm volatile("global_load_dwordx4 %0, %8, off\n\tglobal_load_dwordx4 %1, %9, off\n\tglobal_load_dwordx4 %2, %8, off offset:1024\n\tglobal_load_dwordx4 %3, %9, off offset:1024\n\t"
                     "global_load_dwordx4 %4, %8, off offset:2048\n\tglobal_load_dwordx4 %5, %9, off offset:2048\n\tglobal_load_dwordx4 %6, %8, off offset:3072\n\tglobal_load_dwordx4 %7, %9, off offset:3072\n\t"
                     "s_waitcnt vmcnt(0)"
                     : "=&v"(r[8]), "=&v"(r[9]), "=&v"(r[10]), "=&v"(r[11]), "=&v"(r[12]), "=&v"(r[13]), "=&v"(r[14]), "=&v"(r[15]) : "v"(s0 + 4096), "v"(s1 + 4096) : "memory");
      };
      auto wait_for = [&](int t) {
        for (const u64 due = due_at(t); NOW() < due;) {
          if (!stored && nb < 8 && burst_due()) burst();
          if (stored && !polled && !vm_busy()) settle();
          __builtin_amdgcn_s_sleep(1);
        }
      };
      if (pj < 3) {
        while (!burst_due()) __builtin_amdgcn_s_sleep(1);
        burst();
        WAIT_V(0);
        if (flagger) __hip_atomic_store(fp, call * 16u + 1u, RLX_SYS);
      }
      load_rest();
      settle();
      if (remote_x) { gate(2); BAR(); }
      wait_for(0);
      for (int t = 0; t < NT; ++t) {
        const unsigned need = t + 3 < NT ? t + 3 : NT;
        gate(need);
        wait_for(t);
        BAR();
        const int c = 8 * nb + pj;
        if (nb < 8 || ready < (unsigned)NT) {
          __builtin_amdgcn_s_sleep(DSL);
          if ((stored || polled) && (!vm_busy() || (stored && t + 4 >= c))) settle();
          else if ((t & 1) == 0) {
            if (!stored && nb < 8 && (t + 7 >= c || burst_due())) burst();
            if (!polled && ready < (unsigned)NT && ready < need + LEAD) poll();
          }
        }
      }
      settle();
      while (nb < 8) { burst(); settle(); }
    }
    const unsigned par = call & 1u;
    if (bad && lane == 0) __hip_atomic_store(tmo, call, RLX_AGT);
    GAS u64* gp = (GAS u64*)(pws + WS_GLB + (par * 8 + (lane & 7)) * 256) + rank;
    const unsigned late = (unsigned)(NOW() - due_at(NT - 1));
    if (lane == 0) red[6] = par | (late > LT2 ? 4u : late > LT1 ? 2u : 0u);
    WAIT_L(0);
    BAR();
    {
      const unsigned m = dpp_umax<0x4E>(dpp_umax<0xB1>(red[lane & 3]));
      if (lane == 0) __hip_atomic_store((GAS u64*)(myws + WS_LOC + (par * 16 + (bid >> 4)) * 256) + (bid & 15), ((u64)call << 32) | (bad ? 0x7fc00000u : m), RLX_AGT);
    }
    if (early) {
      BAR();
      if (lane == 0) __hip_atomic_store((GAS unsigned*)(A.ws[drel] + WS_YF + (rank * TPP + tile) * 256), call, RLX_SYS);
      GAS unsigned* f = lane == 1 ? tmo : (GAS unsigned*)(myws + WS_YF + (((rank - drel) & 7) * TPP + tile) * 256);
      for (;;) {
        const unsigned v = __hip_atomic_load(f, RLX_SYS);
        if ((unsigned)__builtin_amdgcn_readlane(v, 0) >= call) break;
        if ((unsigned)__builtin_amdgcn_readlane(v, 1) == call || NOW() - t0 > DEADLINE) { bad = true; break; }
        __builtin_amdgcn_s_sleep(2);
      }
      if (lane == 0) red[5] = bad;
      WAIT_L(0);
      BAR();
    }
    if (bid == 14 * 8) {
      GAS u64* g = (GAS u64*)(myws + WS_LOC + (par * 16 + (lane >> 4)) * 256) + (lane & 15);
      unsigned m;
      for (;;) {
        const u64 a = __hip_atomic_load(g, RLX_AGT), b = __hip_atomic_load(g + 128, RLX_AGT), c = __hip_atomic_load(g + 256, RLX_AGT), d = __hip_atomic_load(g + 384, RLX_AGT);
        const bool ok = (unsigned)(a >> 32) == call && (unsigned)(b >> 32) == call && (unsigned)(c >> 32) == call && (unsigned)(d >> 32) == call;
        m = max(max((unsigned)a, (unsigned)b), max((unsigned)c, (unsigned)d));
        if (__all(ok)) break;
        if (NOW() - t0 > DEADLINE) { bad = true; break; }
      }
      m = wave_umax(m);
      __hip_atomic_store(gp, ((u64)call << 32) | (bad ? 0x7fc00000u : m), RLX_SYS);
    }
    {
      GAS u64* g = lane == 8 ? (GAS u64*)tmo : (GAS u64*)(myws + WS_GLB + (par * 8 + xcd) * 256) + (lane & 7);
      unsigned gm;
      for (;;) {
        const u64 a = __hip_atomic_load(g, RLX_SYS);
        gm = lane == 8 ? 0u : (unsigned)a;
        if (__builtin_amdgcn_readlane((unsigned)a, 8) == call) { bad = true; break; }
        if (__all(lane == 8 || (unsigned)(a >> 32) >= call)) {
          bad = bad || __any(lane != 8 && (unsigned)(a >> 32) != call);
          break;
        }
        if (bad || NOW() - t0 > DEADLINE) { bad = true; break; }
        __builtin_amdgcn_s_sleep(1);
      }
      gm = row_umax(gm);
      if (lane == 0) red[4] = bad ? 0x7fc00000u : __float_as_uint(448.0f / __uint_as_float(gm));
      if (bad && lane == 0) __hip_atomic_store(tmo, call, RLX_AGT);
    }
    WAIT_L(0);
    BAR();
    if (bid == 15 * 8 && lane == 0) __hip_atomic_store((GAS unsigned*)(myws + WS_LAST), (unsigned)NOW(), RLX_AGT);
    BAR();
    if (drel != 0 && !early && lane == 0)
      __hip_atomic_store((GAS unsigned*)(A.ws[drel] + WS_DONE + (rank >> 1) * 256 + (rank & 1) * 64) + tile, call, RLX_SYS);
    if (bid == 15 * 8) {
      const bool mine = (lane >> 3) == rank;
      GAS u64* f = mine ? (GAS u64*)tmo : (GAS u64*)(myws + WS_DONE + (lane >> 4) * 256) + (lane & 15);
      const bool e0 = early_index(2 * (lane & 7)) < NF, e1 = early_index(2 * (lane & 7) + 1) < NF;
      for (;;) {
        const u64 a = __hip_atomic_load(f, RLX_SYS);
        const bool ok = mine || ((e0 || (unsigned)a >= call) && (e1 || (unsigned)(a >> 32) >= call));
        if (__all(ok)) break;
        if (__any(mine && (unsigned)a == call) || NOW() - t0 > DEADLINE) {
          if (lane == 0) { __hip_atomic_store(tmo, call, RLX_AGT); A.out[0][(size_t)(rank * MS + half * BM) * NS + tc * BN] = 0x7f; }
          break;
        }
      }
    }
    return;
  }

  const int wm = wid >> 1, wn = wid & 1;
  const int fr = lane & 15, g = lane >> 4;
  const int xa0 = X_OFF + (wm * 128 + fr) * 128 + ((g ^ ((fr >> 1) & 7)) << 4);
  const int xa1 = xa0 ^ 64;
  const int q = fr >> 2, p = fr & 3, h = q | ((g & 1) << 2);
  int wa[4];
#pragma unroll
  for (int nt = 0; nt < 4; ++nt) wa[nt] = (8 * g + q) * 256 + (((4 * wn + nt) ^ h) << 5) + p * 8;
  f32x4 acc[8][4];
#pragma unroll
  for (int a = 0; a < 8; ++a)
#pragma unroll
    for (int b = 0; b < 4; ++b) { acc[a][b] = (f32x4){0.f, 0.f, 0.f, 0.f}; asm volatile("" : "+v"(acc[a][b])); }
  s16x4 WA[4][4], WC[4][4];
  bf16x8 X[8][2];
#define MF(a, b, Wf, ks) do { if (!skip) { const bf16x8 wv = __builtin_bit_cast(bf16x8, __builtin_shufflevector(Wf[b][2 * ks], Wf[b][2 * ks + 1], 0, 1, 2, 3, 4, 5, 6, 7)); \
    asm volatile("v_mfma_f32_16x16x32_bf16 %0, %1, %2, %0" : "+v"(acc[a][b]) : "v"(wv), "v"(X[a][ks])); } SCHED(); } while (0)
#define RW(Wf, b, i) do { Wf[b][i] = __builtin_amdgcn_ds_read_tr16_b64_v4i16((LAS s16x4*)(lds + wp[b] + wimm + (i >> 1) * 8192 + (i & 1) * 1024)); SCHED(); } while (0)
#define RX(a, ks) do { X[a][ks] = *(LAS bf16x8*)(lds + (ks ? xp1 : xp0) + a * 2048); SCHED(); } while (0)
#define GRP(a, r0, r1, r2, r3) MF(a, 0, Wc, 0); r0; MF(a, 0, Wc, 1); MF(a, 1, Wc, 0); r1; MF(a, 1, Wc, 1); MF(a, 2, Wc, 0); r2; MF(a, 2, Wc, 1); MF(a, 3, Wc, 0); r3; MF(a, 3, Wc, 1);
  auto step = [&](s16x4 (&Wp)[4][4], s16x4 (&Wc)[4][4], int xoff, const int (&wp)[4], int wimm, bool first) {
    bool skip = first;
    const int xp0 = xa0 + xoff, xp1 = xa1 + xoff;
    MF(5, 0, Wp, 0); RW(Wc, 0, 0); MF(5, 0, Wp, 1); RW(Wc, 0, 1); MF(5, 1, Wp, 0); RW(Wc, 0, 2); MF(5, 1, Wp, 1); RW(Wc, 0, 3);
    MF(5, 2, Wp, 0); RW(Wc, 1, 0); MF(5, 2, Wp, 1); RW(Wc, 1, 1); MF(5, 3, Wp, 0); RW(Wc, 1, 2); MF(5, 3, Wp, 1); RW(Wc, 1, 3);
    MF(6, 0, Wp, 0); RW(Wc, 2, 0); MF(6, 0, Wp, 1); RW(Wc, 2, 1); MF(6, 1, Wp, 0); RW(Wc, 2, 2); MF(6, 1, Wp, 1); RW(Wc, 2, 3);
    MF(6, 2, Wp, 0); RW(Wc, 3, 0); MF(6, 2, Wp, 1); RW(Wc, 3, 1); MF(6, 3, Wp, 0); RW(Wc, 3, 2); MF(6, 3, Wp, 1); RW(Wc, 3, 3);
    MF(7, 0, Wp, 0); RX(0, 0); MF(7, 0, Wp, 1); RX(0, 1); MF(7, 1, Wp, 0); MF(7, 1, Wp, 1); RX(1, 0);
    MF(7, 2, Wp, 0); MF(7, 2, Wp, 1); RX(1, 1); MF(7, 3, Wp, 0); MF(7, 3, Wp, 1);
    skip = false;
    GRP(0, RX(2, 0), RX(2, 1), , )
    GRP(1, RX(3, 0), RX(3, 1), , )
    GRP(2, RX(4, 0), RX(4, 1), RX(5, 0), RX(5, 1))
    GRP(3, RX(6, 0), RX(6, 1), RX(7, 0), RX(7, 1))
    GRP(4, , , , )
  };
  if (remote_x) BAR();
  BAR();
  SCHED();
  step(WC, WA, 0, wa, 0, true);
  int xoff = 0;
  for (int t = 1; t < NT - 1; t += 2) {
    xoff = xoff == (NXS - 1) * XB ? 0 : xoff + XB;
    WAIT_L(0); BAR(); SCHED();
    step(WA, WC, xoff, wa, WB, false);
    xoff = xoff == (NXS - 1) * XB ? 0 : xoff + XB;
#pragma unroll
    for (int b = 0; b < 4; ++b) wa[b] ^= 2 * WB;
    WAIT_L(0); BAR(); SCHED();
    step(WC, WA, xoff, wa, 0, false);
  }
  xoff = xoff == (NXS - 1) * XB ? 0 : xoff + XB;
  WAIT_L(0); BAR(); SCHED();
  step(WA, WC, xoff, wa, WB, false);
  float mx = 0.f;
#define MX(a, b) do { asm volatile("v_max3_f32 %0, |%1|, |%2|, %0\n\tv_max3_f32 %0, |%3|, |%4|, %0" : "+v"(mx) : "v"(acc[a][b][0]), "v"(acc[a][b][1]), "v"(acc[a][b][2]), "v"(acc[a][b][3])); SCHED(); } while (0)
#define MX2(a, b) MX(a, b); MX(a + 1, b)
  {
    const bool skip = false;
    auto& Wc = WC;
    GRP(5, MX2(0, 0), MX2(0, 1), MX2(0, 2), MX2(0, 3))
    GRP(6, MX2(2, 0), MX2(2, 1), MX2(2, 2), MX2(2, 3))
    GRP(7, MX2(4, 0), MX2(4, 1), MX2(4, 2), MX2(4, 3))
    MX(6, 0); MX(6, 1); MX(6, 2); MX(6, 3);
    asm volatile("s_nop 7" ::: "memory");
    MX(7, 0); MX(7, 1); MX(7, 2); MX(7, 3);
    asm volatile("s_nop 1" : "+v"(mx));
  }
  {
    const unsigned m = wave_umax(__float_as_uint(mx));
    if (lane == 0) red[wid] = m;
  }
  WAIT_L(0);
  BAR();
  const unsigned lo = (unsigned)((wm * 128 + fr) * NS + wn * 64 + g * 16);
  unsigned char* op = A.out[drel] + (size_t)(src * MS + half * BM) * NS + tc * BN + lo;
  if (early) {
    const unsigned lvl = __builtin_amdgcn_readfirstlane(red[6]) >> 1;
    const size_t slot = WS_YS + (size_t)((red[6] & 1u) * (NR * TPP) + tile) * YS_TILE + wid * YS_WAVE + lane * 16;
    unsigned char* sp = A.ws[drel] + slot + (size_t)rank * TPP * YS_TILE;
#define PUT(off, val) do { const u32x4 o_ = val; asm volatile("global_store_dwordx4 %0, %1, off offset:%c2 sc0 sc1\n\ts_nop 1" ::"v"(sp + c * 4096), "v"(o_), "i"(off) : "memory"); if (lvl == 0) WAIT_V(FTH); else if (lvl == 1) WAIT_V(FTH1); else WAIT_V(FTH2); } while (0)
    unsigned head = (__builtin_bit_cast(u16, (__bf16)mx) >> 8) & 0x7fu;
    const unsigned brep = head * 0x01010101u | 0x80808080u;
#pragma unroll
    for (int c = 0; c < 4; ++c) {
      unsigned L[8], H[8], N[4], ovf = 0;
#pragma unroll
      for (int j = 0; j < 8; ++j) {
        const f32x4 v = acc[2 * c + (j >> 2)][j & 3];
        typedef __bf16 bf16x2 __attribute__((ext_vector_type(2)));
        const unsigned p0 = __builtin_bit_cast(unsigned, (bf16x2){(__bf16)v[0], (__bf16)v[1]}), p1 = __builtin_bit_cast(unsigned, (bf16x2){(__bf16)v[2], (__bf16)v[3]});
        L[j] = __builtin_amdgcn_perm(p1, p0, 0x06040200u);
        H[j] = __builtin_amdgcn_perm(p1, p0, 0x07050301u);
      }
      PUT(0, ((u32x4){L[0], L[1], L[2], L[3]}));
      PUT(1024, ((u32x4){L[4], L[5], L[6], L[7]}));
#pragma unroll
      for (int k = 0; k < 4; ++k) {
        const unsigned d0 = brep - (H[2 * k] & 0x7f7f7f7fu), d1 = brep - (H[2 * k + 1] & 0x7f7f7f7fu);
        ovf |= ((d0 ^ 0x80808080u) | (d1 ^ 0x80808080u)) & 0xf8f8f8f8u;
        N[k] = (d0 & 0x07070707u) | ((H[2 * k] & 0x80808080u) >> 4) | ((d1 & 0x07070707u) << 4) | (H[2 * k + 1] & 0x80808080u);
      }
      if (__any(ovf != 0)) {
        PUT(2048, ((u32x4){H[0], H[1], H[2], H[3]}));
        PUT(3072, ((u32x4){H[4], H[5], H[6], H[7]}));
        head |= 256u << c;
      } else PUT(2048, ((u32x4){N[0], N[1], N[2], N[3]}));
    }
    asm volatile("global_store_dword %0, %1, off sc0 sc1" ::"v"(sp + 16384 - lane * 12), "v"(head) : "memory");
    WAIT_V(0);
    BAR();
    BAR();
    const int from = (rank - drel) & 7;
    op = A.out[0] + (size_t)(from * MS + half * BM) * NS + tc * BN + lo;
    if (__builtin_amdgcn_readfirstlane(red[5]) == 0) {
      const unsigned char* lp = myws + slot + (size_t)from * TPP * YS_TILE;
      u32x4 r[12];
      asm volatile("global_load_dword %0, %13, off sc0 sc1\n\t"
                   "global_load_dwordx4 %1, %14, off offset:-4096 sc0 sc1\n\tglobal_load_dwordx4 %2, %14, off offset:-3072 sc0 sc1\n\tglobal_load_dwordx4 %3, %14, off offset:-2048 sc0 sc1\n\t"
                   "global_load_dwordx4 %4, %14, off sc0 sc1\n\tglobal_load_dwordx4 %5, %14, off offset:1024 sc0 sc1\n\tglobal_load_dwordx4 %6, %14, off offset:2048 sc0 sc1\n\t"
                   "global_load_dwordx4 %7, %15, off offset:-4096 sc0 sc1\n\tglobal_load_dwordx4 %8, %15, off offset:-3072 sc0 sc1\n\tglobal_load_dwordx4 %9, %15, off offset:-2048 sc0 sc1\n\t"
                   "global_load_dwordx4 %10, %15, off sc0 sc1\n\tglobal_load_dwordx4 %11, %15, off offset:1024 sc0 sc1\n\tglobal_load_dwordx4 %12, %15, off offset:2048 sc0 sc1\n\ts_waitcnt vmcnt(0)"
                   : "=&v"(head), "=&v"(r[0]), "=&v"(r[1]), "=&v"(r[2]), "=&v"(r[3]), "=&v"(r[4]), "=&v"(r[5]), "=&v"(r[6]), "=&v"(r[7]), "=&v"(r[8]), "=&v"(r[9]), "=&v"(r[10]), "=&v"(r[11])
                   : "v"(lp + 16384 - lane * 12), "v"(lp + 4096), "v"(lp + 12288) : "memory");
      const unsigned raw = __builtin_amdgcn_readfirstlane(head) >> 8, base = (head & 0x7fu) * 0x01010101u;
#pragma unroll
      for (int c = 0; c < 4; ++c) {
        unsigned H[8];
        const u32x4 n4 = r[3 * c + 2];
        if ((raw >> c) & 1u) {
          u32x4 h2;
          asm volatile("global_load_dwordx4 %0, %1, off offset:3072 sc0 sc1\n\ts_waitcnt vmcnt(0)" : "=&v"(h2) : "v"(lp + c * 4096) : "memory");
#pragma unroll
          for (int k = 0; k < 4; ++k) { H[k] = n4[k]; H[4 + k] = h2[k]; }
        } else {
#pragma unroll
          for (int k = 0; k < 4; ++k) {
            H[2 * k] = (base - (n4[k] & 0x07070707u)) | ((n4[k] & 0x08080808u) << 4);
            H[2 * k + 1] = (base - ((n4[k] >> 4) & 0x07070707u)) | (n4[k] & 0x80808080u);
          }
        }
#pragma unroll
        for (int j = 0; j < 8; ++j) {
          const unsigned l = r[3 * c + (j >> 2)][j & 3];
          acc[2 * c + (j >> 2)][j & 3] = (f32x4){__uint_as_float(__builtin_amdgcn_perm(H[j], l, 0x04000c0cu)), __uint_as_float(__builtin_amdgcn_perm(H[j], l, 0x05010c0cu)),
                                                 __uint_as_float(__builtin_amdgcn_perm(H[j], l, 0x06020c0cu)), __uint_as_float(__builtin_amdgcn_perm(H[j], l, 0x07030c0cu))};
        }
      }
    }
  }
  BAR();
  const float inv = __uint_as_float(red[4]);
  const bool throttle = drel != 0 && !early;
#pragma unroll
  for (int a = 0; a < 8; ++a) {
    unsigned d[4];
#pragma unroll
    for (int b = 0; b < 4; ++b) {
      const f32x4 v = acc[a][b] * inv;
      int pk = __builtin_amdgcn_cvt_pk_fp8_f32(v[0], v[1], 0, false);
      pk = __builtin_amdgcn_cvt_pk_fp8_f32(v[2], v[3], pk, true);
      d[b] = pk;
    }
    { auto r = __builtin_amdgcn_permlane32_swap(d[0], d[2], false, false); d[0] = r[0]; d[2] = r[1]; }
    { auto r = __builtin_amdgcn_permlane32_swap(d[1], d[3], false, false); d[1] = r[0]; d[3] = r[1]; }
    { auto r = __builtin_amdgcn_permlane16_swap(d[0], d[1], false, false); d[0] = r[0]; d[1] = r[1]; }
    { auto r = __builtin_amdgcn_permlane16_swap(d[2], d[3], false, false); d[2] = r[0]; d[3] = r[1]; }
    const u32x4 o = {d[0], d[1], d[2], d[3]};
    asm volatile("global_store_dwordx4 %0, %1, off sc0 sc1\n\ts_nop 1" ::"v"(op + (size_t)a * 16 * NS), "v"(o) : "memory");
    if (throttle) WAIT_V(QTH);
  }
  WAIT_V(0);
  BAR();
}

extern "C" void kernel_launch(int rank, int nranks, ncclComm_t comm, hipStream_t stream,
                              void* const* d_in, const int* in_sizes, int n_in,
                              void* d_out, int out_size, const int* in_dtypes, int out_dtype,
                              void* const* d_out_all, void* const* d_ws_all, size_t ws_bytes) {
  (void)comm; (void)in_sizes; (void)n_in; (void)d_out; (void)out_size; (void)in_dtypes; (void)out_dtype;
  if (nranks != NR || (unsigned)rank >= (unsigned)NR || ws_bytes < WS_END) { fprintf(stderr, "unexpected configuration\n"); return; }
  Args a;
  a.x = (const u16*)d_in[0]; a.w = (const u16*)d_in[1];
  for (int i = 0; i < NR; ++i) { a.out[i] = (unsigned char*)d_out_all[(rank + i) & 7]; a.ws[i] = (unsigned char*)d_ws_all[(rank + i) & 7]; }
  a.rank = rank; a.pad = 0;
  hipLaunchKernelGGL(fused_gemm_a2a, dim3(NWG), dim3(512), 0, stream, a);
  HIP_CHECK(hipGetLastError());
}
```
